```python
import numpy as np

kernel_name = "gpu_gemm_a2a_m4096_k4096_n4096_bf16_reluq8"


ABSMAX_RTOL = 0.02

COMPUTE_DTYPE = "bfloat16"
EPILOGUE = "relu_quant_int8"
IN_QUANT = None
OUT_QUANT = "int8"
IN_DTYPES = {"x": "bfloat16", "w_mat": "bfloat16"}
WORLD_SIZE = 8

def _to_bf16(x):
    u = np.ascontiguousarray(x, dtype=np.float32).view(np.uint32)
    return ((u + 0x7FFF + ((u >> 16) & 1)) & 0xFFFF0000).view(np.float32)

def setup_inputs():

    x = _to_bf16(np.random.randn(4096, 4096).astype(np.float32))
    w_mat = _to_bf16((np.random.randn(4096, 4096) / np.sqrt(4096)).astype(np.float32))
    return {"x": x, "w_mat": w_mat}

def _quant_dequant(y, amax):
    scale = amax / 127.0
    q = np.clip(np.round(y / scale), -127.0, 127.0)
    return (q * scale).astype(np.float32)
def _epilogue(y):
    y = np.maximum(y, 0.0)
    return _quant_dequant(y, np.abs(y).max())

def reference(x, w_mat):
    y = (x @ w_mat).astype(np.float32)
    return _epilogue(y)
```

```cpp
#include <hip/hip_runtime.h>
#include <rccl/rccl.h>
#include <cstdio>

#define HIP_CHECK(expr) do { hipError_t _e = (expr); if (_e != hipSuccess) { \
    fprintf(stderr, "HIP error %s at %s:%d\n", hipGetErrorString(_e), __FILE__, __LINE__); \
    return; } } while (0)

namespace {
typedef unsigned short bf16_t;
typedef short bf16x8 __attribute__((ext_vector_type(8)));
typedef short bf16x4 __attribute__((ext_vector_type(4)));
typedef float f32x4 __attribute__((ext_vector_type(4)));
typedef unsigned u32x4 __attribute__((ext_vector_type(4)));
typedef unsigned u32x2 __attribute__((ext_vector_type(2)));
#define LAS __attribute__((address_space(3)))

constexpr int NR = 8;
constexpr int MS = 512, N = 4096, K = 4096;
constexpr int NCOL = N / NR;
constexpr int NWG = 256;
constexpr int XB = 8192, WB = 16384, SB = XB + WB, NS = 6, NK = K / 64;
constexpr int LDS_BYTES = NS * SB;
constexpr size_t WS_CNT = 0;
constexpr size_t WS_TMO = 4096;
constexpr size_t WS_GRAN = 8192;
constexpr size_t WS_GRAN_SET = (size_t)NR * NWG * 8;
constexpr size_t WS_STAMP = 65536;
constexpr size_t WS_NEED = WS_STAMP + 65536;
constexpr unsigned long long DEADLINE = 50000000ull;

struct Params {
  const bf16_t* x; const bf16_t* w;
  char* out[NR]; char* ws[NR];
  int rank; int pad;
};

#define DMA(m0v, voff, srd, soff) asm volatile("s_mov_b32 m0, %0\n\ts_nop 0\n\tbuffer_load_dwordx4 %1, %2, %3 offen lds" :: "s"(m0v), "v"(voff), "s"(srd), "s"(soff) : "memory")
template <int V> __device__ __forceinline__ void wait_vm() { asm volatile("s_waitcnt vmcnt(%0)" :: "i"(V) : "memory"); }
__device__ __forceinline__ void wait_lgkm0() { __builtin_amdgcn_s_waitcnt(0xC07F); }
__device__ __forceinline__ void bar() { asm volatile("" ::: "memory"); __builtin_amdgcn_s_barrier(); asm volatile("" ::: "memory"); }
__device__ __forceinline__ void st16_sys(void* p, u32x4 v) { asm volatile("global_store_dwordx4 %0, %1, off sc0 sc1\n\ts_nop 1" :: "v"(p), "v"(v) : "memory"); }
#define STAMP(i) do { } while (0)

struct Frags { bf16x8 a[4][2]; bf16x8 b[2][2]; };

__device__ __forceinline__ bf16x8 rd_b(LAS char* st, unsigned o0, unsigned o1, int ks) {
  bf16x4 lo = __builtin_amdgcn_ds_read_tr16_b64_v4i16((LAS bf16x4*)(st + o0 + ks * 8192));
  bf16x4 hi = __builtin_amdgcn_ds_read_tr16_b64_v4i16((LAS bf16x4*)(st + o1 + ks * 8192));
  return __builtin_shufflevector(lo, hi, 0, 1, 2, 3, 4, 5, 6, 7);
}
#define MFMA(acc, w_, x_) acc = __builtin_amdgcn_mfma_f32_16x16x32_bf16(w_, x_, acc, 0, 0, 0)
#define SB0() __builtin_amdgcn_sched_barrier(0)

__global__ __launch_bounds__(256, 2) void fused_kernel(Params P) {
  extern __shared__ __attribute__((aligned(16))) char smem_[];
  LAS char* smem = (LAS char*)smem_;
  const int tid = threadIdx.x, lane = tid & 63;
  const int wave = __builtin_amdgcn_readfirstlane(tid >> 6);
  const int b = blockIdx.x;
  const int grp = b & 7, j = b >> 3;
  const int rb = j & 7, cb = grp * 4 + (j >> 3);
  const int rank = P.rank;
  char* const myws = P.ws[rank];
  STAMP(0);

  const __amdgpu_buffer_rsrc_t sx = __builtin_amdgcn_make_buffer_rsrc((void*)(P.x + (size_t)rb * 64 * K), (short)0, 64 * K * 2, 0x00020000);
  const __amdgpu_buffer_rsrc_t sw = __builtin_amdgcn_make_buffer_rsrc((void*)(P.w + cb * 128), (short)0, K * N * 2 - cb * 256, 0x00020000);
  unsigned vx[2], vw[4];
#pragma unroll
  for (int i = 0; i < 2; ++i) { const int row = (wave * 2 + i) * 8 + (lane >> 3); vx[i] = row * (K * 2) + (((lane & 7) ^ ((row >> 1) & 7)) << 4); }
#pragma unroll
  for (int i = 0; i < 4; ++i) { const int row = (wave * 4 + i) * 4 + (lane >> 4); const int g = ((row & 3) << 2) | ((row >> 2) & 3); vw[i] = row * (N * 2) + (((lane & 15) ^ g) << 4); }
  const unsigned ldsbase = (unsigned)(size_t)smem;

  char* const myout = P.out[rank] + (size_t)b * 8192 + tid * 16;
  { const u32x4 mk = {0x80808080u, 0x80808080u, 0x80808080u, 0x80808080u};
    st16_sys(myout, mk); st16_sys(myout + 4096, mk); }
  unsigned call;
  { unsigned* cnt = (unsigned*)(myws + WS_CNT) + b;
    asm volatile("global_load_dword %0, %1, off sc0 sc1" : "=v"(call) : "v"(cnt) : "memory"); }

  auto issue = [&](int i, unsigned stage_off, int kt) {
    const unsigned mx = ldsbase + stage_off + wave * 2048, mw = ldsbase + stage_off + XB + wave * 4096;
    const unsigned sox = kt * 128, sow = kt * (64 * N * 2);
    if (i == 0) DMA(mw, vw[0], sw, sow);
    if (i == 1) DMA(mw + 1024, vw[1], sw, sow);
    if (i == 2) DMA(mx, vx[0], sx, sox);
    if (i == 3) DMA(mw + 2048, vw[2], sw, sow);
    if (i == 4) DMA(mw + 3072, vw[3], sw, sow);
    if (i == 5) DMA(mx + 1024, vx[1], sx, sox);
  };
#pragma unroll
  for (int s = 0; s < NS; ++s)
#pragma unroll
    for (int i = 0; i < 6; ++i) issue(i, s * SB, s);

  unsigned ao[2], bo[4];
  { const int r = lane & 15, q = lane >> 4;
#pragma unroll
    for (int ks = 0; ks < 2; ++ks) ao[ks] = r * 128 + (((4 * ks + q) ^ ((r >> 1) & 7)) << 4);
    const int g = lane >> 4, qq = (lane & 15) >> 2, p = lane & 3;
#pragma unroll
    for (int n = 0; n < 2; ++n)
#pragma unroll
      for (int t = 0; t < 2; ++t) { const int row = 8 * g + 4 * t + qq, ch = 2 * (2 * wave + n) + (p >> 1);
        bo[n * 2 + t] = XB + 256 * row + 16 * (ch ^ (((row & 3) << 2) | ((row >> 2) & 3))) + 8 * (p & 1); }
  }
  f32x4 acc[4][2];
#pragma unroll
  for (int m = 0; m < 4; ++m)
#pragma unroll
    for (int n = 0; n < 2; ++n) acc[m][n] = (f32x4){0.f, 0.f, 0.f, 0.f};
  Frags f0, f1;

  auto step = [&](Frags& fc, Frags& fn, unsigned cur, unsigned nxt, int kt, const bool RD, const bool IS) {
    LAS char* st = smem + nxt;
#pragma unroll
    for (int ks = 0; ks < 2; ++ks) {
#pragma unroll
      for (int m = 0; m < 4; ++m) {
        const int g = ks * 4 + m;
        if (RD) {
          if (m == 0) fn.b[0][ks] = rd_b(st, bo[0], bo[1], ks);
          if (m == 1) fn.b[1][ks] = rd_b(st, bo[2], bo[3], ks);
          if (m == 2) { fn.a[0][ks] = *(LAS bf16x8*)(st + ao[ks]); fn.a[1][ks] = *(LAS bf16x8*)(st + ao[ks] + 2048); }
          if (m == 3) { fn.a[2][ks] = *(LAS bf16x8*)(st + ao[ks] + 4096); fn.a[3][ks] = *(LAS bf16x8*)(st + ao[ks] + 6144); }
        }
        MFMA(acc[m][0], fc.b[0][ks], fc.a[m][ks]);
        if (IS && g < 6) issue(g, cur, kt);
        MFMA(acc[m][1], fc.b[1][ks], fc.a[m][ks]);
        SB0();
      }
    }
  };
  wait_vm<6 * (NS - 1)>();
  bar();
  { LAS char* st = smem;
#pragma unroll
    for (int ks = 0; ks < 2; ++ks) {
      f0.b[0][ks] = rd_b(st, bo[0], bo[1], ks); f0.b[1][ks] = rd_b(st, bo[2], bo[3], ks);
#pragma unroll
      for (int m = 0; m < 4; ++m) f0.a[m][ks] = *(LAS bf16x8*)(st + ao[ks] + m * 2048);
    } }
  asm volatile("" : "+v"(call));
  STAMP(1);
  unsigned cur = 0;
  auto adv = [](unsigned o) { const unsigned n = o + SB; return n == NS * SB ? 0u : n; };
  constexpr int MAIN = ((NK - NS) / 2) * 2;
  for (int s = 0; s < MAIN; s += 2) {
    { wait_vm<6 * (NS - 2)>(); wait_lgkm0(); bar(); SB0(); const unsigned n = adv(cur); step(f0, f1, cur, n, s + NS, true, true); cur = n; }
    { wait_vm<6 * (NS - 2)>(); wait_lgkm0(); bar(); SB0(); const unsigned n = adv(cur); step(f1, f0, cur, n, s + 1 + NS, true, true); cur = n; }
  }
  static_assert(NK - MAIN == 6 && NS == 6, "tail written for NS = 6");
  { wait_vm<24>(); wait_lgkm0(); bar(); SB0(); const unsigned n = adv(cur); step(f0, f1, cur, n, 0, true, false); cur = n; }
  { wait_vm<18>(); wait_lgkm0(); bar(); SB0(); const unsigned n = adv(cur); step(f1, f0, cur, n, 0, true, false); cur = n; }
  { wait_vm<12>(); wait_lgkm0(); bar(); SB0(); const unsigned n = adv(cur); step(f0, f1, cur, n, 0, true, false); cur = n; }
  { wait_vm<6>();  wait_lgkm0(); bar(); SB0(); const unsigned n = adv(cur); step(f1, f0, cur, n, 0, true, false); cur = n; }
  { wait_vm<0>();  wait_lgkm0(); bar(); SB0(); const unsigned n = adv(cur); step(f0, f1, cur, n, 0, true, false); cur = n; }
  step(f1, f0, 0, 0, 0, false, false);
  STAMP(2);

  call += 1;
  float mx = 0.f;
#pragma unroll
  for (int m = 0; m < 4; ++m)
#pragma unroll
    for (int n = 0; n < 2; ++n)
#pragma unroll
      for (int r = 0; r < 4; ++r) { acc[m][n][r] = fmaxf(acc[m][n][r], 0.f); mx = fmaxf(mx, acc[m][n][r]); }
#pragma unroll
  for (int o = 32; o >= 1; o >>= 1) mx = fmaxf(mx, __shfl_xor(mx, o));
  constexpr int PITCH = 528, RED = 64 * PITCH;
  LAS float* red = (LAS float*)(smem + RED);
  if (lane == 0) red[wave] = mx;
  wait_lgkm0(); bar();
  char* const gset = myws + WS_GRAN + (size_t)(call & 1) * WS_GRAN_SET;
  if (tid < NR) {
    const float t = fmaxf(fmaxf(red[0], red[1]), fmaxf(red[2], red[3]));
    char* dst = P.ws[tid] + WS_GRAN + (size_t)(call & 1) * WS_GRAN_SET + (size_t)(rank * NWG + b) * 8;
    const u32x2 gr = {__float_as_uint(t), call};
    asm volatile("global_store_dwordx2 %0, %1, off sc0 sc1" :: "v"(dst), "v"(gr) : "memory");
  }
  if (tid == 64) { unsigned* cnt = (unsigned*)(myws + WS_CNT) + b; asm volatile("global_store_dword %0, %1, off sc0 sc1" :: "v"(cnt), "v"(call) : "memory"); }
#pragma unroll
  for (int m = 0; m < 4; ++m)
#pragma unroll
    for (int n = 0; n < 2; ++n)
      *(LAS f32x4*)(smem + (m * 16 + (lane & 15)) * PITCH + (wave * 32 + n * 16 + (lane >> 4) * 4) * 4) = acc[m][n];
  wait_lgkm0(); bar();
  f32x4 y[2][4];
#pragma unroll
  for (int h = 0; h < 2; ++h)
#pragma unroll
    for (int c = 0; c < 4; ++c) y[h][c] = *(LAS f32x4*)(smem + (h * 32 + (tid >> 3)) * PITCH + (tid & 7) * 64 + c * 16);
  STAMP(3);

  unsigned* const tmo = (unsigned*)(myws + WS_TMO);
  unsigned gmax = 0; bool dead = false;
  { const unsigned long long t0 = __builtin_amdgcn_s_memrealtime();
    const char* g0 = gset + tid * 16;
    for (;;) {
      u32x4 a, c, d, e;
      asm volatile("global_load_dwordx4 %0, %4, off sc0 sc1\n\tglobal_load_dwordx4 %1, %5, off sc0 sc1\n\t"
                   "global_load_dwordx4 %2, %6, off sc0 sc1\n\tglobal_load_dwordx4 %3, %7, off sc0 sc1\n\ts_waitcnt vmcnt(0)"
                   : "=&v"(a), "=&v"(c), "=&v"(d), "=&v"(e) : "v"(g0), "v"(g0 + 4096), "v"(g0 + 8192), "v"(g0 + 12288) : "memory");
      const bool ok = a[1] >= call && a[3] >= call && c[1] >= call && c[3] >= call && d[1] >= call && d[3] >= call && e[1] >= call && e[3] >= call;
      if (__all(ok)) { gmax = max(max(max(a[0], a[2]), max(c[0], c[2])), max(max(d[0], d[2]), max(e[0], e[2]))); break; }
      if (__builtin_amdgcn_s_memrealtime() - t0 > DEADLINE || __builtin_amdgcn_readfirstlane(__hip_atomic_load(tmo, __ATOMIC_RELAXED, __HIP_MEMORY_SCOPE_AGENT)) == (unsigned)__builtin_amdgcn_readfirstlane(call)) {
        if (lane == 0) __hip_atomic_store(tmo, call, __ATOMIC_RELAXED, __HIP_MEMORY_SCOPE_AGENT);
        dead = true; break; }
      __builtin_amdgcn_s_sleep(1);
    } }
#pragma unroll
  for (int o = 32; o >= 1; o >>= 1) gmax = max(gmax, (unsigned)__shfl_xor((int)gmax, o));
  LAS unsigned* red2 = (LAS unsigned*)(smem + RED + 64);
  if (lane == 0) red2[wave] = dead ? 0xffffffffu : gmax;
  wait_lgkm0(); bar();
  gmax = max(max(red2[0], red2[1]), max(red2[2], red2[3]));
  STAMP(4);

  if (gmax != 0xffffffffu) {
    const float amax = __uint_as_float(gmax);
    const float inv = amax > 0.f ? 127.0f / amax : 0.f;
    const unsigned dstrank = (unsigned)cb >> 2;
    char* dst = P.out[dstrank & (NR - 1)] + (size_t)(rank * MS + rb * 64 + (tid >> 3)) * NCOL + (cb & 3) * 128 + (tid & 7) * 16;
#pragma unroll
    for (int h = 0; h < 2; ++h) {
      u32x4 q;
#pragma unroll
      for (int c = 0; c < 4; ++c) {
        unsigned u[4];
#pragma unroll
        for (int r = 0; r < 4; ++r) u[r] = __float_as_uint(fminf(__builtin_fmaf(y[h][c][r], inv, 12582912.0f), 12583039.0f));
        q[c] = __builtin_amdgcn_perm(__builtin_amdgcn_perm(u[3], u[2], 0x0c0c0400u), __builtin_amdgcn_perm(u[1], u[0], 0x0c0c0400u), 0x05040100u);
      }
      st16_sys(dst + (size_t)h * 32 * NCOL, q);
    }
  }
  STAMP(5);

  { const unsigned long long t0 = __builtin_amdgcn_s_memrealtime();
    for (;;) {
      u32x4 a, c;
      asm volatile("global_load_dwordx4 %0, %2, off sc0 sc1\n\tglobal_load_dwordx4 %1, %3, off sc0 sc1\n\ts_waitcnt vmcnt(0)"
                   : "=&v"(a), "=&v"(c) : "v"(myout), "v"(myout + 4096) : "memory");
      const unsigned m = (a[0] | a[1] | a[2] | a[3] | c[0] | c[1] | c[2] | c[3]) & 0x80808080u;
      if (__all(m == 0)) break;
      if (__builtin_amdgcn_s_memrealtime() - t0 > DEADLINE || __builtin_amdgcn_readfirstlane(__hip_atomic_load(tmo, __ATOMIC_RELAXED, __HIP_MEMORY_SCOPE_AGENT)) == (unsigned)__builtin_amdgcn_readfirstlane(call)) {
        if (lane == 0) __hip_atomic_store(tmo, call, __ATOMIC_RELAXED, __HIP_MEMORY_SCOPE_AGENT);
        break; }
      __builtin_amdgcn_s_sleep(2);
    } }
  STAMP(6);
}
}

extern "C" void kernel_launch(int rank, int nranks, ncclComm_t comm, hipStream_t stream,
                              void* const* d_in, const int* in_sizes, int n_in,
                              void* d_out, int out_size, const int* in_dtypes, int out_dtype,
                              void* const* d_out_all, void* const* d_ws_all, size_t ws_bytes) {
  (void)comm; (void)in_dtypes; (void)out_dtype; (void)d_out;
  if (nranks != NR || n_in < 2 || in_sizes[0] != MS * K || in_sizes[1] != K * N || out_size != NR * MS * NCOL || ws_bytes < WS_NEED || (unsigned)rank >= (unsigned)NR) {
    fprintf(stderr, "kernel_launch: unexpected problem shape\n");
    return;
  }
  HIP_CHECK(hipFuncSetAttribute((const void*)fused_kernel, hipFuncAttributeMaxDynamicSharedMemorySize, LDS_BYTES));
  Params P;
  P.x = (const bf16_t*)d_in[0]; P.w = (const bf16_t*)d_in[1];
  for (int p = 0; p < NR; ++p) { P.out[p] = (char*)d_out_all[p]; P.ws[p] = (char*)d_ws_all[p]; }
  P.rank = rank; P.pad = 0;
  hipLaunchKernelGGL(fused_kernel, dim3(NWG), dim3(256), LDS_BYTES, stream, P);
  HIP_CHECK(hipGetLastError());
}
```

```cpp
#include <hip/hip_runtime.h>
#include <rccl/rccl.h>
#include <cstdio>

#define HIP_CHECK(expr) do { hipError_t _e = (expr); if (_e != hipSuccess) { \
    fprintf(stderr, "HIP error %s at %s:%d\n", hipGetErrorString(_e), __FILE__, __LINE__); \
    return; } } while (0)

namespace {
typedef unsigned short bf16_t;
typedef short bf16x8 __attribute__((ext_vector_type(8)));
typedef short bf16x4 __attribute__((ext_vector_type(4)));
typedef float f32x4 __attribute__((ext_vector_type(4)));
typedef unsigned u32x4 __attribute__((ext_vector_type(4)));
typedef unsigned u32x2 __attribute__((ext_vector_type(2)));
#define LAS __attribute__((address_space(3)))

constexpr int NR = 8;
constexpr int MS = 512, N = 4096, K = 4096;
constexpr int NCOL = N / NR;
constexpr int NWG = 256;
constexpr int XB = 8192, WB = 16384, SB = XB + WB, NS = 6, NK = K / 64;
constexpr int LDS_BYTES = NS * SB;
constexpr size_t WS_CNT = 0;
constexpr size_t WS_TMO = 4096;
constexpr size_t WS_GRAN = 8192;
constexpr size_t WS_GRAN_SET = (size_t)NR * NWG * 8;
constexpr size_t WS_STAMP = 65536;
constexpr size_t WS_NEED = WS_STAMP + 65536;
constexpr unsigned long long DEADLINE = 50000000ull;

struct Params {
  const bf16_t* x; const bf16_t* w;
  char* out[NR]; char* ws[NR];
  int rank; int pad;
};

#define DMA(m0v, voff, srd, soff) asm volatile("s_mov_b32 m0, %0\n\ts_nop 0\n\tbuffer_load_dwordx4 %1, %2, %3 offen lds" :: "s"(m0v), "v"(voff), "s"(srd), "s"(soff) : "memory")
template <int V> __device__ __forceinline__ void wait_vm() { asm volatile("s_waitcnt vmcnt(%0)" :: "i"(V) : "memory"); }
__device__ __forceinline__ void wait_lgkm0() { __builtin_amdgcn_s_waitcnt(0xC07F); }
__device__ __forceinline__ void bar() { asm volatile("" ::: "memory"); __builtin_amdgcn_s_barrier(); asm volatile("" ::: "memory"); }
__device__ __forceinline__ void st16_sys(void* p, u32x4 v) { asm volatile("global_store_dwordx4 %0, %1, off sc0 sc1\n\ts_nop 1" :: "v"(p), "v"(v) : "memory"); }
#define STAMP(i) do { } while (0)

struct Frags { bf16x8 a[4][2]; bf16x8 b[2][2]; };

__device__ __forceinline__ bf16x8 rd_b(LAS char* st, unsigned o0, unsigned o1, int ks) {
  bf16x4 lo = __builtin_amdgcn_ds_read_tr16_b64_v4i16((LAS bf16x4*)(st + o0 + ks * 8192));
  bf16x4 hi = __builtin_amdgcn_ds_read_tr16_b64_v4i16((LAS bf16x4*)(st + o1 + ks * 8192));
  return __builtin_shufflevector(lo, hi, 0, 1, 2, 3, 4, 5, 6, 7);
}
#define MFMA(acc, w_, x_) acc = __builtin_amdgcn_mfma_f32_16x16x32_bf16(w_, x_, acc, 0, 0, 0)
#define SB0() __builtin_amdgcn_sched_barrier(0)

__global__ __launch_bounds__(256, 2) void fused_kernel(Params P) {
  extern __shared__ __attribute__((aligned(16))) char smem_[];
  LAS char* smem = (LAS char*)smem_;
  const int tid = threadIdx.x, lane = tid & 63;
  const int wave = __builtin_amdgcn_readfirstlane(tid >> 6);
  const int b = blockIdx.x;
  const int grp = b & 7, j = b >> 3;
  const int rb = j & 7, cb = grp * 4 + (j >> 3);
  const int rank = P.rank;
  char* const myws = P.ws[rank];
  STAMP(0);

  const __amdgpu_buffer_rsrc_t sx = __builtin_amdgcn_make_buffer_rsrc((void*)(P.x + (size_t)rb * 64 * K), (short)0, 64 * K * 2, 0x00020000);
  const __amdgpu_buffer_rsrc_t sw = __builtin_amdgcn_make_buffer_rsrc((void*)(P.w + cb * 128), (short)0, K * N * 2 - cb * 256, 0x00020000);
  unsigned vx[2], vw[4];
#pragma unroll
  for (int i = 0; i < 2; ++i) { const int row = (wave * 2 + i) * 8 + (lane >> 3); vx[i] = row * (K * 2) + (((lane & 7) ^ ((row >> 1) & 7)) << 4); }
#pragma unroll
  for (int i = 0; i < 4; ++i) { const int row = (wave * 4 + i) * 4 + (lane >> 4); const int g = ((row & 3) << 2) | ((row >> 2) & 3); vw[i] = row * (N * 2) + (((lane & 15) ^ g) << 4); }
  const unsigned ldsbase = (unsigned)(size_t)smem;
  int ordv;
  { const int l0 = (rb + 2 * (j >> 3)) & 7, l1 = (l0 + 1) & 7, lo = min(l0, l1), hi = max(l0, l1);
    int t = lane - 2;
    if (t >= 56 + lo) ++t;
    if (t >= 56 + hi) ++t;
    ordv = ((t & 7) == l0 || (t & 7) == l1) ? t + 8 : t;
    if (lane == 0) ordv = lo;
    if (lane == 1) ordv = hi;
    ordv &= 63; }

  char* const myout = P.out[rank] + (size_t)b * 8192 + tid * 16;
  { const u32x4 mk = {0x80808080u, 0x80808080u, 0x80808080u, 0x80808080u};
    st16_sys(myout, mk); st16_sys(myout + 4096, mk); }
  unsigned call;
  { unsigned* cnt = (unsigned*)(myws + WS_CNT) + b;
    asm volatile("global_load_dword %0, %1, off sc0 sc1" : "=v"(call) : "v"(cnt) : "memory"); }

  auto issue = [&](int i, unsigned stage_off, int kt) {
    const unsigned mx = ldsbase + stage_off + wave * 2048, mw = ldsbase + stage_off + XB + wave * 4096;
    const unsigned sox = kt * 128, sow = kt * (64 * N * 2);
    if (i == 0) DMA(mw, vw[0], sw, sow);
    if (i == 1) DMA(mw + 1024, vw[1], sw, sow);
    if (i == 2) DMA(mx, vx[0], sx, sox);
    if (i == 3) DMA(mw + 2048, vw[2], sw, sow);
    if (i == 4) DMA(mw + 3072, vw[3], sw, sow);
    if (i == 5) DMA(mx + 1024, vx[1], sx, sox);
  };
#pragma unroll
  for (int s = 0; s < NS; ++s)
#pragma unroll
    for (int i = 0; i < 6; ++i) issue(i, s * SB, __builtin_amdgcn_readlane(ordv, s));

  unsigned ao[2], bo[4];
  { const int r = lane & 15, q = lane >> 4;
#pragma unroll
    for (int ks = 0; ks < 2; ++ks) ao[ks] = r * 128 + (((4 * ks + q) ^ ((r >> 1) & 7)) << 4);
    const int g = lane >> 4, qq = (lane & 15) >> 2, p = lane & 3;
#pragma unroll
    for (int n = 0; n < 2; ++n)
#pragma unroll
      for (int t = 0; t < 2; ++t) { const int row = 8 * g + 4 * t + qq, ch = 2 * (2 * wave + n) + (p >> 1);
        bo[n * 2 + t] = XB + 256 * row + 16 * (ch ^ (((row & 3) << 2) | ((row >> 2) & 3))) + 8 * (p & 1); }
  }
  f32x4 acc[4][2];
#pragma unroll
  for (int m = 0; m < 4; ++m)
#pragma unroll
    for (int n = 0; n < 2; ++n) acc[m][n] = (f32x4){0.f, 0.f, 0.f, 0.f};
  Frags f0, f1;

  auto step = [&](Frags& fc, Frags& fn, unsigned cur, unsigned nxt, int kt, const bool RD, const bool IS) {
    LAS char* st = smem + nxt;
#pragma unroll
    for (int ks = 0; ks < 2; ++ks) {
#pragma unroll
      for (int m = 0; m < 4; ++m) {
        const int g = ks * 4 + m;
        if (RD) {
          if (m == 0) fn.b[0][ks] = rd_b(st, bo[0], bo[1], ks);
          if (m == 1) fn.b[1][ks] = rd_b(st, bo[2], bo[3], ks);
          if (m == 2) { fn.a[0][ks] = *(LAS bf16x8*)(st + ao[ks]); fn.a[1][ks] = *(LAS bf16x8*)(st + ao[ks] + 2048); }
          if (m == 3) { fn.a[2][ks] = *(LAS bf16x8*)(st + ao[ks] + 4096); fn.a[3][ks] = *(LAS bf16x8*)(st + ao[ks] + 6144); }
        }
        MFMA(acc[m][0], fc.b[0][ks], fc.a[m][ks]);
        if (IS && g < 6) issue(g, cur, kt);
        MFMA(acc[m][1], fc.b[1][ks], fc.a[m][ks]);
        SB0();
      }
    }
  };
  wait_vm<6 * (NS - 1)>();
  bar();
  { LAS char* st = smem;
#pragma unroll
    for (int ks = 0; ks < 2; ++ks) {
      f0.b[0][ks] = rd_b(st, bo[0], bo[1], ks); f0.b[1][ks] = rd_b(st, bo[2], bo[3], ks);
#pragma unroll
      for (int m = 0; m < 4; ++m) f0.a[m][ks] = *(LAS bf16x8*)(st + ao[ks] + m * 2048);
    } }
  asm volatile("" : "+v"(call));
  STAMP(1);
  unsigned cur = 0;
  auto adv = [](unsigned o) { const unsigned n = o + SB; return n == NS * SB ? 0u : n; };
  constexpr int MAIN = ((NK - NS) / 2) * 2;
  for (int s = 0; s < MAIN; s += 2) {
    { wait_vm<6 * (NS - 2)>(); wait_lgkm0(); bar(); SB0(); const unsigned n = adv(cur); step(f0, f1, cur, n, __builtin_amdgcn_readlane(ordv, s + NS), true, true); cur = n; }
    { wait_vm<6 * (NS - 2)>(); wait_lgkm0(); bar(); SB0(); const unsigned n = adv(cur); step(f1, f0, cur, n, __builtin_amdgcn_readlane(ordv, s + 1 + NS), true, true); cur = n; }
  }
  static_assert(NK - MAIN == 6 && NS == 6, "tail written for NS = 6");
  { wait_vm<24>(); wait_lgkm0(); bar(); SB0(); const unsigned n = adv(cur); step(f0, f1, cur, n, 0, true, false); cur = n; }
  { wait_vm<18>(); wait_lgkm0(); bar(); SB0(); const unsigned n = adv(cur); step(f1, f0, cur, n, 0, true, false); cur = n; }
  { wait_vm<12>(); wait_lgkm0(); bar(); SB0(); const unsigned n = adv(cur); step(f0, f1, cur, n, 0, true, false); cur = n; }
  { wait_vm<6>();  wait_lgkm0(); bar(); SB0(); const unsigned n = adv(cur); step(f1, f0, cur, n, 0, true, false); cur = n; }
  { wait_vm<0>();  wait_lgkm0(); bar(); SB0(); const unsigned n = adv(cur); step(f0, f1, cur, n, 0, true, false); cur = n; }
  step(f1, f0, 0, 0, 0, false, false);
  STAMP(2);

  call += 1;
  float mx = 0.f;
#pragma unroll
  for (int m = 0; m < 4; ++m)
#pragma unroll
    for (int n = 0; n < 2; ++n)
#pragma unroll
      for (int r = 0; r < 4; ++r) { acc[m][n][r] = fmaxf(acc[m][n][r], 0.f); mx = fmaxf(mx, acc[m][n][r]); }
#pragma unroll
  for (int o = 32; o >= 1; o >>= 1) mx = fmaxf(mx, __shfl_xor(mx, o));
  constexpr int PITCH = 528, RED = 64 * PITCH;
  LAS float* red = (LAS float*)(smem + RED);
  if (lane == 0) red[wave] = mx;
  wait_lgkm0(); bar();
  char* const gset = myws + WS_GRAN + (size_t)(call & 1) * WS_GRAN_SET;
  if (tid < NR) {
    const float t = fmaxf(fmaxf(red[0], red[1]), fmaxf(red[2], red[3]));
    char* dst = P.ws[tid] + WS_GRAN + (size_t)(call & 1) * WS_GRAN_SET + (size_t)(rank * NWG + b) * 8;
    const u32x2 gr = {__float_as_uint(t), call};
    asm volatile("global_store_dwordx2 %0, %1, off sc0 sc1" :: "v"(dst), "v"(gr) : "memory");
  }
  if (tid == 64) { unsigned* cnt = (unsigned*)(myws + WS_CNT) + b; asm volatile("global_store_dword %0, %1, off sc0 sc1" :: "v"(cnt), "v"(call) : "memory"); }
#pragma unroll
  for (int m = 0; m < 4; ++m)
#pragma unroll
    for (int n = 0; n < 2; ++n)
      *(LAS f32x4*)(smem + (m * 16 + (lane & 15)) * PITCH + (wave * 32 + n * 16 + (lane >> 4) * 4) * 4) = acc[m][n];
  wait_lgkm0(); bar();
  f32x4 y[2][4];
#pragma unroll
  for (int h = 0; h < 2; ++h)
#pragma unroll
    for (int c = 0; c < 4; ++c) y[h][c] = *(LAS f32x4*)(smem + (h * 32 + (tid >> 3)) * PITCH + (tid & 7) * 64 + c * 16);
  STAMP(3);

  unsigned* const tmo = (unsigned*)(myws + WS_TMO);
  unsigned gmax = 0; bool dead = false;
  { const unsigned long long t0 = __builtin_amdgcn_s_memrealtime();
    const char* g0 = gset + tid * 16;
    for (;;) {
      u32x4 a, c, d, e;
      asm volatile("global_load_dwordx4 %0, %4, off sc0 sc1\n\tglobal_load_dwordx4 %1, %5, off sc0 sc1\n\t"
                   "global_load_dwordx4 %2, %6, off sc0 sc1\n\tglobal_load_dwordx4 %3, %7, off sc0 sc1\n\ts_waitcnt vmcnt(0)"
                   : "=&v"(a), "=&v"(c), "=&v"(d), "=&v"(e) : "v"(g0), "v"(g0 + 4096), "v"(g0 + 8192), "v"(g0 + 12288) : "memory");
      const bool ok = a[1] >= call && a[3] >= call && c[1] >= call && c[3] >= call && d[1] >= call && d[3] >= call && e[1] >= call && e[3] >= call;
      if (__all(ok)) { gmax = max(max(max(a[0], a[2]), max(c[0], c[2])), max(max(d[0], d[2]), max(e[0], e[2]))); break; }
      if (__builtin_amdgcn_s_memrealtime() - t0 > DEADLINE || __builtin_amdgcn_readfirstlane(__hip_atomic_load(tmo, __ATOMIC_RELAXED, __HIP_MEMORY_SCOPE_AGENT)) == (unsigned)__builtin_amdgcn_readfirstlane(call)) {
        if (lane == 0) __hip_atomic_store(tmo, call, __ATOMIC_RELAXED, __HIP_MEMORY_SCOPE_AGENT);
        dead = true; break; }
      __builtin_amdgcn_s_sleep(1);
    } }
#pragma unroll
  for (int o = 32; o >= 1; o >>= 1) gmax = max(gmax, (unsigned)__shfl_xor((int)gmax, o));
  LAS unsigned* red2 = (LAS unsigned*)(smem + RED + 64);
  if (lane == 0) red2[wave] = dead ? 0xffffffffu : gmax;
  wait_lgkm0(); bar();
  gmax = max(max(red2[0], red2[1]), max(red2[2], red2[3]));
  STAMP(4);

  if (gmax != 0xffffffffu) {
    const float amax = __uint_as_float(gmax);
    const float inv = amax > 0.f ? 127.0f / amax : 0.f;
    const unsigned dstrank = (unsigned)cb >> 2;
    char* dst = P.out[dstrank & (NR - 1)] + (size_t)(rank * MS + rb * 64 + (tid >> 3)) * NCOL + (cb & 3) * 128 + (tid & 7) * 16;
#pragma unroll
    for (int h = 0; h < 2; ++h) {
      u32x4 q;
#pragma unroll
      for (int c = 0; c < 4; ++c) {
        unsigned u[4];
#pragma unroll
        for (int r = 0; r < 4; ++r) u[r] = __float_as_uint(fminf(__builtin_fmaf(y[h][c][r], inv, 12582912.0f), 12583039.0f));
        q[c] = __builtin_amdgcn_perm(__builtin_amdgcn_perm(u[3], u[2], 0x0c0c0400u), __builtin_amdgcn_perm(u[1], u[0], 0x0c0c0400u), 0x05040100u);
      }
      st16_sys(dst + (size_t)h * 32 * NCOL, q);
    }
  }
  STAMP(5);

  { const unsigned long long t0 = __builtin_amdgcn_s_memrealtime();
    for (;;) {
      u32x4 a, c;
      asm volatile("global_load_dwordx4 %0, %2, off sc0 sc1\n\tglobal_load_dwordx4 %1, %3, off sc0 sc1\n\ts_waitcnt vmcnt(0)"
                   : "=&v"(a), "=&v"(c) : "v"(myout), "v"(myout + 4096) : "memory");
      const unsigned m = (a[0] | a[1] | a[2] | a[3] | c[0] | c[1] | c[2] | c[3]) & 0x80808080u;
      if (__all(m == 0)) break;
      if (__builtin_amdgcn_s_memrealtime() - t0 > DEADLINE || __builtin_amdgcn_readfirstlane(__hip_atomic_load(tmo, __ATOMIC_RELAXED, __HIP_MEMORY_SCOPE_AGENT)) == (unsigned)__builtin_amdgcn_readfirstlane(call)) {
        if (lane == 0) __hip_atomic_store(tmo, call, __ATOMIC_RELAXED, __HIP_MEMORY_SCOPE_AGENT);
        break; }
      __builtin_amdgcn_s_sleep(2);
    } }
  STAMP(6);
}
}

extern "C" void kernel_launch(int rank, int nranks, ncclComm_t comm, hipStream_t stream,
                              void* const* d_in, const int* in_sizes, int n_in,
                              void* d_out, int out_size, const int* in_dtypes, int out_dtype,
                              void* const* d_out_all, void* const* d_ws_all, size_t ws_bytes) {
  (void)comm; (void)in_dtypes; (void)out_dtype; (void)d_out;
  if (nranks != NR || n_in < 2 || in_sizes[0] != MS * K || in_sizes[1] != K * N || out_size != NR * MS * NCOL || ws_bytes < WS_NEED || (unsigned)rank >= (unsigned)NR) {
    fprintf(stderr, "kernel_launch: unexpected problem shape\n");
    return;
  }
  HIP_CHECK(hipFuncSetAttribute((const void*)fused_kernel, hipFuncAttributeMaxDynamicSharedMemorySize, LDS_BYTES));
  Params P;
  P.x = (const bf16_t*)d_in[0]; P.w = (const bf16_t*)d_in[1];
  for (int p = 0; p < NR; ++p) { P.out[p] = (char*)d_out_all[p]; P.ws[p] = (char*)d_ws_all[p]; }
  P.rank = rank; P.pad = 0;
  hipLaunchKernelGGL(fused_kernel, dim3(NWG), dim3(256), LDS_BYTES, stream, P);
  HIP_CHECK(hipGetLastError());
}
```

```cpp
#include <hip/hip_runtime.h>
#include <rccl/rccl.h>
#include <cstdio>

#define HIP_CHECK(expr) do { hipError_t _e = (expr); if (_e != hipSuccess) { \
    fprintf(stderr, "HIP error %s at %s:%d\n", hipGetErrorString(_e), __FILE__, __LINE__); \
    return; } } while (0)

namespace {
typedef unsigned short bf16_t;
typedef short bf16x8 __attribute__((ext_vector_type(8)));
typedef short bf16x4 __attribute__((ext_vector_type(4)));
typedef float f32x4 __attribute__((ext_vector_type(4)));
typedef unsigned u32x4 __attribute__((ext_vector_type(4)));
typedef unsigned u32x2 __attribute__((ext_vector_type(2)));
#define LAS __attribute__((address_space(3)))

constexpr int NR = 8;
constexpr int MS = 512, N = 4096, K = 4096;
constexpr int NCOL = N / NR;
constexpr int NWG = 256, NTHR = 512;
constexpr int TB = 16384, NS = 4, NK = 32;
constexpr int WREG = NS * TB;
constexpr int LDS_BYTES = 2 * NS * TB;
constexpr int NGRP = 16, GSZ = NWG / NGRP, NREP = 8;
constexpr size_t WS_CNT = 0;
constexpr size_t WS_TMO = 4096;
constexpr size_t WS_GRP = 8192;
constexpr size_t WS_GRAN = 16384;
constexpr size_t GRAN_SET = (size_t)NR * NGRP * 8;
constexpr size_t WS_STAMP = 65536;
constexpr size_t WS_XCH = 1 << 20;
constexpr size_t WS_NEED = WS_XCH + (size_t)NWG * 32768;
constexpr unsigned long long DEADLINE = 50000000ull;

struct Params {
  const bf16_t* x; const bf16_t* w;
  char* myws; char* myout;
  int rank; int pad;
  char* out[NR]; char* ws[NR];
};

#define DMA(m0v, voff, srd, soff) asm volatile("s_mov_b32 m0, %0\n\ts_nop 0\n\tbuffer_load_dwordx4 %1, %2, %3 offen lds" :: "s"(m0v), "v"(voff), "s"(srd), "s"(soff) : "memory")
template <int V> __device__ __forceinline__ void wait_vm() { asm volatile("s_waitcnt vmcnt(%0)" :: "i"(V) : "memory"); }
__device__ __forceinline__ void wait_lgkm0() { __builtin_amdgcn_s_waitcnt(0xC07F); }
__device__ __forceinline__ void bar() { asm volatile("" ::: "memory"); __builtin_amdgcn_s_barrier(); asm volatile("" ::: "memory"); }
__device__ __forceinline__ void st16_sys(void* p, u32x4 v) { asm volatile("global_store_dwordx4 %0, %1, off sc0 sc1\n\ts_nop 1" :: "v"(p), "v"(v) : "memory"); }
template <int O> __device__ __forceinline__ void st16_sys_o(void* p, u32x4 v) { asm volatile("global_store_dwordx4 %0, %1, off offset:%2 sc0 sc1\n\ts_nop 1" :: "v"(p), "v"(v), "i"(O) : "memory"); }
__device__ __forceinline__ unsigned wave_umax(unsigned v) {
  v = max(v, (unsigned)__builtin_amdgcn_update_dpp(0, (int)v, 0x111, 0xf, 0xf, true));
  v = max(v, (unsigned)__builtin_amdgcn_update_dpp(0, (int)v, 0x112, 0xf, 0xf, true));
  v = max(v, (unsigned)__builtin_amdgcn_update_dpp(0, (int)v, 0x114, 0xf, 0xf, true));
  v = max(v, (unsigned)__builtin_amdgcn_update_dpp(0, (int)v, 0x118, 0xf, 0xf, true));
  v = max(v, (unsigned)__builtin_amdgcn_update_dpp(0, (int)v, 0x142, 0xa, 0xf, true));
  v = max(v, (unsigned)__builtin_amdgcn_update_dpp(0, (int)v, 0x143, 0xc, 0xf, true));
  return (unsigned)__builtin_amdgcn_readlane((int)v, 63);
}
__device__ __forceinline__ bool expired(unsigned* tmo, unsigned call, unsigned long long t0) {
  const unsigned long long dt = __builtin_amdgcn_s_memrealtime() - t0;
  if (dt < 5000) return false;
  if (dt > DEADLINE ||
      (unsigned)__builtin_amdgcn_readfirstlane(__hip_atomic_load(tmo, __ATOMIC_RELAXED, __HIP_MEMORY_SCOPE_AGENT)) == call) {
    __hip_atomic_store(tmo, call, __ATOMIC_RELAXED, __HIP_MEMORY_SCOPE_AGENT);
    return true;
  }
  return false;
}
#define STAMP(i) do { } while (0)

struct Frags { bf16x8 a[4]; bf16x8 b[4]; };

#define MFMA(acc, w_, x_) acc = __builtin_amdgcn_mfma_f32_16x16x32_bf16(w_, x_, acc, 0, 0, 0)
#define SB0() __builtin_amdgcn_sched_barrier(0)

__global__ __launch_bounds__(NTHR, 2) void fused_kernel(Params P) {
  extern __shared__ __attribute__((aligned(16))) char smem_[];
  LAS char* smem = (LAS char*)smem_;
  const int tid = threadIdx.x, lane = tid & 63;
  const int wave = __builtin_amdgcn_readfirstlane(tid >> 6);
  const int b = blockIdx.x;
  const int g = b & 7, j = b >> 3;
  const int rb = j & 3, kh = g & 1, cb = (g >> 1) * 8 + (j >> 2);
  const int rank = P.rank;
  char* const myws = P.myws;
  STAMP(0);
  unsigned call;
  unsigned* const tmo = (unsigned*)(myws + WS_TMO);
  constexpr int PITCH = 528, RED = 64 * PITCH;
  LAS unsigned* red = (LAS unsigned*)(smem + RED);

  if (__builtin_expect(wave >= 4, 1)) {
    const int lw = wave - 4;
    const __amdgpu_buffer_rsrc_t sx = __builtin_amdgcn_make_buffer_rsrc((void*)(P.x + (size_t)rb * 128 * K), (short)0, 128 * K * 2, 0x00020000);
    const __amdgpu_buffer_rsrc_t sw = __builtin_amdgcn_make_buffer_rsrc((void*)(P.w + cb * 128), (short)0, K * N * 2 - cb * 256, 0x00020000);
    unsigned vx[4], vw[4];
#pragma unroll
    for (int i = 0; i < 4; ++i) { const int row = (lw * 4 + i) * 8 + (lane >> 3); vx[i] = row * (K * 2) + (((lane & 7) ^ ((row >> 1) & 7)) << 4); }
#pragma unroll
    for (int i = 0; i < 4; ++i) { const int row = (lw * 4 + i) * 4 + (lane >> 4); const int f = ((row & 3) << 2) | ((row >> 2) & 3); vw[i] = row * (N * 2) + (((lane & 15) ^ f) << 4); }
    int ordv;
    { const int l0 = ((j >> 2) + 2 * rb) & 7, l1 = (l0 + 1) & 7, lo = min(l0, l1), hi = max(l0, l1);
      int t = lane - 2;
      if (t >= NK - 8 + lo) ++t;
      if (t >= NK - 8 + hi) ++t;
      ordv = ((t & 7) == l0 || (t & 7) == l1) ? t + 8 : t;
      if (lane == 0) ordv = lo;
      if (lane == 1) ordv = hi;
      ordv = (ordv & (NK - 1)) + kh * NK; }
    const unsigned ldsw = (unsigned)(size_t)smem + lw * 4096;
    auto issue = [&](int s) {
      const int kt = __builtin_amdgcn_readlane(ordv, s);
      const unsigned mx = ldsw + (s & (NS - 1)) * TB, mw = mx + WREG, sox = kt * 128, sow = kt * (64 * N * 2);
#pragma unroll
      for (int i = 0; i < 4; ++i) { DMA(mw + i * 1024, vw[i], sw, sow); DMA(mx + i * 1024, vx[i], sx, sox); }
    };
#pragma unroll
    for (int s = 0; s < NS; ++s) issue(s);
    wait_vm<8 * (NS - 1)>(); bar();
    for (int s = 0; s < NK - NS; ++s) { wait_vm<8 * (NS - 2)>(); bar(); issue(s + NS); }
    static_assert(NS == 4, "tail written for NS = 4");
    wait_vm<16>(); bar();
    wait_vm<8>(); bar();
    wait_vm<0>(); bar();
    bar();
    call = __builtin_amdgcn_readfirstlane(red[5]);
  } else {
    unsigned cnt0;
    asm volatile("global_load_dword %0, %1, off sc0 sc1" : "=v"(cnt0) : "v"((unsigned*)(myws + WS_CNT) + b) : "memory");
    { char* const fill = P.myout + (size_t)b * 8192 + tid * 16;
      const u32x4 mk = {0x80808080u, 0x80808080u, 0x80808080u, 0x80808080u};
      st16_sys(fill, mk); st16_sys(fill + 4096, mk); }
    const int wm = wave & 1, wn = wave >> 1;
    unsigned ao[2][2], bo[4][2];
    { const int r = lane & 15, q = lane >> 4;
#pragma unroll
      for (int h = 0; h < 2; ++h)
#pragma unroll
        for (int ks = 0; ks < 2; ++ks) ao[h][ks] = ((h ^ kh) * 64 + wm * 32 + r) * 128 + (((4 * ks + q) ^ ((r >> 1) & 7)) << 4);
      const int qq = (lane & 15) >> 2, p = lane & 3;
#pragma unroll
      for (int n = 0; n < 4; ++n)
#pragma unroll
        for (int t = 0; t < 2; ++t) { const int row = 8 * q + 4 * t + qq, ch = 2 * (4 * wn + n) + (p >> 1);
          bo[n][t] = WREG + 256 * row + 16 * (ch ^ (((row & 3) << 2) | ((row >> 2) & 3))) + 8 * (p & 1); }
    }
    f32x4 acc[4][4];
#pragma unroll
    for (int m = 0; m < 4; ++m)
#pragma unroll
      for (int n = 0; n < 4; ++n) acc[m][n] = (f32x4){0.f, 0.f, 0.f, 0.f};
    Frags f0, f1;
    auto rd_a = [&](int m, int st, int ks) { return *(LAS bf16x8*)(smem + ao[m >> 1][ks] + (m & 1) * 2048 + st * TB); };
    auto rd_b = [&](int n, int st, int ks) {
      bf16x4 lo = __builtin_amdgcn_ds_read_tr16_b64_v4i16((LAS bf16x4*)(smem + bo[n][0] + st * TB + ks * 8192));
      bf16x4 hi = __builtin_amdgcn_ds_read_tr16_b64_v4i16((LAS bf16x4*)(smem + bo[n][1] + st * TB + ks * 8192));
      return __builtin_shufflevector(lo, hi, 0, 1, 2, 3, 4, 5, 6, 7); };
    auto half = [&](Frags& fc, Frags& fn, const int st, const int ks, const bool RD) {
      constexpr int OM[16] = {0, 1, 0, 1, 2, 2, 0, 1, 2, 3, 3, 3, 0, 1, 2, 3};
      constexpr int ON[16] = {0, 0, 1, 1, 0, 1, 2, 2, 2, 0, 1, 2, 3, 3, 3, 3};
#pragma unroll
      for (int i = 0; i < 16; ++i) {
        if (RD) {
          if (i == 0) fn.b[0] = rd_b(0, st, ks);
          if (i == 1) { fn.a[0] = rd_a(0, st, ks); fn.a[1] = rd_a(1, st, ks); }
          if (i == 3) fn.b[1] = rd_b(1, st, ks);
          if (i == 4) fn.a[2] = rd_a(2, st, ks);
          if (i == 6) fn.b[2] = rd_b(2, st, ks);
          if (i == 7) fn.a[3] = rd_a(3, st, ks);
          if (i == 9) fn.b[3] = rd_b(3, st, ks);
        }
        MFMA(acc[OM[i]][ON[i]], fc.b[ON[i]], fc.a[OM[i]]);
        SB0();
      }
    };
    bar();
#pragma unroll
    for (int i = 0; i < 4; ++i) { f0.b[i] = rd_b(i, 0, 0); f0.a[i] = rd_a(i, 0, 0); }
    STAMP(1);
    SB0();
    half(f0, f1, 0, 1, true);
    for (int s = 0; s < NK - NS; s += NS) {
#pragma unroll
      for (int u = 1; u <= NS; ++u) { wait_lgkm0(); bar(); SB0(); half(f1, f0, u & (NS - 1), 0, true); half(f0, f1, u & (NS - 1), 1, true); }
    }
#pragma unroll
    for (int u = 1; u < NS; ++u) { wait_lgkm0(); bar(); SB0(); half(f1, f0, u, 0, true); half(f0, f1, u, 1, true); }
    half(f1, f0, 0, 0, false);
    STAMP(2);

    asm volatile("s_waitcnt vmcnt(0)\n\tv_readfirstlane_b32 %0, %1" : "=s"(call) : "v"(cnt0) : "memory");
    call += 1;
    const unsigned tag = 1u + (call & 1u);
    { char* dst = myws + WS_XCH + (size_t)(b ^ 1) * 32768 + wave * 8192 + lane * 16 + 4096;
      u32x4 v[8];
#pragma unroll
      for (int i = 0; i < 8; ++i)
#pragma unroll
        for (int r = 0; r < 4; ++r) v[i][r] = (__float_as_uint(acc[2 + (i >> 2)][i & 3][r]) & ~3u) | tag;
      st16_sys_o<-4096>(dst, v[0]); st16_sys_o<-3072>(dst, v[1]); st16_sys_o<-2048>(dst, v[2]); st16_sys_o<-1024>(dst, v[3]);
      st16_sys_o<0>(dst, v[4]); st16_sys_o<1024>(dst, v[5]); st16_sys_o<2048>(dst, v[6]); st16_sys_o<3072>(dst, v[7]); }
    if (tid == 0) {
      unsigned* cnt = (unsigned*)(myws + WS_CNT) + b;
      asm volatile("global_store_dword %0, %1, off sc0 sc1" :: "v"(cnt), "v"(call) : "memory");
      if (b < NGRP) {
        char* nx = myws + WS_GRP + (size_t)(((call + 1) & 1) * NGRP + b) * 128 + 4;
        asm volatile("global_store_dword %0, %1, off sc0 sc1" :: "v"(nx), "v"(0u) : "memory"); }
    }
    bool dead = false;
    { const char* src = myws + WS_XCH + (size_t)b * 32768 + wave * 8192 + lane * 16 + 4096;
      const unsigned long long t0 = __builtin_amdgcn_s_memrealtime();
      u32x4 v[8];
      for (;;) {
        asm volatile("global_load_dwordx4 %0, %8, off offset:-4096 sc0 sc1\n\tglobal_load_dwordx4 %1, %8, off offset:-3072 sc0 sc1\n\t"
                     "global_load_dwordx4 %2, %8, off offset:-2048 sc0 sc1\n\tglobal_load_dwordx4 %3, %8, off offset:-1024 sc0 sc1\n\t"
                     "global_load_dwordx4 %4, %8, off sc0 sc1\n\tglobal_load_dwordx4 %5, %8, off offset:1024 sc0 sc1\n\t"
                     "global_load_dwordx4 %6, %8, off offset:2048 sc0 sc1\n\tglobal_load_dwordx4 %7, %8, off offset:3072 sc0 sc1\n\ts_waitcnt vmcnt(0)"
                     : "=&v"(v[0]), "=&v"(v[1]), "=&v"(v[2]), "=&v"(v[3]), "=&v"(v[4]), "=&v"(v[5]), "=&v"(v[6]), "=&v"(v[7]) : "v"(src) : "memory");
        unsigned all = tag;
#pragma unroll
        for (int i = 0; i < 8; ++i) all &= (v[i][0] & v[i][1]) & (v[i][2] & v[i][3]);
        if (__all(all != 0)) break;
        if (expired(tmo, call, t0)) { dead = true; break; }
      }
#pragma unroll
      for (int i = 0; i < 8; ++i)
#pragma unroll
        for (int r = 0; r < 4; ++r) acc[i >> 2][i & 3][r] += __uint_as_float(v[i][r] & ~3u);
    }
    STAMP(3);
    float mx = 0.f;
#pragma unroll
    for (int i = 0; i < 8; ++i) mx = fmaxf(fmaxf(mx, fmaxf(acc[i >> 2][i & 3][0], acc[i >> 2][i & 3][1])), fmaxf(acc[i >> 2][i & 3][2], acc[i >> 2][i & 3][3]));
    const unsigned wmx = wave_umax(__float_as_uint(mx));
    if (lane == 0) { red[wave] = dead ? 0xffffffffu : wmx; red[5] = call; }
    wait_lgkm0(); bar();
#pragma unroll
    for (int i = 0; i < 8; ++i)
      *(LAS f32x4*)(smem + (wm * 32 + (i >> 2) * 16 + (lane & 15)) * PITCH + (wn * 64 + (i & 3) * 16 + (lane >> 4) * 4) * 4) = acc[i >> 2][i & 3];
  }

  u32x2 arr;
  char* const grp = myws + WS_GRP + (size_t)((call & 1) * NGRP + (b & (NGRP - 1))) * 128;
  if (wave == 4) {
    const unsigned t = max(max(red[0], red[1]), max(red[2], red[3]));
    const u32x2 one = {1u, 0u};
    if (lane == 0)
      asm volatile("global_atomic_umax %1, %2, off offset:4 sc1\n\tglobal_atomic_add_x2 %0, %1, %3, off sc0 sc1" : "=&v"(arr) : "v"(grp), "v"(t), "v"(one) : "memory");
  }
  wait_lgkm0(); bar();
  f32x4 y[4];
#pragma unroll
  for (int c = 0; c < 4; ++c) y[c] = *(LAS f32x4*)(smem + (tid >> 3) * PITCH + (tid & 7) * 64 + c * 16);
  if (wave == 4) {
    char* pw = P.ws[0];
#pragma unroll
    for (int i = 1; i < NR; ++i) { char* q = P.ws[i]; asm volatile("" : "+s"(q)); pw = (lane >> 3) == i ? q : pw; }
    asm volatile("s_waitcnt vmcnt(0)" : "+v"(arr) :: "memory");
    const unsigned n = __builtin_amdgcn_readfirstlane(arr[0]), gm = __builtin_amdgcn_readfirstlane(arr[1]);
    if ((n & (GSZ - 1)) == GSZ - 1) {
      char* dst = pw + WS_GRAN + (size_t)((call & 1) * NREP + (lane & 7)) * GRAN_SET + (size_t)(rank * NGRP + (b & (NGRP - 1))) * 8;
      const u32x2 gr = {gm, call};
      asm volatile("global_store_dwordx2 %0, %1, off sc0 sc1\n\ts_nop 1" :: "v"(dst), "v"(gr) : "memory");
    }
    static_assert(GRAN_SET == 1024, "one 16-B load per lane");
    const char* gs = myws + WS_GRAN + (size_t)((call & 1) * NREP + (b & (NREP - 1))) * GRAN_SET + lane * 16;
    const unsigned long long t0 = __builtin_amdgcn_s_memrealtime();
    unsigned gmax;
    for (;;) {
      u32x4 a;
      asm volatile("global_load_dwordx4 %0, %1, off sc0 sc1\n\ts_waitcnt vmcnt(0)" : "=&v"(a) : "v"(gs) : "memory");
      if (__all(a[1] == call && a[3] == call)) { gmax = wave_umax(max(a[0], a[2])); break; }
      if (expired(tmo, call, t0)) { gmax = 0xffffffffu; break; }
    }
    if (lane == 0) red[4] = gmax;
  }
  wait_lgkm0(); bar();
  const unsigned gmax = red[4];
  STAMP(4);

  if (gmax != 0xffffffffu) {
    const float amax = __uint_as_float(gmax);
    const float inv = amax > 0.f ? 127.0f / amax : 0.f;
    const unsigned dstrank = (unsigned)cb >> 2;
    char* dst = P.out[dstrank & (NR - 1)] + (size_t)(rank * MS + rb * 128 + kh * 64 + (tid >> 3)) * NCOL + (cb & 3) * 128 + (tid & 7) * 16;
    u32x4 q;
#pragma unroll
    for (int c = 0; c < 4; ++c) {
      unsigned u[4];
#pragma unroll
      for (int r = 0; r < 4; ++r) u[r] = __float_as_uint(__builtin_amdgcn_fmed3f(__builtin_fmaf(y[c][r], inv, 12582912.0f), 12582912.0f, 12583039.0f));
      q[c] = __builtin_amdgcn_perm(__builtin_amdgcn_perm(u[3], u[2], 0x0c0c0400u), __builtin_amdgcn_perm(u[1], u[0], 0x0c0c0400u), 0x05040100u);
    }
    st16_sys(dst, q);
  }
  STAMP(5);

  { const char* mine = P.myout + (size_t)b * 8192 + tid * 16;
    const unsigned long long t0 = __builtin_amdgcn_s_memrealtime();
    for (;;) {
      u32x4 a;
      asm volatile("global_load_dwordx4 %0, %1, off sc0 sc1\n\ts_waitcnt vmcnt(0)" : "=&v"(a) : "v"(mine) : "memory");
      if (__all(((a[0] | a[1] | a[2] | a[3]) & 0x80808080u) == 0)) break;
      if (expired(tmo, call, t0)) break;
      __builtin_amdgcn_s_sleep(2);
    } }
  STAMP(6);
}
}

extern "C" void kernel_launch(int rank, int nranks, ncclComm_t comm, hipStream_t stream,
                              void* const* d_in, const int* in_sizes, int n_in,
                              void* d_out, int out_size, const int* in_dtypes, int out_dtype,
                              void* const* d_out_all, void* const* d_ws_all, size_t ws_bytes) {
  (void)comm; (void)in_dtypes; (void)out_dtype; (void)d_out;
  if (nranks != NR || n_in < 2 || in_sizes[0] != MS * K || in_sizes[1] != K * N || out_size != NR * MS * NCOL || ws_bytes < WS_NEED || (unsigned)rank >= (unsigned)NR) {
    fprintf(stderr, "kernel_launch: unexpected problem shape\n");
    return;
  }
  HIP_CHECK(hipFuncSetAttribute((const void*)fused_kernel, hipFuncAttributeMaxDynamicSharedMemorySize, LDS_BYTES));
  Params P;
  P.x = (const bf16_t*)d_in[0]; P.w = (const bf16_t*)d_in[1];
  for (int p = 0; p < NR; ++p) { P.out[p] = (char*)d_out_all[p]; P.ws[p] = (char*)d_ws_all[p]; }
  P.myws = P.ws[rank]; P.myout = P.out[rank];
  P.rank = rank; P.pad = 0;
  hipLaunchKernelGGL(fused_kernel, dim3(NWG), dim3(NTHR), LDS_BYTES, stream, P);
  HIP_CHECK(hipGetLastError());
}
```

```cpp
#include <hip/hip_runtime.h>
#include <rccl/rccl.h>
#include <cstdio>
#include <cstdlib>

#define HIP_CHECK(expr) do { hipError_t _e = (expr); if (_e != hipSuccess) { \
    fprintf(stderr, "HIP error %s at %s:%d\n", hipGetErrorString(_e), __FILE__, __LINE__); \
    return; } } while (0)

namespace {
typedef unsigned short bf16_t;
typedef short bf16x8 __attribute__((ext_vector_type(8)));
typedef short bf16x4 __attribute__((ext_vector_type(4)));
typedef float f32x4 __attribute__((ext_vector_type(4)));
typedef unsigned u32x4 __attribute__((ext_vector_type(4)));
typedef unsigned u32x2 __attribute__((ext_vector_type(2)));
#define LAS __attribute__((address_space(3)))

constexpr int NR = 8;
constexpr int MS = 512, N = 4096, K = 4096;
constexpr int NCOL = N / NR;
constexpr int NWG = 256, NTHR = 512;
#define TAIL 8
#define XL 3
constexpr int TB = 16384, NS = 4;
constexpr int NT = TAIL, NF = (64 - NT) / 2, NK = NF + NT;
constexpr int WREG = NS * TB;
constexpr int RED = 2 * NS * TB;
constexpr int LDS_BYTES = RED + 64;
constexpr int NGRP = 16, GSZ = NWG / NGRP, NREP = 8;
constexpr size_t WS_CNT = 0;
constexpr size_t WS_TMO = 4096;
constexpr size_t WS_GRP = 8192;
constexpr size_t WS_GRAN = 16384;
constexpr size_t GRAN_SET = (size_t)NR * NGRP * 8;
constexpr size_t WS_STAMP = 65536;
constexpr size_t WS_XCH = 1 << 20;
constexpr size_t WS_NEED = WS_XCH + (size_t)NWG * 32768;
constexpr unsigned long long DEADLINE = 50000000ull;

struct Params {
  const bf16_t* x; const bf16_t* w;
  char* myws; char* myout;
  int rank; int pad;
  char* out[NR]; char* ws[NR];
  unsigned char ord[16][64];
};

#define DMA(m0v, voff, srd, soff) asm volatile("s_mov_b32 m0, %0\n\ts_nop 0\n\tbuffer_load_dwordx4 %1, %2, %3 offen lds" :: "s"(m0v), "v"(voff), "s"(srd), "s"(soff) : "memory")
template <int V> __device__ __forceinline__ void wait_vm() { asm volatile("s_waitcnt vmcnt(%0)" :: "i"(V) : "memory"); }
__device__ __forceinline__ void wait_lgkm0() { __builtin_amdgcn_s_waitcnt(0xC07F); }
__device__ __forceinline__ void bar() { asm volatile("" ::: "memory"); __builtin_amdgcn_s_barrier(); asm volatile("" ::: "memory"); }
__device__ __forceinline__ void st16_sys(void* p, u32x4 v) { asm volatile("global_store_dwordx4 %0, %1, off sc0 sc1\n\ts_nop 1" :: "v"(p), "v"(v) : "memory"); }
template <int O> __device__ __forceinline__ void st16_sys_o(void* p, u32x4 v) { asm volatile("global_store_dwordx4 %0, %1, off offset:%2 sc0 sc1\n\ts_nop 1" :: "v"(p), "v"(v), "i"(O) : "memory"); }
__device__ __forceinline__ unsigned wave_umax(unsigned v) {
  v = max(v, (unsigned)__builtin_amdgcn_update_dpp(0, (int)v, 0x111, 0xf, 0xf, true));
  v = max(v, (unsigned)__builtin_amdgcn_update_dpp(0, (int)v, 0x112, 0xf, 0xf, true));
  v = max(v, (unsigned)__builtin_amdgcn_update_dpp(0, (int)v, 0x114, 0xf, 0xf, true));
  v = max(v, (unsigned)__builtin_amdgcn_update_dpp(0, (int)v, 0x118, 0xf, 0xf, true));
  v = max(v, (unsigned)__builtin_amdgcn_update_dpp(0, (int)v, 0x142, 0xa, 0xf, true));
  v = max(v, (unsigned)__builtin_amdgcn_update_dpp(0, (int)v, 0x143, 0xc, 0xf, true));
  return (unsigned)__builtin_amdgcn_readlane((int)v, 63);
}
__device__ __forceinline__ bool expired(unsigned* tmo, unsigned call, unsigned long long t0) {
  const unsigned long long dt = __builtin_amdgcn_s_memrealtime() - t0;
  if (dt < 5000) return false;
  if (dt > DEADLINE ||
      (unsigned)__builtin_amdgcn_readfirstlane(__hip_atomic_load(tmo, __ATOMIC_RELAXED, __HIP_MEMORY_SCOPE_AGENT)) == call) {
    __hip_atomic_store(tmo, call, __ATOMIC_RELAXED, __HIP_MEMORY_SCOPE_AGENT);
    return true;
  }
  return false;
}
#define STAMP(i) do { } while (0)

struct Frags { bf16x8 a[4]; bf16x8 b[4]; };

#define MFMA(acc, w_, x_) acc = __builtin_amdgcn_mfma_f32_16x16x32_bf16(w_, x_, acc, 0, 0, 0)
#define SB0() __builtin_amdgcn_sched_barrier(0)

__global__ __launch_bounds__(NTHR, 2) void fused_kernel(Params P) {
  extern __shared__ __attribute__((aligned(16))) char smem_[];
  LAS char* smem = (LAS char*)smem_;
  const int tid = threadIdx.x, lane = tid & 63;
  const int wave = __builtin_amdgcn_readfirstlane(tid >> 6);
  const int b = blockIdx.x;
  const int g = b & 7, j = b >> 3;
  const int rb = j & 3, kh = g & 1, cb = (g >> 1) * 8 + (j >> 2);
  const int rank = P.rank;
  char* const myws = P.myws;
  STAMP(0);
  unsigned call;
  unsigned* const tmo = (unsigned*)(myws + WS_TMO);
  constexpr int PITCH = 528;
  LAS unsigned* red = (LAS unsigned*)(smem + RED);

  if (__builtin_expect(wave >= 4, 1)) {
    const int lw = wave - 4;
    const int ordv = P.ord[kh * 8 + (((j >> 2) + 2 * rb) & 7)][lane] & 63;
    const __amdgpu_buffer_rsrc_t sx = __builtin_amdgcn_make_buffer_rsrc((void*)(P.x + (size_t)rb * 128 * K), (short)0, 128 * K * 2, 0x00020000);
    const __amdgpu_buffer_rsrc_t sw = __builtin_amdgcn_make_buffer_rsrc((void*)(P.w + cb * 128), (short)0, K * N * 2 - cb * 256, 0x00020000);
    unsigned vx[4], vw[4];
#pragma unroll
    for (int i = 0; i < 4; ++i) { const int row = (lw * 4 + i) * 8 + (lane >> 3); vx[i] = row * (K * 2) + (((lane & 7) ^ ((row >> 1) & 7)) << 4); }
#pragma unroll
    for (int i = 0; i < 4; ++i) { const int row = (lw * 4 + i) * 4 + (lane >> 4); const int f = ((row & 3) << 2) | ((row >> 2) & 3); vw[i] = row * (N * 2) + (((lane & 15) ^ f) << 4); }
    unsigned vt[2];
#pragma unroll
    for (int i = 0; i < 2; ++i) { const int row = kh * 64 + (lw * 2 + i) * 8 + (lane >> 3); vt[i] = row * (K * 2) + (((lane & 7) ^ ((row >> 1) & 7)) << 4); }
    const unsigned ldsw = (unsigned)(size_t)smem + lw * 4096, ldst = (unsigned)(size_t)smem + kh * 8192 + lw * 2048;
    auto issue = [&](int s) {
      const int kt = __builtin_amdgcn_readlane(ordv, s);
      const unsigned mx = ldsw + (s & (NS - 1)) * TB, mw = mx + WREG, sox = kt * 128, sow = kt * (64 * N * 2);
#pragma unroll
      for (int i = 0; i < 4; ++i) { DMA(mw + i * 1024, vw[i], sw, sow); DMA(mx + i * 1024, vx[i], sx, sox); }
    };
    auto issue_t = [&](int s) {
      const int kt = __builtin_amdgcn_readlane(ordv, s);
      const unsigned mx = ldst + (s & (NS - 1)) * TB, mw = ldsw + (s & (NS - 1)) * TB + WREG, sox = kt * 128, sow = kt * (64 * N * 2);
#pragma unroll
      for (int i = 0; i < 4; ++i) { DMA(mw + i * 1024, vw[i], sw, sow); if (i & 1) DMA(mx + (i >> 1) * 1024, vt[i >> 1], sx, sox); }
    };
    static_assert(NS == 4 && NT >= 4, "waits written for NS = 4");
#pragma unroll
    for (int s = 0; s < NS; ++s) issue(s);
    wait_vm<24>(); bar();
    for (int s = 1; s <= NF - 4; ++s) { wait_vm<16>(); bar(); issue(s + 3); }
    wait_vm<16>(); bar(); issue_t(NF);
    wait_vm<14>(); bar(); issue_t(NF + 1);
    for (int s = NF - 1; s <= NK - 4; ++s) { wait_vm<12>(); bar(); issue_t(s + 3); }
    wait_vm<12>(); bar();
    wait_vm<6>(); bar();
    wait_vm<0>(); bar();
    bar();
    call = __builtin_amdgcn_readfirstlane(red[5]);
  } else {
    unsigned cnt0;
    asm volatile("global_load_dword %0, %1, off sc0 sc1" : "=v"(cnt0) : "v"((unsigned*)(myws + WS_CNT) + b) : "memory");
    { char* const fill = P.myout + (size_t)b * 8192 + tid * 16;
      const u32x4 mk = {0x80808080u, 0x80808080u, 0x80808080u, 0x80808080u};
      st16_sys(fill, mk); st16_sys(fill + 4096, mk); }
    const int wm = wave & 1, wn = wave >> 1;
    unsigned ao[2][2], bo[4][2];
    { const int r = lane & 15, q = lane >> 4;
#pragma unroll
      for (int h = 0; h < 2; ++h)
#pragma unroll
        for (int ks = 0; ks < 2; ++ks) ao[h][ks] = ((h ^ kh) * 64 + wm * 32 + r) * 128 + (((4 * ks + q) ^ ((r >> 1) & 7)) << 4);
      const int qq = (lane & 15) >> 2, p = lane & 3;
#pragma unroll
      for (int n = 0; n < 4; ++n)
#pragma unroll
        for (int t = 0; t < 2; ++t) { const int row = 8 * q + 4 * t + qq, ch = 2 * (4 * wn + n) + (p >> 1);
          bo[n][t] = WREG + 256 * row + 16 * (ch ^ (((row & 3) << 2) | ((row >> 2) & 3))) + 8 * (p & 1); }
    }
    f32x4 acc[4][4];
#pragma unroll
    for (int m = 0; m < 4; ++m)
#pragma unroll
      for (int n = 0; n < 4; ++n) acc[m][n] = (f32x4){0.f, 0.f, 0.f, 0.f};
    Frags f0, f1;
    auto rd_a = [&](int m, int st, int ks) { return *(LAS bf16x8*)(smem + ao[m >> 1][ks] + (m & 1) * 2048 + st * TB); };
    auto rd_b = [&](int n, int st, int ks) {
      bf16x4 lo = __builtin_amdgcn_ds_read_tr16_b64_v4i16((LAS bf16x4*)(smem + bo[n][0] + st * TB + ks * 8192));
      bf16x4 hi = __builtin_amdgcn_ds_read_tr16_b64_v4i16((LAS bf16x4*)(smem + bo[n][1] + st * TB + ks * 8192));
      return __builtin_shufflevector(lo, hi, 0, 1, 2, 3, 4, 5, 6, 7); };
    auto half = [&](Frags& fc, Frags& fn, const int st, const int ks, const bool CF, const int RD, auto&& extra) {
      constexpr int OM[16] = {0, 1, 0, 1, 2, 2, 0, 1, 2, 3, 3, 3, 0, 1, 2, 3};
      constexpr int ON[16] = {0, 0, 1, 1, 0, 1, 2, 2, 2, 0, 1, 2, 3, 3, 3, 3};
      if (CF) {
#pragma unroll
        for (int i = 0; i < 16; ++i) {
          if (RD) {
            if (i == 0) fn.b[0] = rd_b(0, st, ks);
            if (i == 1) { fn.a[0] = rd_a(0, st, ks); fn.a[1] = rd_a(1, st, ks); }
            if (i == 3) fn.b[1] = rd_b(1, st, ks);
            if (i == 4 && RD == 2) fn.a[2] = rd_a(2, st, ks);
            if (i == 6) fn.b[2] = rd_b(2, st, ks);
            if (i == 7 && RD == 2) fn.a[3] = rd_a(3, st, ks);
            if (i == 9) fn.b[3] = rd_b(3, st, ks);
          }
          MFMA(acc[OM[i]][ON[i]], fc.b[ON[i]], fc.a[OM[i]]);
          SB0();
        }
      } else {
#pragma unroll
        for (int i = 0; i < 8; ++i) {
          if (RD) {
            if (i == 0) fn.b[0] = rd_b(0, st, ks);
            if (i == 1) { fn.a[0] = rd_a(0, st, ks); fn.a[1] = rd_a(1, st, ks); }
            if (i == 2) fn.b[1] = rd_b(1, st, ks);
            if (i == 3) fn.b[2] = rd_b(2, st, ks);
            if (i == 4) fn.b[3] = rd_b(3, st, ks);
          }
          MFMA(acc[i & 1][i >> 1], fc.b[i >> 1], fc.a[i & 1]);
          extra(i);
          SB0();
        }
      }
    };
    auto none = [](int) {};
    bar();
#pragma unroll
    for (int i = 0; i < 4; ++i) { f0.b[i] = rd_b(i, 0, 0); f0.a[i] = rd_a(i, 0, 0); }
    STAMP(1);
    SB0();
    half(f0, f1, 0, 1, true, 2, none);
    constexpr int NL = (NF - 1) / NS * NS;
    for (int s = 0; s < NL; s += NS) {
#pragma unroll
      for (int u = 1; u <= NS; ++u) { wait_lgkm0(); bar(); SB0(); half(f1, f0, u & (NS - 1), 0, true, 2, none); half(f0, f1, u & (NS - 1), 1, true, 2, none); }
    }
#pragma unroll
    for (int s = NL + 1; s < NF; ++s) { wait_lgkm0(); bar(); SB0(); half(f1, f0, s & (NS - 1), 0, true, 2, none); half(f0, f1, s & (NS - 1), 1, true, 2, none); }
    wait_lgkm0(); bar(); SB0(); half(f1, f0, NF & (NS - 1), 0, true, 1, none);
    asm volatile("s_waitcnt vmcnt(0)\n\tv_readfirstlane_b32 %0, %1" : "=s"(call) : "v"(cnt0) : "memory");
    call += 1;
    const unsigned tag = 1u + (call & 1u);
    { char* dst = myws + WS_XCH + (size_t)(b ^ 1) * 32768 + wave * 8192 + lane * 16;
      half(f0, f1, NF & (NS - 1), 1, false, 1, [&](int i) {
        u32x4 v;
#pragma unroll
        for (int r = 0; r < 4; ++r) v[r] = (__float_as_uint(acc[2 + (i >> 2)][i & 3][r]) & ~3u) | tag;
        asm volatile("global_store_dwordx4 %0, %1, off offset:%2 sc0 sc1\n\ts_nop 1" :: "v"(dst + 4096), "v"(v), "i"(i * 1024 - 4096) : "memory"); }); }
    if (tid == 0) {
      unsigned* cnt = (unsigned*)(myws + WS_CNT) + b;
      asm volatile("global_store_dword %0, %1, off sc0 sc1" :: "v"(cnt), "v"(call) : "memory");
      if (b < NGRP) {
        char* nx = myws + WS_GRP + (size_t)(((call + 1) & 1) * NGRP + b) * 128 + 4;
        asm volatile("global_store_dword %0, %1, off sc0 sc1" :: "v"(nx), "v"(0u) : "memory"); }
    }
    const char* src = myws + WS_XCH + (size_t)b * 32768 + wave * 8192 + lane * 16 + 4096;
    u32x4 v[8];
#define XLOAD "global_load_dwordx4 %0, %8, off offset:-4096 sc0 sc1\n\tglobal_load_dwordx4 %1, %8, off offset:-3072 sc0 sc1\n\t" \
              "global_load_dwordx4 %2, %8, off offset:-2048 sc0 sc1\n\tglobal_load_dwordx4 %3, %8, off offset:-1024 sc0 sc1\n\t" \
              "global_load_dwordx4 %4, %8, off sc0 sc1\n\tglobal_load_dwordx4 %5, %8, off offset:1024 sc0 sc1\n\t" \
              "global_load_dwordx4 %6, %8, off offset:2048 sc0 sc1\n\tglobal_load_dwordx4 %7, %8, off offset:3072 sc0 sc1"
#pragma unroll
    for (int s = NF + 1; s < NK; ++s) {
      wait_lgkm0(); bar(); SB0();
      if (s == NK - XL)
        asm volatile(XLOAD : "=&v"(v[0]), "=&v"(v[1]), "=&v"(v[2]), "=&v"(v[3]), "=&v"(v[4]), "=&v"(v[5]), "=&v"(v[6]), "=&v"(v[7]) : "v"(src) : "memory");
      half(f1, f0, s & (NS - 1), 0, false, 1, none); half(f0, f1, s & (NS - 1), 1, false, 1, none); }
    half(f1, f0, 0, 0, false, 0, none);
    STAMP(2);
    asm volatile("s_waitcnt vmcnt(0)" : "+v"(v[0]), "+v"(v[1]), "+v"(v[2]), "+v"(v[3]), "+v"(v[4]), "+v"(v[5]), "+v"(v[6]), "+v"(v[7]) :: "memory");
    bool dead = false;
    { const unsigned long long t0 = __builtin_amdgcn_s_memrealtime();
      for (;;) {
        unsigned all = tag;
#pragma unroll
        for (int i = 0; i < 8; ++i) all &= (v[i][0] & v[i][1]) & (v[i][2] & v[i][3]);
        if (__all(all != 0)) break;
        if (expired(tmo, call, t0)) { dead = true; break; }
        asm volatile(XLOAD "\n\ts_waitcnt vmcnt(0)" : "=&v"(v[0]), "=&v"(v[1]), "=&v"(v[2]), "=&v"(v[3]), "=&v"(v[4]), "=&v"(v[5]), "=&v"(v[6]), "=&v"(v[7]) : "v"(src) : "memory");
      }
#pragma unroll
      for (int i = 0; i < 8; ++i)
#pragma unroll
        for (int r = 0; r < 4; ++r) acc[i >> 2][i & 3][r] += __uint_as_float(v[i][r] & ~3u);
    }
    STAMP(3);
    float mx = 0.f;
#pragma unroll
    for (int i = 0; i < 8; ++i) mx = fmaxf(fmaxf(mx, fmaxf(acc[i >> 2][i & 3][0], acc[i >> 2][i & 3][1])), fmaxf(acc[i >> 2][i & 3][2], acc[i >> 2][i & 3][3]));
    const unsigned wmx = wave_umax(__float_as_uint(mx));
    if (lane == 0) { red[wave] = dead ? 0xffffffffu : wmx; red[5] = call; }
    wait_lgkm0(); bar();
#pragma unroll
    for (int i = 0; i < 8; ++i)
      *(LAS f32x4*)(smem + (wm * 32 + (i >> 2) * 16 + (lane & 15)) * PITCH + (wn * 64 + (i & 3) * 16 + (lane >> 4) * 4) * 4) = acc[i >> 2][i & 3];
  }

  u32x2 arr;
  char* const grp = myws + WS_GRP + (size_t)((call & 1) * NGRP + (b & (NGRP - 1))) * 128;
  if (wave == 4) {
    const unsigned t = max(max(red[0], red[1]), max(red[2], red[3]));
    const u32x2 one = {1u, 0u};
    if (lane == 0)
      asm volatile("global_atomic_umax %1, %2, off offset:4 sc1\n\tglobal_atomic_add_x2 %0, %1, %3, off sc0 sc1" : "=&v"(arr) : "v"(grp), "v"(t), "v"(one) : "memory");
  }
  wait_lgkm0(); bar();
  f32x4 y[4];
#pragma unroll
  for (int c = 0; c < 4; ++c) y[c] = *(LAS f32x4*)(smem + (tid >> 3) * PITCH + (tid & 7) * 64 + c * 16);
  if (wave == 4) {
    char* pw = P.ws[0];
#pragma unroll
    for (int i = 1; i < NR; ++i) { char* q = P.ws[i]; asm volatile("" : "+s"(q)); pw = (lane >> 3) == i ? q : pw; }
    asm volatile("s_waitcnt vmcnt(0)" : "+v"(arr) :: "memory");
    const unsigned n = __builtin_amdgcn_readfirstlane(arr[0]), gm = __builtin_amdgcn_readfirstlane(arr[1]);
    if ((n & (GSZ - 1)) == GSZ - 1) {
      char* dst = pw + WS_GRAN + (size_t)((call & 1) * NREP + (lane & 7)) * GRAN_SET + (size_t)(rank * NGRP + (b & (NGRP - 1))) * 8;
      const u32x2 gr = {gm, call};
      asm volatile("global_store_dwordx2 %0, %1, off sc0 sc1\n\ts_nop 1" :: "v"(dst), "v"(gr) : "memory");
    }
    static_assert(GRAN_SET == 1024, "one 16-B load per lane");
    const char* gs = myws + WS_GRAN + (size_t)((call & 1) * NREP + (b & (NREP - 1))) * GRAN_SET + lane * 16;
    const unsigned long long t0 = __builtin_amdgcn_s_memrealtime();
    unsigned gmax;
    for (;;) {
      u32x4 a;
      asm volatile("global_load_dwordx4 %0, %1, off sc0 sc1\n\ts_waitcnt vmcnt(0)" : "=&v"(a) : "v"(gs) : "memory");
      if (__all(a[1] == call && a[3] == call)) { gmax = wave_umax(max(a[0], a[2])); break; }
      if (expired(tmo, call, t0)) { gmax = 0xffffffffu; break; }
    }
    if (lane == 0) red[4] = gmax;
  }
  wait_lgkm0(); bar();
  const unsigned gmax = red[4];
  STAMP(4);

  if (gmax != 0xffffffffu) {
    const float amax = __uint_as_float(gmax);
    const float inv = amax > 0.f ? 127.0f / amax : 0.f;
    const unsigned dstrank = (unsigned)cb >> 2;
    char* dst = P.out[dstrank & (NR - 1)] + (size_t)(rank * MS + rb * 128 + kh * 64 + (tid >> 3)) * NCOL + (cb & 3) * 128 + (tid & 7) * 16;
    u32x4 q;
#pragma unroll
    for (int c = 0; c < 4; ++c) {
      unsigned u[4];
#pragma unroll
      for (int r = 0; r < 4; ++r) u[r] = __float_as_uint(__builtin_amdgcn_fmed3f(__builtin_fmaf(y[c][r], inv, 12582912.0f), 12582912.0f, 12583039.0f));
      q[c] = __builtin_amdgcn_perm(__builtin_amdgcn_perm(u[3], u[2], 0x0c0c0400u), __builtin_amdgcn_perm(u[1], u[0], 0x0c0c0400u), 0x05040100u);
    }
    st16_sys(dst, q);
  }
  STAMP(5);

  { const char* mine = P.myout + (size_t)b * 8192 + tid * 16;
    const unsigned long long t0 = __builtin_amdgcn_s_memrealtime();
    for (;;) {
      u32x4 a;
      asm volatile("global_load_dwordx4 %0, %1, off sc0 sc1\n\ts_waitcnt vmcnt(0)" : "=&v"(a) : "v"(mine) : "memory");
      if (__all(((a[0] | a[1] | a[2] | a[3]) & 0x80808080u) == 0)) break;
      if (expired(tmo, call, t0)) break;
      __builtin_amdgcn_s_sleep(2);
    } }
  STAMP(6);
}
}

extern "C" void kernel_launch(int rank, int nranks, ncclComm_t comm, hipStream_t stream,
                              void* const* d_in, const int* in_sizes, int n_in,
                              void* d_out, int out_size, const int* in_dtypes, int out_dtype,
                              void* const* d_out_all, void* const* d_ws_all, size_t ws_bytes) {
  (void)comm; (void)in_dtypes; (void)out_dtype; (void)d_out;
  if (nranks != NR || n_in < 2 || in_sizes[0] != MS * K || in_sizes[1] != K * N || out_size != NR * MS * NCOL || ws_bytes < WS_NEED || (unsigned)rank >= (unsigned)NR) {
    fprintf(stderr, "kernel_launch: unexpected problem shape\n");
    return;
  }
  HIP_CHECK(hipFuncSetAttribute((const void*)fused_kernel, hipFuncAttributeMaxDynamicSharedMemorySize, LDS_BYTES));
  Params P;
  P.x = (const bf16_t*)d_in[0]; P.w = (const bf16_t*)d_in[1];
  for (int p = 0; p < NR; ++p) { P.out[p] = (char*)d_out_all[p]; P.ws[p] = (char*)d_ws_all[p]; }
  P.myws = P.ws[rank]; P.myout = P.out[rank];
  P.rank = rank; P.pad = 0;
  int A1 = 8, A2 = 8, A3 = 8;
  for (int c = 0; c < 16; ++c) {
    const int l0 = c & 7, l1 = (l0 + 1) & 7, kh = c >> 3;
    int n = 0;
    for (int part = 0; part < 2; ++part) {
      const int cnt = part ? NT : NF, base = part ? 2 * NF : kh * NF, a1 = part ? A3 : A1, a2 = part ? A3 : A2;
      for (int t = -16; t < cnt; ++t) {
        if (t + a1 >= 0 && t + a1 < cnt && ((t + a1) & 7) == l0) P.ord[c][n++] = base + t + a1;
        if (t + a2 >= 0 && t + a2 < cnt && ((t + a2) & 7) == l1) P.ord[c][n++] = base + t + a2;
        if (t >= 0 && (t & 7) != l0 && (t & 7) != l1) P.ord[c][n++] = base + t;
      }
    }
    while (n < 64) P.ord[c][n++] = 0;
  }
  hipLaunchKernelGGL(fused_kernel, dim3(NWG), dim3(NTHR), LDS_BYTES, stream, P);
  HIP_CHECK(hipGetLastError());
}
```

```cpp
#include <hip/hip_runtime.h>
#include <rccl/rccl.h>
#include <cstdio>
#include <cstdlib>

#define HIP_CHECK(expr) do { hipError_t _e = (expr); if (_e != hipSuccess) { \
    fprintf(stderr, "HIP error %s at %s:%d\n", hipGetErrorString(_e), __FILE__, __LINE__); \
    return; } } while (0)

namespace {
typedef unsigned short bf16_t;
typedef short bf16x8 __attribute__((ext_vector_type(8)));
typedef short bf16x4 __attribute__((ext_vector_type(4)));
typedef float f32x4 __attribute__((ext_vector_type(4)));
typedef float f32x2 __attribute__((ext_vector_type(2)));
typedef unsigned u32x4 __attribute__((ext_vector_type(4)));
typedef unsigned u32x2 __attribute__((ext_vector_type(2)));
#define LAS __attribute__((address_space(3)))

constexpr int NR = 8;
constexpr int MS = 512, N = 4096, K = 4096;
constexpr int NCOL = N / NR;
constexpr int NWG = 256, NTHR = 512;
#define TAIL 8
#define PACE 1
#define PACEB 576
#define XL 3
constexpr int TB = 16384, NS = 4;
constexpr int NT = TAIL, NF = (64 - NT) / 2, NK = NF + NT;
constexpr int WREG = NS * TB;
constexpr int RED = 2 * NS * TB;
constexpr int STRM = 2304, SLOT = 4 * STRM;
constexpr int HDR = 132, STRL = 3072;
constexpr int LDS_BYTES = RED + 64;
constexpr int OUTB = (NK & 3) * TB, INB = OUTB + WREG;
constexpr int NGRP = 16, GSZ = NWG / NGRP, NREP = 8;
constexpr size_t WS_CNT = 0;
constexpr size_t WS_TMO = 4096;
constexpr size_t WS_GRP = 8192;
constexpr size_t WS_GRAN = 16384;
constexpr size_t GRAN_SET = (size_t)NR * NGRP * 8;
constexpr size_t WS_STAMP = 65536;
constexpr size_t WS_XCH = 1 << 20;
constexpr size_t WS_IN = WS_XCH + (size_t)NWG * 32768;
constexpr size_t WS_NEED = WS_IN + (size_t)2 * NWG * SLOT;
constexpr unsigned long long DEADLINE = 50000000ull;

struct Params {
  const bf16_t* x; const bf16_t* w;
  char* myws; char* myout;
  int rank; int pad;
  char* out[NR]; char* ws[NR];
  unsigned char ord[16][64];
};

#define RSRC(p, n) __builtin_amdgcn_make_buffer_rsrc((void*)(p), (short)0, (int)(n), 0x00020000)
#define STORE3(a, b, c, rs) asm volatile("buffer_store_dwordx4 %0, %3, %4, 0 offen sc0 sc1\n\tbuffer_store_dwordx4 %1, %3, %4, 0 offen offset:1024 sc0 sc1\n\t" \
    "buffer_store_dwordx4 %2, %3, %4, 0 offen offset:2048 sc0 sc1\n\ts_nop 1" :: "v"(a), "v"(b), "v"(c), "v"(o16), "s"(rs) : "memory")
#define DMA(m0v, voff, srd, soff) asm volatile("s_mov_b32 m0, %0\n\ts_nop 0\n\tbuffer_load_dwordx4 %1, %2, %3 offen lds" :: "s"(m0v), "v"(voff), "s"(srd), "s"(soff) : "memory")
template <int V> __device__ __forceinline__ void wait_vm() { asm volatile("s_waitcnt vmcnt(%0)" :: "i"(V) : "memory"); }
__device__ __forceinline__ void wait_lgkm0() { __builtin_amdgcn_s_waitcnt(0xC07F); }
__device__ __forceinline__ void bar() { asm volatile("" ::: "memory"); __builtin_amdgcn_s_barrier(); asm volatile("" ::: "memory"); }
__device__ __forceinline__ void st16_sys(void* p, u32x4 v) { asm volatile("global_store_dwordx4 %0, %1, off sc0 sc1\n\ts_nop 1" :: "v"(p), "v"(v) : "memory"); }
template <int O> __device__ __forceinline__ void st16_sys_o(void* p, u32x4 v) { asm volatile("global_store_dwordx4 %0, %1, off offset:%2 sc0 sc1\n\ts_nop 1" :: "v"(p), "v"(v), "i"(O) : "memory"); }
__device__ __forceinline__ unsigned wave_umax(unsigned v) {
  v = max(v, (unsigned)__builtin_amdgcn_update_dpp(0, (int)v, 0x111, 0xf, 0xf, true));
  v = max(v, (unsigned)__builtin_amdgcn_update_dpp(0, (int)v, 0x112, 0xf, 0xf, true));
  v = max(v, (unsigned)__builtin_amdgcn_update_dpp(0, (int)v, 0x114, 0xf, 0xf, true));
  v = max(v, (unsigned)__builtin_amdgcn_update_dpp(0, (int)v, 0x118, 0xf, 0xf, true));
  v = max(v, (unsigned)__builtin_amdgcn_update_dpp(0, (int)v, 0x142, 0xa, 0xf, true));
  v = max(v, (unsigned)__builtin_amdgcn_update_dpp(0, (int)v, 0x143, 0xc, 0xf, true));
  return (unsigned)__builtin_amdgcn_readlane((int)v, 63);
}
__device__ __forceinline__ unsigned wave_scan(unsigned v) {
  v += (unsigned)__builtin_amdgcn_update_dpp(0, (int)v, 0x111, 0xf, 0xf, true);
  v += (unsigned)__builtin_amdgcn_update_dpp(0, (int)v, 0x112, 0xf, 0xf, true);
  v += (unsigned)__builtin_amdgcn_update_dpp(0, (int)v, 0x114, 0xf, 0xf, true);
  v += (unsigned)__builtin_amdgcn_update_dpp(0, (int)v, 0x118, 0xf, 0xf, true);
  v += (unsigned)__builtin_amdgcn_update_dpp(0, (int)v, 0x142, 0xa, 0xf, true);
  v += (unsigned)__builtin_amdgcn_update_dpp(0, (int)v, 0x143, 0xc, 0xf, true);
  return v;
}
__device__ __forceinline__ bool expired(unsigned* tmo, unsigned call, unsigned long long t0) {
  const unsigned long long dt = __builtin_amdgcn_s_memrealtime() - t0;
  if (dt < 5000) return false;
  if (dt > DEADLINE ||
      (unsigned)__builtin_amdgcn_readfirstlane(__hip_atomic_load(tmo, __ATOMIC_RELAXED, __HIP_MEMORY_SCOPE_AGENT)) == call) {
    __hip_atomic_store(tmo, call, __ATOMIC_RELAXED, __HIP_MEMORY_SCOPE_AGENT);
    return true;
  }
  return false;
}
#define STAMPT(i, t) do { } while (0)
#define STAMP(i) STAMPT(i, 0)

__device__ __forceinline__ unsigned pack4(f32x2 lo, f32x2 hi) {
  return __builtin_amdgcn_perm(__builtin_amdgcn_perm(__float_as_uint(hi[1]), __float_as_uint(hi[0]), 0x0c0c0400u), __builtin_amdgcn_perm(__float_as_uint(lo[1]), __float_as_uint(lo[0]), 0x0c0c0400u), 0x05040100u);
}
struct Frags { bf16x8 a[4]; bf16x8 b[4]; };

#define MFMA(acc, w_, x_) acc = __builtin_amdgcn_mfma_f32_16x16x32_bf16(w_, x_, acc, 0, 0, 0)
#define SB0() __builtin_amdgcn_sched_barrier(0)

__global__ __launch_bounds__(NTHR, 2) void fused_kernel(Params P) {
  extern __shared__ __attribute__((aligned(16))) char smem_[];
  LAS char* smem = (LAS char*)smem_;
  const int tid = threadIdx.x, lane = tid & 63;
  const int wave = __builtin_amdgcn_readfirstlane(tid >> 6);
  const int b = blockIdx.x;
  const int g = b & 7, j = b >> 3;
  const int rb = j & 3, kh = g & 1, cb = (j >> 2) * 4 + (g >> 1);
  const int rank = P.rank;
  char* const myws = P.myws;
  char* myout = P.myout;
  STAMP(0);
  unsigned call;
  unsigned* const tmo = (unsigned*)(myws + WS_TMO);
  LAS unsigned* red = (LAS unsigned*)(smem + RED);

  if (__builtin_expect(wave >= 4, 1)) {
    const int lw = wave - 4;
    const int ordv = P.ord[kh * 8 + (((j >> 2) + 2 * rb) & 7)][lane] & 63;
    const __amdgpu_buffer_rsrc_t sx = __builtin_amdgcn_make_buffer_rsrc((void*)(P.x + (size_t)rb * 128 * K), (short)0, 128 * K * 2, 0x00020000);
    const __amdgpu_buffer_rsrc_t sw = __builtin_amdgcn_make_buffer_rsrc((void*)(P.w + cb * 128), (short)0, K * N * 2 - cb * 256, 0x00020000);
    unsigned vx[4], vw[4];
#pragma unroll
    for (int i = 0; i < 4; ++i) { const int row = (lw * 4 + i) * 8 + (lane >> 3); vx[i] = row * (K * 2) + (((lane & 7) ^ ((row >> 1) & 7)) << 4); }
#pragma unroll
    for (int i = 0; i < 4; ++i) { const int row = (lw * 4 + i) * 4 + (lane >> 4); vw[i] = row * (N * 2) + (((lane & 15) ^ (row & 11)) << 4); }
    unsigned vt[2];
#pragma unroll
    for (int i = 0; i < 2; ++i) { const int row = kh * 64 + (lw * 2 + i) * 8 + (lane >> 3); vt[i] = row * (K * 2) + (((lane & 7) ^ ((row >> 1) & 7)) << 4); }
    const unsigned ldsw = (unsigned)(size_t)smem + lw * 4096, ldst = (unsigned)(size_t)smem + kh * 8192 + lw * 2048;
    auto issue = [&](int s) {
      const int kt = __builtin_amdgcn_readlane(ordv, s);
      const unsigned mx = ldsw + (s & (NS - 1)) * TB, mw = mx + WREG, sox = kt * 128, sow = kt * (64 * N * 2);
#pragma unroll
      for (int i = 0; i < 4; ++i) { DMA(mw + i * 1024, vw[i], sw, sow); DMA(mx + i * 1024, vx[i], sx, sox); }
    };
    auto issue_t = [&](int s) {
      const int kt = __builtin_amdgcn_readlane(ordv, s);
      const unsigned mx = ldst + (s & (NS - 1)) * TB, mw = ldsw + (s & (NS - 1)) * TB + WREG, sox = kt * 128, sow = kt * (64 * N * 2);
#pragma unroll
      for (int i = 0; i < 4; ++i) { DMA(mw + i * 1024, vw[i], sw, sow); if (i & 1) DMA(mx + (i >> 1) * 1024, vt[i >> 1], sx, sox); }
    };
    static_assert(NS == 4 && NT >= 4, "waits written for NS = 4");
#pragma unroll
    for (int s = 0; s < NS; ++s) issue(s);
    wait_vm<24>(); bar();
    for (int s = 1; s <= NF - 4; ++s) { wait_vm<16>(); bar(); issue(s + 3); }
    wait_vm<16>(); bar(); issue_t(NF);
    wait_vm<14>(); bar(); issue_t(NF + 1);
    for (int s = NF - 1; s <= NK - 4; ++s) { wait_vm<12>(); bar(); issue_t(s + 3); }
    wait_vm<12>(); bar();
    wait_vm<6>(); bar();
    wait_vm<0>(); bar();
    call = __builtin_amdgcn_readfirstlane(red[5]);
    const unsigned long long t0 = __builtin_amdgcn_s_memrealtime();
    const unsigned o16 = lane * 16;
    if (lw == 0) {
      static_assert(GRAN_SET == 1024, "one 16-B load per lane");
      const char* gs = myws + WS_GRAN + (size_t)((call & 1) * NREP + (b & (NREP - 1))) * GRAN_SET + o16;
      unsigned gm;
      for (;;) {
        u32x4 a;
        asm volatile("global_load_dwordx4 %0, %1, off sc0 sc1\n\ts_waitcnt vmcnt(0)" : "=&v"(a) : "v"(gs) : "memory");
        if (__all(a[1] == call && a[3] == call)) { gm = wave_umax(max(a[0], a[2])); break; }
        if (expired(tmo, call, t0)) { gm = 0xffffffffu; break; }
      }
      red[4] = gm;
    }
    wait_lgkm0(); bar();
    const unsigned gmax = red[4];
    bool dead = gmax == 0xffffffffu;
    char* const src = myws + WS_IN + (size_t)((call & 1) * NWG + b) * SLOT + lw * STRM;
    unsigned mk = 0, pos = 0;
    while (!dead) {
      u32x4 v[3];
      asm volatile("buffer_load_ushort %3, %4, %6, 0 offen sc0 sc1\n\t"
                   "buffer_load_dwordx4 %0, %5, %6, 0 offen sc0 sc1\n\tbuffer_load_dwordx4 %1, %5, %6, 0 offen offset:1024 sc0 sc1\n\t"
                   "buffer_load_dwordx4 %2, %5, %6, 0 offen offset:2048 sc0 sc1\n\ts_waitcnt vmcnt(0)"
                   : "=&v"(v[0]), "=&v"(v[1]), "=&v"(v[2]), "=&v"(mk) : "v"(lane * 2), "v"(o16), "s"(RSRC(src, STRM)) : "memory");
      const unsigned cnt = __popc(mk), incl = wave_scan(cnt);
      const unsigned len = (HDR + 2 * (unsigned)__builtin_amdgcn_readlane((int)incl, 63) + 15) & ~15u;
      bool ok = mk != 0;
#pragma unroll
      for (int c = 0; c < 3; ++c) ok &= (min(min(v[c][0], v[c][1]), min(v[c][2], v[c][3])) != 0) | (c * 1024 + o16 >= len);
      if (__all(ok)) {
#pragma unroll
        for (int c = 0; c < 3; ++c) *(LAS u32x4*)(smem + INB + lw * STRL + c * 1024 + o16) = v[c];
        const u32x4 z = {0u, 0u, 0u, 0u};
        STORE3(z, z, z, RSRC(src, len));
        pos = 2 * (incl - cnt);
        break;
      }
      dead = expired(tmo, call, t0);
      __builtin_amdgcn_s_sleep(2);
    }
    STAMPT(7, 256);
    char* const mine = myout + (size_t)((b >> 2) * 64 + (lw & 1) * 32 + (lane & 15)) * NCOL + (b & 3) * 128 + (lw >> 1) * 64 + (lane >> 4) * 16;
    { const float wmf = *(LAS float*)(smem + INB + lw * STRL + 128);
      unsigned a = pos + (unsigned)(size_t)smem + INB + lw * STRL + HDR;
      unsigned cd[16]; int nt[16];
#pragma unroll
      for (int i = 0; i < 16; ++i) {
        cd[i] = *(LAS unsigned short*)(size_t)a;
        nt[i] = __builtin_amdgcn_sbfe((int)mk, 15 - i, 1);
        asm("v_mad_i32_i24 %0, %1, -2, %2" : "=v"(a) : "v"(nt[i]), "v"(a));
      }
      SB0();
      const float kq = wmf / fmaxf(__uint_as_float(gmax), 0x1p-110f) * (127.0f / 65534.0f);
      SB0();
      f32x2 f[8];
#pragma unroll
      for (int i = 0; i < 16; ++i) f[i >> 1][i & 1] = __uint_as_float(__builtin_amdgcn_bitop3_b32(cd[i], nt[i], 0x4b000000u, 0xea));
#pragma unroll
      for (int i = 0; i < 8; ++i) f[i] = __builtin_elementwise_fma(f[i] - 8388609.0f, (f32x2){kq, kq}, (f32x2){12582912.0f, 12582912.0f});
      u32x4 q;
#pragma unroll
      for (int c = 0; c < 4; c += 2) { const unsigned d0 = pack4(f[2 * c], f[2 * c + 1]), d1 = pack4(f[2 * c + 2], f[2 * c + 3]);
        q[c] = dead ? 0x80808080u : (lane & 16) ? d1 : d0; q[c + 1] = dead ? 0x80808080u : (lane & 16) ? d0 : d1; }
      st16_sys(mine, q); }
    STAMPT(8, 256);
    for (;;) {
      u32x4 a;
      asm volatile("global_load_dwordx4 %0, %1, off sc0 sc1\n\ts_waitcnt vmcnt(0)" : "=&v"(a) : "v"(mine + 16 * NCOL) : "memory");
      if (__all(((a[0] | a[1] | a[2] | a[3]) & 0x80808080u) == 0)) break;
      if (expired(tmo, call, t0)) break;
      __builtin_amdgcn_s_sleep(1);
    }
    STAMPT(9, 256);
  } else {
    unsigned cnt0;
    asm volatile("global_load_dword %0, %1, off sc0 sc1" : "=v"(cnt0) : "v"((unsigned*)(myws + WS_CNT) + b) : "memory");
    if (tid < 8) red[tid] = 0;
    const unsigned ooff = ((wave & 1) * 32 + 16 + (lane & 15)) * NCOL + (wave >> 1) * 64 + (lane >> 4) * 16;
    { const u32x4 mk = {0x80808080u, 0x80808080u, 0x80808080u, 0x80808080u};
      st16_sys(myout + (size_t)(b >> 2) * 64 * NCOL + (b & 3) * 128 + ooff, mk); }
    char* dout = P.out[((unsigned)cb >> 2) & (NR - 1)] + (size_t)(rank * MS + rb * 128 + kh * 64) * NCOL + (cb & 3) * 128 + ooff;
    asm volatile("" : "+v"(dout));
    char* pw = P.ws[0];
#pragma unroll
    for (int i = 1; i < NR; ++i) { char* q = P.ws[i]; asm volatile("" : "+s"(q)); pw = (lane >> 3) == i ? q : pw; }
    pw += WS_GRAN + (size_t)(lane & 7) * GRAN_SET + (size_t)(rank * NGRP + (b & (NGRP - 1))) * 8;
    char* dslot = P.ws[((unsigned)cb >> 2) & (NR - 1)] + WS_IN + (size_t)(rank * 32 + (rb * 2 + kh) * 4 + (cb & 3)) * SLOT + wave * STRM;
    asm volatile("" : "+v"(pw));
    asm volatile("" : "+s"(dslot));
    const int wm = wave & 1, wn = wave >> 1;
    unsigned ao[2][2], bo[4][2];
    { const int r = lane & 15, q = lane >> 4;
#pragma unroll
      for (int h = 0; h < 2; ++h)
#pragma unroll
        for (int ks = 0; ks < 2; ++ks) ao[h][ks] = ((h ^ kh) * 64 + wm * 32 + r) * 128 + (((4 * ks + q) ^ ((r >> 1) & 7)) << 4);
      const int qq = (lane & 15) >> 2, p = lane & 3;
#pragma unroll
      for (int n = 0; n < 4; ++n)
#pragma unroll
        for (int t = 0; t < 2; ++t) { const int row = 8 * q + 4 * t + qq, ch = 8 * wn + 2 * p + (n >> 1);
          bo[n][t] = WREG + 256 * row + 16 * (ch ^ (row & 11)) + 8 * ((n ^ p) & 1); }
    }
    f32x4 acc[4][4];
#pragma unroll
    for (int m = 0; m < 4; ++m)
#pragma unroll
      for (int n = 0; n < 4; ++n) acc[m][n] = (f32x4){0.f, 0.f, 0.f, 0.f};
    Frags f0, f1;
    auto rd_a = [&](int m, int st, int ks) { return *(LAS bf16x8*)(smem + ao[m >> 1][ks] + (m & 1) * 2048 + st * TB); };
    auto rd_b = [&](int n, int st, int ks) {
      bf16x4 lo = __builtin_amdgcn_ds_read_tr16_b64_v4i16((LAS bf16x4*)(smem + bo[n][0] + st * TB + ks * 8192));
      bf16x4 hi = __builtin_amdgcn_ds_read_tr16_b64_v4i16((LAS bf16x4*)(smem + bo[n][1] + st * TB + ks * 8192));
      return __builtin_shufflevector(lo, hi, 0, 1, 2, 3, 4, 5, 6, 7); };
    auto half = [&](Frags& fc, Frags& fn, const int st, const int ks, const bool CF, const int RD, auto&& extra) {
      constexpr int OM[16] = {0, 1, 0, 1, 2, 2, 0, 1, 2, 3, 3, 3, 0, 1, 2, 3};
      constexpr int ON[16] = {0, 0, 1, 1, 0, 1, 2, 2, 2, 0, 1, 2, 3, 3, 3, 3};
      if (CF) {
#pragma unroll
        for (int i = 0; i < 16; ++i) {
          if (RD) {
            if (i == 0) fn.b[0] = rd_b(0, st, ks);
            if (i == 1) { fn.a[0] = rd_a(0, st, ks); fn.a[1] = rd_a(1, st, ks); }
            if (i == 3) fn.b[1] = rd_b(1, st, ks);
            if (i == 4 && RD == 2) fn.a[2] = rd_a(2, st, ks);
            if (i == 6) fn.b[2] = rd_b(2, st, ks);
            if (i == 7 && RD == 2) fn.a[3] = rd_a(3, st, ks);
            if (i == 9) fn.b[3] = rd_b(3, st, ks);
          }
          MFMA(acc[OM[i]][ON[i]], fc.b[ON[i]], fc.a[OM[i]]);
          SB0();
        }
      } else {
#pragma unroll
        for (int i = 0; i < 8; ++i) {
          if (RD) {
            if (i == 0) fn.b[0] = rd_b(0, st, ks);
            if (i == 1) { fn.a[0] = rd_a(0, st, ks); fn.a[1] = rd_a(1, st, ks); }
            if (i == 2) fn.b[1] = rd_b(1, st, ks);
            if (i == 3) fn.b[2] = rd_b(2, st, ks);
            if (i == 4) fn.b[3] = rd_b(3, st, ks);
          }
          MFMA(acc[i & 1][i >> 1], fc.b[i >> 1], fc.a[i & 1]);
          extra(i);
          SB0();
        }
      }
    };
    auto none = [](int) {};
    wait_lgkm0(); bar();
#pragma unroll
    for (int i = 0; i < 4; ++i) { f0.b[i] = rd_b(i, 0, 0); f0.a[i] = rd_a(i, 0, 0); }
    STAMP(1);
    SB0();
    half(f0, f1, 0, 1, true, 2, none);
    constexpr int NL = (NF - 1) / NS * NS;
    for (int s = 0; s < NL; s += NS) {
#pragma unroll
      for (int u = 1; u <= NS; ++u) { wait_lgkm0(); bar(); SB0(); half(f1, f0, u & (NS - 1), 0, true, 2, none); half(f0, f1, u & (NS - 1), 1, true, 2, none); }
    }
#pragma unroll
    for (int s = NL + 1; s < NF; ++s) { wait_lgkm0(); bar(); SB0(); half(f1, f0, s & (NS - 1), 0, true, 2, none); half(f0, f1, s & (NS - 1), 1, true, 2, none); }
    wait_lgkm0(); bar(); SB0(); half(f1, f0, NF & (NS - 1), 0, true, 1, none);
    asm volatile("s_waitcnt vmcnt(0)\n\tv_readfirstlane_b32 %0, %1" : "=s"(call) : "v"(cnt0) : "memory");
    call += 1;
    const unsigned tag = 1u + (call & 1u);
    { char* dst = myws + WS_XCH + (size_t)(b ^ 1) * 32768 + wave * 8192 + lane * 16;
      half(f0, f1, NF & (NS - 1), 1, false, 1, [&](int i) {
        u32x4 v;
#pragma unroll
        for (int r = 0; r < 4; ++r) v[r] = (__float_as_uint(acc[2 + (i >> 2)][i & 3][r]) & ~3u) | tag;
        asm volatile("global_store_dwordx4 %0, %1, off offset:%2 sc0 sc1\n\ts_nop 1" :: "v"(dst + 4096), "v"(v), "i"(i * 1024 - 4096) : "memory"); }); }
    if (tid == 0) {
      red[5] = call;
      unsigned* cnt = (unsigned*)(myws + WS_CNT) + b;
      asm volatile("global_store_dword %0, %1, off sc0 sc1" :: "v"(cnt), "v"(call) : "memory");
      if (b < NGRP) {
        char* nx = myws + WS_GRP + (size_t)(((call + 1) & 1) * NGRP + b) * 128 + 4;
        asm volatile("global_store_dword %0, %1, off sc0 sc1" :: "v"(nx), "v"(0u) : "memory"); }
    }
    const char* src = myws + WS_XCH + (size_t)b * 32768 + wave * 8192 + lane * 16 + 4096;
    u32x4 v[8];
#define XLOAD "global_load_dwordx4 %0, %8, off offset:-4096 sc0 sc1\n\tglobal_load_dwordx4 %1, %8, off offset:-3072 sc0 sc1\n\t" \
              "global_load_dwordx4 %2, %8, off offset:-2048 sc0 sc1\n\tglobal_load_dwordx4 %3, %8, off offset:-1024 sc0 sc1\n\t" \
              "global_load_dwordx4 %4, %8, off sc0 sc1\n\tglobal_load_dwordx4 %5, %8, off offset:1024 sc0 sc1\n\t" \
              "global_load_dwordx4 %6, %8, off offset:2048 sc0 sc1\n\tglobal_load_dwordx4 %7, %8, off offset:3072 sc0 sc1"
#pragma unroll
    for (int s = NF + 1; s < NK; ++s) {
      wait_lgkm0(); bar(); SB0();
      if (s == NK - XL)
        asm volatile(XLOAD : "=&v"(v[0]), "=&v"(v[1]), "=&v"(v[2]), "=&v"(v[3]), "=&v"(v[4]), "=&v"(v[5]), "=&v"(v[6]), "=&v"(v[7]) : "v"(src) : "memory");
      half(f1, f0, s & (NS - 1), 0, false, 1, none); half(f0, f1, s & (NS - 1), 1, false, 1, none); }
    half(f1, f0, 0, 0, false, 0, none);
    STAMP(2);
    asm volatile("s_waitcnt vmcnt(0)" : "+v"(v[0]), "+v"(v[1]), "+v"(v[2]), "+v"(v[3]), "+v"(v[4]), "+v"(v[5]), "+v"(v[6]), "+v"(v[7]) :: "memory");
    bool dead = false;
    { const unsigned long long t0 = __builtin_amdgcn_s_memrealtime();
      for (;;) {
        unsigned all = tag;
#pragma unroll
        for (int i = 0; i < 8; ++i) all &= (v[i][0] & v[i][1]) & (v[i][2] & v[i][3]);
        if (__all(all != 0)) break;
        if (expired(tmo, call, t0)) { dead = true; break; }
        asm volatile(XLOAD "\n\ts_waitcnt vmcnt(0)" : "=&v"(v[0]), "=&v"(v[1]), "=&v"(v[2]), "=&v"(v[3]), "=&v"(v[4]), "=&v"(v[5]), "=&v"(v[6]), "=&v"(v[7]) : "v"(src) : "memory");
      }
#pragma unroll
      for (int i = 0; i < 8; ++i)
#pragma unroll
        for (int r = 0; r < 4; ++r) acc[i >> 2][i & 3][r] += __uint_as_float(v[i][r] & ~3u);
    }
    STAMP(3);
    float mx = 0.f;
#pragma unroll
    for (int i = 0; i < 8; ++i) mx = fmaxf(fmaxf(mx, fmaxf(acc[i >> 2][i & 3][0], acc[i >> 2][i & 3][1])), fmaxf(acc[i >> 2][i & 3][2], acc[i >> 2][i & 3][3]));
    const unsigned wmx = wave_umax(__float_as_uint(mx));
    unsigned wn_ = 0, wgm = 0;
    if (lane == 0)
      asm volatile("ds_max_u32 %2, %3\n\tds_add_rtn_u32 %0, %2, %4 offset:4\n\tds_read_b32 %1, %2" : "=&v"(wn_), "=&v"(wgm) : "v"(RED), "v"(dead ? 0xffffffffu : wmx), "v"(1u) : "memory");
    unsigned mi = 0xffffffffu;
#pragma unroll
    for (int i = 0; i < 16; ++i) mi = __builtin_amdgcn_alignbit(mi, __float_as_uint(acc[0][i >> 2][i & 3]), 31);
    asm volatile("s_waitcnt lgkmcnt(0)" : "+v"(wn_), "+v"(wgm), "+v"(mi) :: "memory");
    const bool last = __builtin_amdgcn_readfirstlane(wn_) == 3;
    u32x2 arr;
    if (last && lane == 0) {
      char* const grp = myws + WS_GRP + (size_t)((call & 1) * NGRP + (b & (NGRP - 1))) * 128;
      const u32x2 one = {1u, 0u};
      asm volatile("global_atomic_umax %1, %2, off offset:4 sc1\n\tglobal_atomic_add_x2 %0, %1, %3, off sc0 sc1" : "=&v"(arr) : "v"(grp), "v"(wgm), "v"(one) : "memory");
    }
    if (mi == 0xffffffffu) { mi = 0xfffffffeu; acc[0][3][3] = 0.f; }
    const unsigned cnt = __popc(~mi), incl = wave_scan(cnt);
    const unsigned len = HDR + 2 * (unsigned)__builtin_amdgcn_readlane((int)incl, 63);
    const float wmf = fmaxf(__uint_as_float(wmx), 0x1p-110f), ks = 65534.0f / wmf;
    const unsigned ob = OUTB + wave * STRL;
    *(LAS unsigned short*)(smem + ob + lane * 2) = (unsigned short)~mi;
    *(LAS float*)(smem + ob + 128) = wmf;
    *(LAS unsigned short*)(smem + ob + len + lane * 2) = 0xffff;
    { unsigned a = (unsigned)(size_t)smem + ob + HDR + 2 * (incl - cnt);
      const unsigned sink = (unsigned)(size_t)smem + ob + 2560;
#pragma unroll
      for (int i = 0; i < 16; i += 2) {
        const f32x2 y = {acc[0][i >> 2][i & 3], acc[0][i >> 2][(i & 3) + 1]};
        const f32x2 f = __builtin_elementwise_fma(y, (f32x2){ks, ks}, (f32x2){8388609.0f, 8388609.0f});
#pragma unroll
        for (int h = 0; h < 2; ++h) {
          const unsigned t = (unsigned)min(max((int)__float_as_uint(y[h]), -2), 0);
          ((LAS unsigned short*)(size_t)__builtin_amdgcn_bitop3_b32(t, sink, a, 0xca))[i + h] = (unsigned short)__float_as_uint(f[h]);
          a += t;
        }
      } }
    asm volatile("" ::: "memory");
    { const unsigned o16 = lane * 16;
      u32x4 v3[3];
#pragma unroll
      for (int c = 0; c < 3; ++c) v3[c] = *(LAS u32x4*)(smem + ob + c * 1024 + o16);
      if (last) {
        asm volatile("s_waitcnt vmcnt(0)" : "+v"(arr) :: "memory");
        const unsigned n = __builtin_amdgcn_readfirstlane(arr[0]), gm = __builtin_amdgcn_readfirstlane(arr[1]);
        if ((n & (GSZ - 1)) == GSZ - 1) {
          const u32x2 gr = {gm, call};
          asm volatile("global_store_dwordx2 %0, %1, off sc0 sc1\n\ts_nop 1" :: "v"(pw + (size_t)(call & 1) * NREP * GRAN_SET), "v"(gr) : "memory");
        }
      }
      const __amdgpu_buffer_rsrc_t rd = RSRC(dslot + (size_t)(call & 1) * NWG * SLOT, (len + 15) & ~15u);
#pragma unroll
      for (int r = 0; r < PACE; ++r) {
        if (o16 >= r * PACEB && o16 < (r + 1) * PACEB) asm volatile("buffer_store_dwordx4 %0, %1, %2, 0 offen sc0 sc1\n\ts_nop 1" :: "v"(v3[0]), "v"(o16), "s"(rd) : "memory");
        wait_vm<0>();
      }
      if (o16 >= PACE * PACEB) asm volatile("buffer_store_dwordx4 %0, %1, %2, 0 offen sc0 sc1\n\ts_nop 1" :: "v"(v3[0]), "v"(o16), "s"(rd) : "memory");
      asm volatile("buffer_store_dwordx4 %0, %2, %3, 0 offen offset:1024 sc0 sc1\n\tbuffer_store_dwordx4 %1, %2, %3, 0 offen offset:2048 sc0 sc1\n\ts_nop 1" :: "v"(v3[1]), "v"(v3[2]), "v"(o16), "s"(rd) : "memory");
    }
    STAMP(4);
#pragma unroll
    for (int c = 0; c < 4; c += 2)
#pragma unroll
      for (int r = 0; r < 4; ++r) { const float u = fmaxf(acc[1][c][r], 0.f), w = fmaxf(acc[1][c + 1][r], 0.f);
        acc[1][c][r] = (lane & 16) ? w : u; acc[1][c + 1][r] = (lane & 16) ? u : w; }
    asm volatile("" : "+v"(acc[1][0]), "+v"(acc[1][1]), "+v"(acc[1][2]), "+v"(acc[1][3]));
    bar();
    const unsigned gmax = red[4];
    STAMP(5);
    if (gmax != 0xffffffffu) {
      const float inv = 127.0f / fmaxf(__uint_as_float(gmax), 0x1p-110f);
      u32x4 q;
#pragma unroll
      for (int c = 0; c < 4; ++c)
        q[c] = pack4(__builtin_elementwise_fma((f32x2){acc[1][c][0], acc[1][c][1]}, (f32x2){inv, inv}, (f32x2){12582912.0f, 12582912.0f}),
                     __builtin_elementwise_fma((f32x2){acc[1][c][2], acc[1][c][3]}, (f32x2){inv, inv}, (f32x2){12582912.0f, 12582912.0f}));
      st16_sys(dout, q);
    }
    STAMP(6);
  }
}
}

extern "C" void kernel_launch(int rank, int nranks, ncclComm_t comm, hipStream_t stream,
                              void* const* d_in, const int* in_sizes, int n_in,
                              void* d_out, int out_size, const int* in_dtypes, int out_dtype,
                              void* const* d_out_all, void* const* d_ws_all, size_t ws_bytes) {
  (void)comm; (void)in_dtypes; (void)out_dtype; (void)d_out;
  if (nranks != NR || n_in < 2 || in_sizes[0] != MS * K || in_sizes[1] != K * N || out_size != NR * MS * NCOL || ws_bytes < WS_NEED || (unsigned)rank >= (unsigned)NR) {
    fprintf(stderr, "kernel_launch: unexpected problem shape\n");
    return;
  }
  HIP_CHECK(hipFuncSetAttribute((const void*)fused_kernel, hipFuncAttributeMaxDynamicSharedMemorySize, LDS_BYTES));
  Params P;
  P.x = (const bf16_t*)d_in[0]; P.w = (const bf16_t*)d_in[1];
  for (int p = 0; p < NR; ++p) { P.out[p] = (char*)d_out_all[p]; P.ws[p] = (char*)d_ws_all[p]; }
  P.myws = P.ws[rank]; P.myout = P.out[rank];
  P.rank = rank; P.pad = 0;
  int A1 = 8, A2 = 8, A3 = 8;
  for (int c = 0; c < 16; ++c) {
    const int l0 = c & 7, l1 = (l0 + 1) & 7, kh = c >> 3;
    int n = 0;
    for (int part = 0; part < 2; ++part) {
      const int cnt = part ? NT : NF, base = part ? 2 * NF : kh * NF, a1 = part ? A3 : A1, a2 = part ? A3 : A2;
      for (int t = -16; t < cnt; ++t) {
        if (t + a1 >= 0 && t + a1 < cnt && ((t + a1) & 7) == l0) P.ord[c][n++] = base + t + a1;
        if (t + a2 >= 0 && t + a2 < cnt && ((t + a2) & 7) == l1) P.ord[c][n++] = base + t + a2;
        if (t >= 0 && (t & 7) != l0 && (t & 7) != l1) P.ord[c][n++] = base + t;
      }
    }
    while (n < 64) P.ord[c][n++] = 0;
  }
  hipLaunchKernelGGL(fused_kernel, dim3(NWG), dim3(NTHR), LDS_BYTES, stream, P);
  HIP_CHECK(hipGetLastError());
}
```

```cpp
#include <hip/hip_runtime.h>
#include <rccl/rccl.h>
#include <cstdio>
#include <cstdlib>
#include <cstddef>

#define HIP_CHECK(expr) do { hipError_t _e = (expr); if (_e != hipSuccess) { \
    fprintf(stderr, "HIP error %s at %s:%d\n", hipGetErrorString(_e), __FILE__, __LINE__); \
    return; } } while (0)

namespace {
typedef unsigned short bf16_t;
typedef short bf16x8 __attribute__((ext_vector_type(8)));
typedef short bf16x4 __attribute__((ext_vector_type(4)));
typedef float f32x4 __attribute__((ext_vector_type(4)));
typedef float f32x2 __attribute__((ext_vector_type(2)));
typedef unsigned u32x4 __attribute__((ext_vector_type(4)));
typedef unsigned u32x2 __attribute__((ext_vector_type(2)));
#define LAS __attribute__((address_space(3)))

constexpr int NR = 8;
constexpr int MS = 512, N = 4096, K = 4096;
constexpr int NCOL = N / NR;
constexpr int NWG = 256, NTHR = 512;
#define SHR 8
#define PSF 2
#define PACE 1
#define PACEB 576
#define XL 2
constexpr int TB = 16384, NS = 4;
constexpr int NOWN = (64 - SHR) / 2, NT = SHR - PSF, NF = NOWN + PSF, NK = NF + NT;
constexpr int WREG = NS * TB;
constexpr int RED = 2 * NS * TB;
constexpr int STRM = 2304, SLOT = 4 * STRM;
constexpr int HDR = 132, STRL = 3072;
constexpr int LDS_BYTES = RED + 64;
#define LST(s) do { } while (0)
constexpr int OUTB = (NK & 3) * TB, INB = OUTB + WREG;
constexpr int NGRP = 16, GSZ = NWG / NGRP, NREP = 8;
constexpr size_t WS_CNT = 0;
constexpr size_t WS_TMO = 4096;
constexpr size_t WS_GRP = 8192;
constexpr size_t WS_GRAN = 16384;
constexpr size_t GRAN_SET = (size_t)NR * NGRP * 8;
constexpr size_t WS_STAMP = 65536;
constexpr size_t WS_XCH = 1 << 20;
constexpr size_t WS_IN = WS_XCH + (size_t)NWG * 32768;
constexpr size_t WS_NEED = WS_IN + (size_t)2 * NWG * SLOT;
constexpr unsigned long long DEADLINE = 50000000ull;

struct Params {
  const bf16_t* x; const bf16_t* w;
  char* myws; char* myout;
  int rank; int pad;
  char* out[NR]; char* ws[NR];
  unsigned char ord[16][64];
};

#define RSRC(p, n) __builtin_amdgcn_make_buffer_rsrc((void*)(p), (short)0, (int)(n), 0x00020000)
#define STORE3(a, b, c, rs) asm volatile("buffer_store_dwordx4 %0, %3, %4, 0 offen sc0 sc1\n\tbuffer_store_dwordx4 %1, %3, %4, 0 offen offset:1024 sc0 sc1\n\t" \
    "buffer_store_dwordx4 %2, %3, %4, 0 offen offset:2048 sc0 sc1\n\ts_nop 1" :: "v"(a), "v"(b), "v"(c), "v"(o16), "s"(rs) : "memory")
#define DBITS ""
#define DMA(m0v, voff, srd, soff) asm volatile("s_mov_b32 m0, %0\n\ts_nop 0\n\tbuffer_load_dwordx4 %1, %2, %3 offen " DBITS " lds" :: "s"(m0v), "v"(voff), "s"(srd), "s"(soff) : "memory")
template <int V> __device__ __forceinline__ void wait_vm() { asm volatile("s_waitcnt vmcnt(%0)" :: "i"(V) : "memory"); }
__device__ __forceinline__ void wait_lgkm0() { __builtin_amdgcn_s_waitcnt(0xC07F); }
__device__ __forceinline__ void bar() { asm volatile("" ::: "memory"); __builtin_amdgcn_s_barrier(); asm volatile("" ::: "memory"); }
__device__ __forceinline__ void st16_sys(void* p, u32x4 v) { asm volatile("global_store_dwordx4 %0, %1, off sc0 sc1\n\ts_nop 1" :: "v"(p), "v"(v) : "memory"); }
template <int O> __device__ __forceinline__ void st16_sys_o(void* p, u32x4 v) { asm volatile("global_store_dwordx4 %0, %1, off offset:%2 sc0 sc1\n\ts_nop 1" :: "v"(p), "v"(v), "i"(O) : "memory"); }
__device__ __forceinline__ unsigned wave_umax(unsigned v) {
  v = max(v, (unsigned)__builtin_amdgcn_update_dpp(0, (int)v, 0x111, 0xf, 0xf, true));
  v = max(v, (unsigned)__builtin_amdgcn_update_dpp(0, (int)v, 0x112, 0xf, 0xf, true));
  v = max(v, (unsigned)__builtin_amdgcn_update_dpp(0, (int)v, 0x114, 0xf, 0xf, true));
  v = max(v, (unsigned)__builtin_amdgcn_update_dpp(0, (int)v, 0x118, 0xf, 0xf, true));
  v = max(v, (unsigned)__builtin_amdgcn_update_dpp(0, (int)v, 0x142, 0xa, 0xf, true));
  v = max(v, (unsigned)__builtin_amdgcn_update_dpp(0, (int)v, 0x143, 0xc, 0xf, true));
  return (unsigned)__builtin_amdgcn_readlane((int)v, 63);
}
__device__ __forceinline__ unsigned wave_scan(unsigned v) {
  v += (unsigned)__builtin_amdgcn_update_dpp(0, (int)v, 0x111, 0xf, 0xf, true);
  v += (unsigned)__builtin_amdgcn_update_dpp(0, (int)v, 0x112, 0xf, 0xf, true);
  v += (unsigned)__builtin_amdgcn_update_dpp(0, (int)v, 0x114, 0xf, 0xf, true);
  v += (unsigned)__builtin_amdgcn_update_dpp(0, (int)v, 0x118, 0xf, 0xf, true);
  v += (unsigned)__builtin_amdgcn_update_dpp(0, (int)v, 0x142, 0xa, 0xf, true);
  v += (unsigned)__builtin_amdgcn_update_dpp(0, (int)v, 0x143, 0xc, 0xf, true);
  return v;
}
__device__ __forceinline__ bool expired(unsigned* tmo, unsigned call, unsigned long long t0) {
  const unsigned long long dt = __builtin_amdgcn_s_memrealtime() - t0;
  if (dt < 5000) return false;
  if (dt > DEADLINE ||
      (unsigned)__builtin_amdgcn_readfirstlane(__hip_atomic_load(tmo, __ATOMIC_RELAXED, __HIP_MEMORY_SCOPE_AGENT)) == call) {
    __hip_atomic_store(tmo, call, __ATOMIC_RELAXED, __HIP_MEMORY_SCOPE_AGENT);
    return true;
  }
  return false;
}
#define STAMPT(i, t) do { } while (0)
#define STAMP(i) STAMPT(i, 0)
#define TS(v) do { } while (0)
#define TSW(i, v) do { } while (0)

__device__ __forceinline__ unsigned pack4(f32x2 lo, f32x2 hi) {
  return __builtin_amdgcn_perm(__builtin_amdgcn_perm(__float_as_uint(hi[1]), __float_as_uint(hi[0]), 0x0c0c0400u), __builtin_amdgcn_perm(__float_as_uint(lo[1]), __float_as_uint(lo[0]), 0x0c0c0400u), 0x05040100u);
}
struct Frags { bf16x8 a[4]; bf16x8 b[4]; };

#define MFMA(acc, w_, x_) acc = __builtin_amdgcn_mfma_f32_16x16x32_bf16(w_, x_, acc, 0, 0, 0)
#define SB0() __builtin_amdgcn_sched_barrier(0)

__global__ __launch_bounds__(NTHR, 2) void fused_kernel(Params P) {
  TS(ta);
  extern __shared__ __attribute__((aligned(16))) char smem_[];
  LAS char* smem = (LAS char*)smem_;
  const int tid = threadIdx.x, lane = tid & 63;
  const int wave = __builtin_amdgcn_readfirstlane(tid >> 6);
  const int b = blockIdx.x;
  const int g = b & 7, j = b >> 3;
  const int rb = j & 3, kh = g & 1, cb = (j >> 2) * 4 + (((g >> 1) + (j >> 2)) & 3);
  unsigned call;
  LAS unsigned* red = (LAS unsigned*)(smem + RED);

  if (__builtin_expect(wave >= 4, 1)) {
    const int lw = wave - 4;
    u32x4 xw; unsigned ord0; int ordv;
    { const unsigned oo = offsetof(Params, ord) + (kh * 8 + (((j >> 2) + 2 * rb) & 7)) * 64;
      asm volatile("s_load_dwordx4 %0, %3, 0x0\n\ts_load_dword %1, %3, %4\n\tglobal_load_ubyte %2, %5, %3"
                   : "=&s"(xw), "=&s"(ord0), "=&v"(ordv) : "s"(__builtin_amdgcn_kernarg_segment_ptr()), "s"(oo), "v"(oo + lane) : "memory"); }
    unsigned vx[4], vw[4];
#pragma unroll
    for (int i = 0; i < 4; ++i) { const int row = (lw * 4 + i) * 8 + (lane >> 3); vx[i] = row * (K * 2) + (((lane & 7) ^ ((row >> 1) & 7)) << 4); }
#pragma unroll
    for (int i = 0; i < 4; ++i) { const int row = (lw * 4 + i) * 4 + (lane >> 4); vw[i] = row * (N * 2) + (((lane & 15) ^ (row & 11)) << 4); }
    unsigned vt[2];
#pragma unroll
    for (int i = 0; i < 2; ++i) { const int row = kh * 64 + (lw * 2 + i) * 8 + (lane >> 3); vt[i] = row * (K * 2) + (((lane & 7) ^ ((row >> 1) & 7)) << 4); }
    const unsigned ldsw = (unsigned)(size_t)smem + lw * 4096, ldst = (unsigned)(size_t)smem + kh * 8192 + lw * 2048;
    asm volatile("s_waitcnt lgkmcnt(0)" : "+s"(xw), "+s"(ord0) :: "memory");
    const char* const px = (const char*)(((unsigned long long)xw[1] << 32) | xw[0]) + (size_t)rb * 128 * K * 2;
    const char* const pwm = (const char*)(((unsigned long long)xw[3] << 32) | xw[2]) + cb * 256;
    const __amdgpu_buffer_rsrc_t sx = RSRC(px, 128 * K * 2), sw = RSRC(pwm, K * N * 2 - cb * 256);
    const unsigned other = (lw >> 1) != kh ? 128 : 0;
    auto issue = [&](int s, unsigned e) {
      const int kt = e & 63;
      const unsigned mx = ldsw + (s & (NS - 1)) * TB, mw = mx + WREG, sox = kt * 128, sow = kt * (64 * N * 2);
      const __amdgpu_buffer_rsrc_t sz = RSRC(px, (e & other) ? 0 : 128 * K * 2);
#pragma unroll
      for (int i = 0; i < 4; ++i) { DMA(mw + i * 1024, vw[i], sw, sow); DMA(mx + i * 1024, vx[i], sz, sox); }
    };
    auto issue_t = [&](int s) {
      const int kt = __builtin_amdgcn_readlane(ordv, s) & 63;
      const unsigned mx = ldst + (s & (NS - 1)) * TB, mw = ldsw + (s & (NS - 1)) * TB + WREG, sox = kt * 128, sow = kt * (64 * N * 2);
#pragma unroll
      for (int i = 0; i < 4; ++i) { DMA(mw + i * 1024, vw[i], sw, sow); if (i & 1) DMA(mx + (i >> 1) * 1024, vt[i >> 1], sx, sox); }
    };
    static_assert(NS == 4 && NT >= 4, "waits written for NS = 4");
    TS(tb);
    issue(0, ord0 & 63);
    TS(tc);
    issue(1, (ord0 >> 8) & 63);
    TS(td);
    issue(2, (ord0 >> 16) & 63);
    TS(te);
    asm volatile("s_waitcnt vmcnt(24)" : "+v"(ordv) :: "memory");
    for (int s = 0; s <= NF - 4; ++s) { wait_vm<16>(); bar(); LST(s); issue(s + 3, __builtin_amdgcn_readlane(ordv, s + 3)); }
    wait_vm<16>(); bar(); LST(NF - 3); issue_t(NF);
    wait_vm<14>(); bar(); LST(NF - 2); issue_t(NF + 1);
    for (int s = NF - 1; s <= NK - 4; ++s) { wait_vm<12>(); bar(); LST(s); issue_t(s + 3); }
    wait_vm<12>(); bar(); LST(NK - 3);
    wait_vm<6>(); bar(); LST(NK - 2);
    wait_vm<0>(); bar(); LST(NK - 1);
    char* const myws = P.myws;
    char* const myout = P.myout;
    unsigned* const tmo = (unsigned*)(myws + WS_TMO);
    call = __builtin_amdgcn_readfirstlane(red[5]);
    const unsigned long long t0 = __builtin_amdgcn_s_memrealtime();
    const unsigned o16 = lane * 16;
    if (lw == 0) {
      static_assert(GRAN_SET == 1024, "one 16-B load per lane");
      const char* gs = myws + WS_GRAN + (size_t)((call & 1) * NREP + (b & (NREP - 1))) * GRAN_SET + o16;
      unsigned gm;
      for (;;) {
        u32x4 a;
        asm volatile("global_load_dwordx4 %0, %1, off sc0 sc1\n\ts_waitcnt vmcnt(0)" : "=&v"(a) : "v"(gs) : "memory");
        if (__all(a[1] == call && a[3] == call)) { gm = wave_umax(max(a[0], a[2])); break; }
        if (expired(tmo, call, t0)) { gm = 0xffffffffu; break; }
      }
      red[4] = gm;
    }
    wait_lgkm0(); bar();
    const unsigned gmax = red[4];
    bool dead = gmax == 0xffffffffu;
    char* const src = myws + WS_IN + (size_t)((call & 1) * NWG + b) * SLOT + lw * STRM;
    unsigned mk = 0, pos = 0;
    while (!dead) {
      u32x4 v[3];
      asm volatile("buffer_load_ushort %3, %4, %6, 0 offen sc0 sc1\n\t"
                   "buffer_load_dwordx4 %0, %5, %6, 0 offen sc0 sc1\n\tbuffer_load_dwordx4 %1, %5, %6, 0 offen offset:1024 sc0 sc1\n\t"
                   "buffer_load_dwordx4 %2, %5, %6, 0 offen offset:2048 sc0 sc1\n\ts_waitcnt vmcnt(0)"
                   : "=&v"(v[0]), "=&v"(v[1]), "=&v"(v[2]), "=&v"(mk) : "v"(lane * 2), "v"(o16), "s"(RSRC(src, STRM)) : "memory");
      const unsigned cnt = __popc(mk), incl = wave_scan(cnt);
      const unsigned len = (HDR + 2 * (unsigned)__builtin_amdgcn_readlane((int)incl, 63) + 15) & ~15u;
      bool ok = mk != 0;
#pragma unroll
      for (int c = 0; c < 3; ++c) ok &= (min(min(v[c][0], v[c][1]), min(v[c][2], v[c][3])) != 0) | (c * 1024 + o16 >= len);
      if (__all(ok)) {
#pragma unroll
        for (int c = 0; c < 3; ++c) *(LAS u32x4*)(smem + INB + lw * STRL + c * 1024 + o16) = v[c];
        const u32x4 z = {0u, 0u, 0u, 0u};
        STORE3(z, z, z, RSRC(src, len));
        pos = 2 * (incl - cnt);
        break;
      }
      dead = expired(tmo, call, t0);
      __builtin_amdgcn_s_sleep(2);
    }
    STAMPT(7, 256);
    char* const mine = myout + (size_t)((b >> 2) * 64 + (lw & 1) * 32 + (lane & 15)) * NCOL + (b & 3) * 128 + (lw >> 1) * 64 + (lane >> 4) * 16;
    { const float wmf = *(LAS float*)(smem + INB + lw * STRL + 128);
      unsigned a = pos + (unsigned)(size_t)smem + INB + lw * STRL + HDR;
      unsigned cd[16]; int nt[16];
#pragma unroll
      for (int i = 0; i < 16; ++i) {
        cd[i] = *(LAS unsigned short*)(size_t)a;
        nt[i] = __builtin_amdgcn_sbfe((int)mk, 15 - i, 1);
        asm("v_mad_i32_i24 %0, %1, -2, %2" : "=v"(a) : "v"(nt[i]), "v"(a));
      }
      SB0();
      const float kq = wmf / fmaxf(__uint_as_float(gmax), 0x1p-110f) * (127.0f / 65534.0f);
      SB0();
      f32x2 f[8];
#pragma unroll
      for (int i = 0; i < 16; ++i) f[i >> 1][i & 1] = __uint_as_float(__builtin_amdgcn_bitop3_b32(cd[i], nt[i], 0x4b000000u, 0xea));
#pragma unroll
      for (int i = 0; i < 8; ++i) f[i] = __builtin_elementwise_fma(f[i] - 8388609.0f, (f32x2){kq, kq}, (f32x2){12582912.0f, 12582912.0f});
      u32x4 q;
#pragma unroll
      for (int c = 0; c < 4; c += 2) { const unsigned d0 = pack4(f[2 * c], f[2 * c + 1]), d1 = pack4(f[2 * c + 2], f[2 * c + 3]);
        q[c] = dead ? 0x80808080u : (lane & 16) ? d1 : d0; q[c + 1] = dead ? 0x80808080u : (lane & 16) ? d0 : d1; }
      st16_sys(mine, q); }
    STAMPT(8, 256);
    for (;;) {
      u32x4 a;
      asm volatile("global_load_dwordx4 %0, %1, off sc0 sc1\n\ts_waitcnt vmcnt(0)" : "=&v"(a) : "v"(mine + 16 * NCOL) : "memory");
      if (__all(((a[0] | a[1] | a[2] | a[3]) & 0x80808080u) == 0)) break;
      if (expired(tmo, call, t0)) break;
      __builtin_amdgcn_s_sleep(1);
    }
    STAMPT(9, 256);
    TSW(11, ta); TSW(12, tb); TSW(13, tc); TSW(14, td); TSW(15, te);
  } else {
    const int rank = P.rank;
    char* const myws = P.myws;
    char* const myout = P.myout;
    unsigned* const tmo = (unsigned*)(myws + WS_TMO);
    STAMP(0);
    unsigned cnt0;
    asm volatile("global_load_dword %0, %1, off sc0 sc1" : "=v"(cnt0) : "v"((unsigned*)(myws + WS_CNT) + b) : "memory");
    if (tid < 8) red[tid] = 0;
    const unsigned ooff = ((wave & 1) * 32 + 16 + (lane & 15)) * NCOL + (wave >> 1) * 64 + (lane >> 4) * 16;
    { const u32x4 mk = {0x80808080u, 0x80808080u, 0x80808080u, 0x80808080u};
      st16_sys(myout + (size_t)(b >> 2) * 64 * NCOL + (b & 3) * 128 + ooff, mk); }
    char* dout = P.out[((unsigned)cb >> 2) & (NR - 1)] + (size_t)(rank * MS + rb * 128 + kh * 64) * NCOL + (cb & 3) * 128 + ooff;
    asm volatile("" : "+v"(dout));
    char* pw = P.ws[0];
#pragma unroll
    for (int i = 1; i < NR; ++i) { char* q = P.ws[i]; asm volatile("" : "+s"(q)); pw = (lane >> 3) == i ? q : pw; }
    pw += WS_GRAN + (size_t)(lane & 7) * GRAN_SET + (size_t)(rank * NGRP + (b & (NGRP - 1))) * 8;
    char* dslot = P.ws[((unsigned)cb >> 2) & (NR - 1)] + WS_IN + (size_t)(rank * 32 + (rb * 2 + kh) * 4 + (cb & 3)) * SLOT + wave * STRM;
    asm volatile("" : "+v"(pw));
    asm volatile("" : "+s"(dslot));
    const int wm = wave & 1, wn = wave >> 1;
    unsigned ao[2][2], bo[4][2];
    { const int r = lane & 15, q = lane >> 4;
#pragma unroll
      for (int h = 0; h < 2; ++h)
#pragma unroll
        for (int ks = 0; ks < 2; ++ks) ao[h][ks] = ((h ^ kh) * 64 + wm * 32 + r) * 128 + (((4 * ks + q) ^ ((r >> 1) & 7)) << 4);
      const int qq = (lane & 15) >> 2, p = lane & 3;
#pragma unroll
      for (int n = 0; n < 4; ++n)
#pragma unroll
        for (int t = 0; t < 2; ++t) { const int row = 8 * q + 4 * t + qq, ch = 8 * wn + 2 * p + (n >> 1);
          bo[n][t] = WREG + 256 * row + 16 * (ch ^ (row & 11)) + 8 * ((n ^ p) & 1); }
    }
    f32x4 acc[4][4];
#pragma unroll
    for (int m = 0; m < 4; ++m)
#pragma unroll
      for (int n = 0; n < 4; ++n) acc[m][n] = (f32x4){0.f, 0.f, 0.f, 0.f};
    Frags f0, f1;
    auto rd_a = [&](int m, int st, int ks) { return *(LAS bf16x8*)(smem + ao[m >> 1][ks] + (m & 1) * 2048 + st * TB); };
    auto rd_b = [&](int n, int st, int ks) {
      bf16x4 lo = __builtin_amdgcn_ds_read_tr16_b64_v4i16((LAS bf16x4*)(smem + bo[n][0] + st * TB + ks * 8192));
      bf16x4 hi = __builtin_amdgcn_ds_read_tr16_b64_v4i16((LAS bf16x4*)(smem + bo[n][1] + st * TB + ks * 8192));
      return __builtin_shufflevector(lo, hi, 0, 1, 2, 3, 4, 5, 6, 7); };
    auto half = [&](Frags& fc, Frags& fn, const int st, const int ks, const bool CF, const int RD, auto&& extra) {
      constexpr int OM[16] = {0, 1, 0, 1, 2, 2, 0, 1, 2, 3, 3, 3, 0, 1, 2, 3};
      constexpr int ON[16] = {0, 0, 1, 1, 0, 1, 2, 2, 2, 0, 1, 2, 3, 3, 3, 3};
      if (CF) {
#pragma unroll
        for (int i = 0; i < 16; ++i) {
          if (RD) {
            if (i == 0) fn.b[0] = rd_b(0, st, ks);
            if (i == 1) { fn.a[0] = rd_a(0, st, ks); fn.a[1] = rd_a(1, st, ks); }
            if (i == 3) fn.b[1] = rd_b(1, st, ks);
            if (i == 4 && RD == 2) fn.a[2] = rd_a(2, st, ks);
            if (i == 6) fn.b[2] = rd_b(2, st, ks);
            if (i == 7 && RD == 2) fn.a[3] = rd_a(3, st, ks);
            if (i == 9) fn.b[3] = rd_b(3, st, ks);
          }
          MFMA(acc[OM[i]][ON[i]], fc.b[ON[i]], fc.a[OM[i]]);
          SB0();
        }
      } else {
#pragma unroll
        for (int i = 0; i < 8; ++i) {
          if (RD) {
            if (i == 0) fn.b[0] = rd_b(0, st, ks);
            if (i == 1) { fn.a[0] = rd_a(0, st, ks); fn.a[1] = rd_a(1, st, ks); }
            if (i == 2) fn.b[1] = rd_b(1, st, ks);
            if (i == 3) fn.b[2] = rd_b(2, st, ks);
            if (i == 4) fn.b[3] = rd_b(3, st, ks);
          }
          MFMA(acc[i & 1][i >> 1], fc.b[i >> 1], fc.a[i & 1]);
          extra(i);
          SB0();
        }
      }
    };
    auto none = [](int) {};
    wait_lgkm0(); bar();
#pragma unroll
    for (int i = 0; i < 4; ++i) { f0.b[i] = rd_b(i, 0, 0); f0.a[i] = rd_a(i, 0, 0); }
    STAMP(1);
    SB0();
    half(f0, f1, 0, 1, true, 2, none);
    constexpr int NL = (NF - 1) / NS * NS;
    for (int s = 0; s < NL; s += NS) {
#pragma unroll
      for (int u = 1; u <= NS; ++u) { wait_lgkm0(); bar(); SB0(); half(f1, f0, u & (NS - 1), 0, true, 2, none); half(f0, f1, u & (NS - 1), 1, true, 2, none); }
    }
#pragma unroll
    for (int s = NL + 1; s < NF; ++s) { wait_lgkm0(); bar(); SB0(); half(f1, f0, s & (NS - 1), 0, true, 2, none); half(f0, f1, s & (NS - 1), 1, true, 2, none); }
    wait_lgkm0(); bar(); SB0(); half(f1, f0, NF & (NS - 1), 0, true, 1, none);
    asm volatile("s_waitcnt vmcnt(0)\n\tv_readfirstlane_b32 %0, %1" : "=s"(call) : "v"(cnt0) : "memory");
    call += 1;
    const unsigned tag = 1u + (call & 1u);
    { char* dst = myws + WS_XCH + (size_t)(b ^ 1) * 32768 + wave * 8192 + lane * 16;
      half(f0, f1, NF & (NS - 1), 1, false, 1, [&](int i) {
        u32x4 v;
#pragma unroll
        for (int r = 0; r < 4; ++r) v[r] = (__float_as_uint(acc[2 + (i >> 2)][i & 3][r]) & ~3u) | tag;
        asm volatile("global_store_dwordx4 %0, %1, off offset:%2 sc0 sc1\n\ts_nop 1" :: "v"(dst + 4096), "v"(v), "i"(i * 1024 - 4096) : "memory"); }); }
    if (tid == 0) {
      red[5] = call;
      unsigned* cnt = (unsigned*)(myws + WS_CNT) + b;
      asm volatile("global_store_dword %0, %1, off sc0 sc1" :: "v"(cnt), "v"(call) : "memory");
      if (b < NGRP) {
        char* nx = myws + WS_GRP + (size_t)(((call + 1) & 1) * NGRP + b) * 128 + 4;
        asm volatile("global_store_dword %0, %1, off sc0 sc1" :: "v"(nx), "v"(0u) : "memory"); }
    }
    const char* src = myws + WS_XCH + (size_t)b * 32768 + wave * 8192 + lane * 16 + 4096;
    u32x4 v[8];
#define XLOAD "global_load_dwordx4 %0, %8, off offset:-4096 sc0 sc1\n\tglobal_load_dwordx4 %1, %8, off offset:-3072 sc0 sc1\n\t" \
              "global_load_dwordx4 %2, %8, off offset:-2048 sc0 sc1\n\tglobal_load_dwordx4 %3, %8, off offset:-1024 sc0 sc1\n\t" \
              "global_load_dwordx4 %4, %8, off sc0 sc1\n\tglobal_load_dwordx4 %5, %8, off offset:1024 sc0 sc1\n\t" \
              "global_load_dwordx4 %6, %8, off offset:2048 sc0 sc1\n\tglobal_load_dwordx4 %7, %8, off offset:3072 sc0 sc1"
#pragma unroll
    for (int s = NF + 1; s < NK; ++s) {
      wait_lgkm0(); bar(); SB0();
      if (s == NK - XL)
        asm volatile(XLOAD : "=&v"(v[0]), "=&v"(v[1]), "=&v"(v[2]), "=&v"(v[3]), "=&v"(v[4]), "=&v"(v[5]), "=&v"(v[6]), "=&v"(v[7]) : "v"(src) : "memory");
      half(f1, f0, s & (NS - 1), 0, false, 1, none); half(f0, f1, s & (NS - 1), 1, false, 1, none); }
    half(f1, f0, 0, 0, false, 0, none);
    STAMP(2);
    asm volatile("s_waitcnt vmcnt(0)" : "+v"(v[0]), "+v"(v[1]), "+v"(v[2]), "+v"(v[3]), "+v"(v[4]), "+v"(v[5]), "+v"(v[6]), "+v"(v[7]) :: "memory");
    bool dead = false;
    { const unsigned long long t0 = __builtin_amdgcn_s_memrealtime();
      for (;;) {
        unsigned all = tag;
#pragma unroll
        for (int i = 0; i < 8; ++i) all &= (v[i][0] & v[i][1]) & (v[i][2] & v[i][3]);
        if (__all(all != 0)) break;
        if (expired(tmo, call, t0)) { dead = true; break; }
        asm volatile(XLOAD "\n\ts_waitcnt vmcnt(0)" : "=&v"(v[0]), "=&v"(v[1]), "=&v"(v[2]), "=&v"(v[3]), "=&v"(v[4]), "=&v"(v[5]), "=&v"(v[6]), "=&v"(v[7]) : "v"(src) : "memory");
      }
#pragma unroll
      for (int i = 0; i < 8; ++i)
#pragma unroll
        for (int r = 0; r < 4; ++r) acc[i >> 2][i & 3][r] += __uint_as_float(v[i][r] & ~3u);
    }
    STAMP(3);
    float mx = 0.f;
#pragma unroll
    for (int i = 0; i < 8; ++i) mx = fmaxf(fmaxf(mx, fmaxf(acc[i >> 2][i & 3][0], acc[i >> 2][i & 3][1])), fmaxf(acc[i >> 2][i & 3][2], acc[i >> 2][i & 3][3]));
    const unsigned wmx = wave_umax(__float_as_uint(mx));
    unsigned wn_ = 0, wgm = 0;
    if (lane == 0)
      asm volatile("ds_max_u32 %2, %3\n\tds_add_rtn_u32 %0, %2, %4 offset:4\n\tds_read_b32 %1, %2" : "=&v"(wn_), "=&v"(wgm) : "v"(RED), "v"(dead ? 0xffffffffu : wmx), "v"(1u) : "memory");
    unsigned mi = 0xffffffffu;
#pragma unroll
    for (int i = 0; i < 16; ++i) mi = __builtin_amdgcn_alignbit(mi, __float_as_uint(acc[0][i >> 2][i & 3]), 31);
    asm volatile("s_waitcnt lgkmcnt(0)" : "+v"(wn_), "+v"(wgm), "+v"(mi) :: "memory");
    const bool last = __builtin_amdgcn_readfirstlane(wn_) == 3;
    u32x2 arr;
    if (last && lane == 0) {
      char* const grp = myws + WS_GRP + (size_t)((call & 1) * NGRP + (b & (NGRP - 1))) * 128;
      const u32x2 one = {1u, 0u};
      asm volatile("global_atomic_umax %1, %2, off offset:4 sc1\n\tglobal_atomic_add_x2 %0, %1, %3, off sc0 sc1" : "=&v"(arr) : "v"(grp), "v"(wgm), "v"(one) : "memory");
    }
    if (mi == 0xffffffffu) { mi = 0xfffffffeu; acc[0][3][3] = 0.f; }
    const unsigned cnt = __popc(~mi), incl = wave_scan(cnt);
    const unsigned len = HDR + 2 * (unsigned)__builtin_amdgcn_readlane((int)incl, 63);
    const float wmf = fmaxf(__uint_as_float(wmx), 0x1p-110f), ks = 65534.0f / wmf;
    const unsigned ob = OUTB + wave * STRL;
    *(LAS unsigned short*)(smem + ob + lane * 2) = (unsigned short)~mi;
    *(LAS float*)(smem + ob + 128) = wmf;
    *(LAS unsigned short*)(smem + ob + len + lane * 2) = 0xffff;
    { unsigned a = (unsigned)(size_t)smem + ob + HDR + 2 * (incl - cnt);
      const unsigned sink = (unsigned)(size_t)smem + ob + 2560;
#pragma unroll
      for (int i = 0; i < 16; i += 2) {
        const f32x2 y = {acc[0][i >> 2][i & 3], acc[0][i >> 2][(i & 3) + 1]};
        const f32x2 f = __builtin_elementwise_fma(y, (f32x2){ks, ks}, (f32x2){8388609.0f, 8388609.0f});
#pragma unroll
        for (int h = 0; h < 2; ++h) {
          const unsigned t = (unsigned)min(max((int)__float_as_uint(y[h]), -2), 0);
          ((LAS unsigned short*)(size_t)__builtin_amdgcn_bitop3_b32(t, sink, a, 0xca))[i + h] = (unsigned short)__float_as_uint(f[h]);
          a += t;
        }
      } }
    asm volatile("" ::: "memory");
    { const unsigned o16 = lane * 16;
      u32x4 v3[3];
#pragma unroll
      for (int c = 0; c < 3; ++c) v3[c] = *(LAS u32x4*)(smem + ob + c * 1024 + o16);
      if (last) {
        asm volatile("s_waitcnt vmcnt(0)" : "+v"(arr) :: "memory");
        const unsigned n = __builtin_amdgcn_readfirstlane(arr[0]), gm = __builtin_amdgcn_readfirstlane(arr[1]);
        if ((n & (GSZ - 1)) == GSZ - 1) {
          const u32x2 gr = {gm, call};
          asm volatile("global_store_dwordx2 %0, %1, off sc0 sc1\n\ts_nop 1" :: "v"(pw + (size_t)(call & 1) * NREP * GRAN_SET), "v"(gr) : "memory");
        }
      }
      const __amdgpu_buffer_rsrc_t rd = RSRC(dslot + (size_t)(call & 1) * NWG * SLOT, (len + 15) & ~15u);
#pragma unroll
      for (int r = 0; r < PACE; ++r) {
        if (o16 >= r * PACEB && o16 < (r + 1) * PACEB) asm volatile("buffer_store_dwordx4 %0, %1, %2, 0 offen sc0 sc1\n\ts_nop 1" :: "v"(v3[0]), "v"(o16), "s"(rd) : "memory");
        wait_vm<0>();
      }
      if (o16 >= PACE * PACEB) asm volatile("buffer_store_dwordx4 %0, %1, %2, 0 offen sc0 sc1\n\ts_nop 1" :: "v"(v3[0]), "v"(o16), "s"(rd) : "memory");
      asm volatile("buffer_store_dwordx4 %0, %2, %3, 0 offen offset:1024 sc0 sc1\n\tbuffer_store_dwordx4 %1, %2, %3, 0 offen offset:2048 sc0 sc1\n\ts_nop 1" :: "v"(v3[1]), "v"(v3[2]), "v"(o16), "s"(rd) : "memory");
    }
    STAMP(4);
#pragma unroll
    for (int c = 0; c < 4; c += 2)
#pragma unroll
      for (int r = 0; r < 4; ++r) { const float u = fmaxf(acc[1][c][r], 0.f), w = fmaxf(acc[1][c + 1][r], 0.f);
        acc[1][c][r] = (lane & 16) ? w : u; acc[1][c + 1][r] = (lane & 16) ? u : w; }
    asm volatile("" : "+v"(acc[1][0]), "+v"(acc[1][1]), "+v"(acc[1][2]), "+v"(acc[1][3]));
    bar();
    const unsigned gmax = red[4];
    STAMP(5);
    if (gmax != 0xffffffffu) {
      const float inv = 127.0f / fmaxf(__uint_as_float(gmax), 0x1p-110f);
      u32x4 q;
#pragma unroll
      for (int c = 0; c < 4; ++c)
        q[c] = pack4(__builtin_elementwise_fma((f32x2){acc[1][c][0], acc[1][c][1]}, (f32x2){inv, inv}, (f32x2){12582912.0f, 12582912.0f}),
                     __builtin_elementwise_fma((f32x2){acc[1][c][2], acc[1][c][3]}, (f32x2){inv, inv}, (f32x2){12582912.0f, 12582912.0f}));
      st16_sys(dout, q);
    }
    STAMP(6);
  }
}
}

extern "C" void kernel_launch(int rank, int nranks, ncclComm_t comm, hipStream_t stream,
                              void* const* d_in, const int* in_sizes, int n_in,
                              void* d_out, int out_size, const int* in_dtypes, int out_dtype,
                              void* const* d_out_all, void* const* d_ws_all, size_t ws_bytes) {
  (void)comm; (void)in_dtypes; (void)out_dtype; (void)d_out;
  if (nranks != NR || n_in < 2 || in_sizes[0] != MS * K || in_sizes[1] != K * N || out_size != NR * MS * NCOL || ws_bytes < WS_NEED || (unsigned)rank >= (unsigned)NR) {
    fprintf(stderr, "kernel_launch: unexpected problem shape\n");
    return;
  }
  HIP_CHECK(hipFuncSetAttribute((const void*)fused_kernel, hipFuncAttributeMaxDynamicSharedMemorySize, LDS_BYTES));
  Params P;
  P.x = (const bf16_t*)d_in[0]; P.w = (const bf16_t*)d_in[1];
  for (int p = 0; p < NR; ++p) { P.out[p] = (char*)d_out_all[p]; P.ws[p] = (char*)d_ws_all[p]; }
  P.myws = P.ws[rank]; P.myout = P.out[rank];
  P.rank = rank; P.pad = 0;
  int A0 = 30, AS = 3, A1 = 80, RO = 2;
  static_assert(SHR == 8 && PSF == 2, "two of the eight shared steps are early ones");
  for (int c = 0; c < 16; ++c) {
    const int l0 = c & 7, l1 = (l0 + 1) & 7, kh = c >> 3;
    int n = 0, key[NK];
    for (int t = 0; t < NK; ++t) {
      const int u = t - 4, v = (u + RO * (u >> 3)) & 7;
      const bool lead = t < 4 ? t == (l0 >> 1) : (v == l0 || v == l1);
      const int a = A0 + u * AS < A1 ? A0 + u * AS : A1;
      key[t] = lead ? t * 10 - (t < 4 ? 100 : a) - 1 : t * 10;
      if (lead && key[t] > NOWN * 10 - 11) key[t] = NOWN * 10 - 11;
    }
    for (int k = -300; k < NK * 10; ++k)
      for (int t = 0; t < NK; ++t) if (key[t] == k) P.ord[c][n++] = t < NOWN ? kh * NOWN + t : (NOWN + t) | 128;
    while (n < 64) P.ord[c][n++] = 0;
  }
  hipLaunchKernelGGL(fused_kernel, dim3(NWG), dim3(NTHR), LDS_BYTES, stream, P);
  HIP_CHECK(hipGetLastError());
}
```

```cpp
#include <hip/hip_runtime.h>
#include <rccl/rccl.h>
#include <cstdio>
#include <cstdlib>
#include <cstddef>

#define HIP_CHECK(expr) do { hipError_t _e = (expr); if (_e != hipSuccess) { \
    fprintf(stderr, "HIP error %s at %s:%d\n", hipGetErrorString(_e), __FILE__, __LINE__); \
    return; } } while (0)

namespace {
typedef unsigned short bf16_t;
typedef short bf16x8 __attribute__((ext_vector_type(8)));
typedef short bf16x4 __attribute__((ext_vector_type(4)));
typedef float f32x4 __attribute__((ext_vector_type(4)));
typedef float f32x2 __attribute__((ext_vector_type(2)));
typedef unsigned u32x4 __attribute__((ext_vector_type(4)));
typedef unsigned u32x2 __attribute__((ext_vector_type(2)));
#define LAS __attribute__((address_space(3)))

constexpr int NR = 8;
constexpr int MS = 512, N = 4096, K = 4096;
constexpr int NCOL = N / NR;
#define NODMA 0
#define SKIPD 0
#define PRESL 2
#define XO 0
#define TD 8
constexpr int NWG = 256, NTHR = 512;
#define SHR 8
#define PSF 2
#define PACEC 384
#define PACEB 512
#define XL 2
constexpr int TB = 16384, NS = 4;
constexpr int NOWN = (64 - SHR) / 2, NT = SHR - PSF, NF = NOWN + PSF, NK = NF + NT;
constexpr int WREG = NS * TB;
constexpr int RED = 2 * NS * TB;
#define EV 16
constexpr int HDR = 260, STRA = 512 + 128 * EV, STRB = 64 * (32 - EV), STRM = STRA + STRB, SLOT = 4 * STRM, STRL = 3072;
static_assert(EV % 4 == 0 && EV >= 4 && EV <= 20, "whole words");
#define BP 3
#define EW 3
#define RS 0
constexpr int LDS_BYTES = RED + 512;
#define LST(s) do { } while (0)
static_assert(NK % NS == 0, "the last K-step is in the last stage");
constexpr int OUTA = 0, OUTB = TB, INA = WREG, INB = WREG + TB;
constexpr int NGRP = 16, GSZ = NWG / NGRP, NREP = 8;
constexpr size_t WS_CNT = 0;
constexpr size_t WS_TMO = 4096;
constexpr size_t WS_GRP = 8192;
constexpr size_t WS_GRAN = 16384;
constexpr size_t GRAN_SET = (size_t)NR * NGRP * 8;
constexpr size_t WS_STAMP = 65536;
constexpr size_t WS_XCH = 1 << 20;
constexpr size_t WS_IN = WS_XCH + (size_t)NWG * 32768;
constexpr size_t WS_NEED = WS_IN + (size_t)2 * NWG * SLOT;
constexpr unsigned long long DEADLINE = 50000000ull;

struct Params {
  const bf16_t* x; const bf16_t* w;
  char* myws; char* myout;
  int rank; int pad;
  char* out[NR]; char* ws[NR];
  unsigned char ord[32][64];
};

#define RSRC(p, n) __builtin_amdgcn_make_buffer_rsrc((void*)(p), (short)0, (int)(n), 0x00020000)
#define STORE3(a, b, c, rs) asm volatile("buffer_store_dwordx4 %0, %3, %4, 0 offen sc0 sc1\n\tbuffer_store_dwordx4 %1, %3, %4, 0 offen offset:1024 sc0 sc1\n\t" \
    "buffer_store_dwordx4 %2, %3, %4, 0 offen offset:2048 sc0 sc1\n\ts_nop 1" :: "v"(a), "v"(b), "v"(c), "v"(o16), "s"(rs) : "memory")
#define WBITS ""
#define XBITS ""
#define TBITS ""
#define DMA(m0v, voff, srd, soff, bits) asm volatile("s_mov_b32 m0, %0\n\ts_nop 0\n\tbuffer_load_dwordx4 %1, %2, %3 offen " bits " lds" :: "s"(m0v), "v"(voff), "s"(srd), "s"(soff) : "memory")
template <int V> __device__ __forceinline__ void wait_vm() { asm volatile("s_waitcnt vmcnt(%0)" :: "i"(V) : "memory"); }
__device__ __forceinline__ void wait_lgkm0() { __builtin_amdgcn_s_waitcnt(0xC07F); }
__device__ __forceinline__ void bar() { asm volatile("" ::: "memory"); __builtin_amdgcn_s_barrier(); asm volatile("" ::: "memory"); }
__device__ __forceinline__ unsigned lds_word(unsigned a) {
  unsigned v;
  asm volatile("ds_read_b32 %0, %1\n\ts_waitcnt lgkmcnt(0)" : "=v"(v) : "v"(a) : "memory");
  return __builtin_amdgcn_readfirstlane(v);
}
__device__ __forceinline__ void st16_sys(void* p, u32x4 v) { asm volatile("global_store_dwordx4 %0, %1, off sc0 sc1\n\ts_nop 1" :: "v"(p), "v"(v) : "memory"); }
template <int O> __device__ __forceinline__ void st16_sys_o(void* p, u32x4 v) { asm volatile("global_store_dwordx4 %0, %1, off offset:%2 sc0 sc1\n\ts_nop 1" :: "v"(p), "v"(v), "i"(O) : "memory"); }
__device__ __forceinline__ unsigned wave_umax(unsigned v) {
  v = max(v, (unsigned)__builtin_amdgcn_update_dpp(0, (int)v, 0x111, 0xf, 0xf, true));
  v = max(v, (unsigned)__builtin_amdgcn_update_dpp(0, (int)v, 0x112, 0xf, 0xf, true));
  v = max(v, (unsigned)__builtin_amdgcn_update_dpp(0, (int)v, 0x114, 0xf, 0xf, true));
  v = max(v, (unsigned)__builtin_amdgcn_update_dpp(0, (int)v, 0x118, 0xf, 0xf, true));
  v = max(v, (unsigned)__builtin_amdgcn_update_dpp(0, (int)v, 0x142, 0xa, 0xf, true));
  v = max(v, (unsigned)__builtin_amdgcn_update_dpp(0, (int)v, 0x143, 0xc, 0xf, true));
  return (unsigned)__builtin_amdgcn_readlane((int)v, 63);
}
__device__ __forceinline__ unsigned wave_scan(unsigned v) {
  v += (unsigned)__builtin_amdgcn_update_dpp(0, (int)v, 0x111, 0xf, 0xf, true);
  v += (unsigned)__builtin_amdgcn_update_dpp(0, (int)v, 0x112, 0xf, 0xf, true);
  v += (unsigned)__builtin_amdgcn_update_dpp(0, (int)v, 0x114, 0xf, 0xf, true);
  v += (unsigned)__builtin_amdgcn_update_dpp(0, (int)v, 0x118, 0xf, 0xf, true);
  v += (unsigned)__builtin_amdgcn_update_dpp(0, (int)v, 0x142, 0xa, 0xf, true);
  v += (unsigned)__builtin_amdgcn_update_dpp(0, (int)v, 0x143, 0xc, 0xf, true);
  return v;
}
__device__ __forceinline__ bool expired(unsigned* tmo, unsigned call, unsigned long long t0) {
  const unsigned long long dt = __builtin_amdgcn_s_memrealtime() - t0;
  if (dt < 5000) return false;
  if (dt > DEADLINE ||
      (unsigned)__builtin_amdgcn_readfirstlane(__hip_atomic_load(tmo, __ATOMIC_RELAXED, __HIP_MEMORY_SCOPE_AGENT)) == call) {
    __hip_atomic_store(tmo, call, __ATOMIC_RELAXED, __HIP_MEMORY_SCOPE_AGENT);
    return true;
  }
  return false;
}
#define STAMPT(i, t) do { } while (0)
#define STAMP(i) STAMPT(i, 0)
#define STK(i) do { } while (0)
#define STW(i, t) do { } while (0)
#define TS(v) do { } while (0)
#define TSW(i, v) do { } while (0)

__device__ __forceinline__ unsigned pack4(f32x2 lo, f32x2 hi) {
  return __builtin_amdgcn_perm(__builtin_amdgcn_perm(__float_as_uint(hi[1]), __float_as_uint(hi[0]), 0x0c0c0400u), __builtin_amdgcn_perm(__float_as_uint(lo[1]), __float_as_uint(lo[0]), 0x0c0c0400u), 0x05040100u);
}
struct Frags { bf16x8 a[4]; bf16x4 b[4][2]; };

#define MFMA(acc, w_, x_) acc = __builtin_amdgcn_mfma_f32_16x16x32_bf16(w_, x_, acc, 0, 0, 0)
#define SB0() __builtin_amdgcn_sched_barrier(0)

#define TOUCH(sink, base, off) asm volatile("s_load_dwordx2 %0, %1, %2 offset:60" : "+s"(sink) : "s"(base), "s"(off) : "memory")

__global__ __launch_bounds__(NTHR) void fused_kernel(Params P) {
  TS(ta);
  extern __shared__ __attribute__((aligned(16))) char smem_[];
  LAS char* smem = (LAS char*)smem_;
  const int tid = threadIdx.x, lane = tid & 63;
  const int wave = __builtin_amdgcn_readfirstlane(tid >> 6);
  const int b = blockIdx.x;
  const int g = b & 7, j = b >> 3;
  const int rb = j & 3, kh = g & 1, cb = (j >> 2) * 4 + (((g >> 1) + (j >> 2)) & 3);
  unsigned call;
  LAS unsigned* red = (LAS unsigned*)(smem + RED);

  if (__builtin_expect(wave >= 4, 1)) {
    const int lw = wave - 4;
    u32x4 xw; unsigned ord0; int ordv;
    { const unsigned oo = offsetof(Params, ord) + (((cb & 3) == 1) * 16 + kh * 8 + (((j >> 2) + 2 * rb) & 7)) * 64;
      asm volatile("s_load_dwordx4 %0, %3, 0x0\n\ts_load_dword %1, %3, %4\n\tglobal_load_ubyte %2, %5, %3"
                   : "=&s"(xw), "=&s"(ord0), "=&v"(ordv) : "s"(__builtin_amdgcn_kernarg_segment_ptr()), "s"(oo), "v"(oo + lane) : "memory"); }
    unsigned vx[4], vw[4];
    const int xo = (j >> 2 & 1) * XO;
#pragma unroll
    for (int i = 0; i < 4; ++i) { const int row = (lw * 4 + (i ^ xo)) * 8 + (lane >> 3); vx[i] = row * (K * 2) + (((lane & 7) ^ ((row >> 1) & 7)) << 4); }
#pragma unroll
    for (int i = 0; i < 4; ++i) { const int row = (lw * 4 + i) * 4 + (lane >> 4); vw[i] = row * (N * 2) + (((lane & 15) ^ (row & 11)) << 4); }
    unsigned vt[2];
#pragma unroll
    for (int i = 0; i < 2; ++i) { const int row = kh * 64 + (lw * 2 + i) * 8 + (lane >> 3); vt[i] = row * (K * 2) + (((lane & 7) ^ ((row >> 1) & 7)) << 4); }
    const unsigned ldsw = (unsigned)(size_t)smem + lw * 4096, ldst = (unsigned)(size_t)smem + kh * 8192 + lw * 2048;
    asm volatile("s_waitcnt lgkmcnt(0)" : "+s"(xw), "+s"(ord0) :: "memory");
    const char* const px = (const char*)(((unsigned long long)xw[1] << 32) | xw[0]) + (size_t)rb * 128 * K * 2;
    const char* const pwm = (const char*)(((unsigned long long)xw[3] << 32) | xw[2]) + cb * 256;
    const __amdgpu_buffer_rsrc_t sx = RSRC(px, 128 * K * 2), sw = RSRC(pwm, K * N * 2 - cb * 256);
    const unsigned other = (lw >> 1) != kh ? 128 : 0;
    const __amdgpu_buffer_rsrc_t s0 = RSRC(px, 0);
    auto issue = [&](int s, unsigned e) {
      const int kt = e & 63;
      const unsigned mx = ldsw + (s & (NS - 1)) * TB, mw = mx + WREG, sox = kt * 128, sow = kt * (64 * N * 2);
      const __amdgpu_buffer_rsrc_t sz = RSRC(px, (e & other) ? 0 : 128 * K * 2);
#pragma unroll
      for (int i = 0; i < 4; ++i) {
        if (SKIPD && i == 4 - SKIPD && s >= 4) asm volatile("s_mov_b64 exec, 0");
        DMA(mw + i * 1024, vw[i], (NODMA & 1) ? s0 : sw, sow, WBITS); DMA(mx + (i ^ xo) * 1024, vx[i], (NODMA & 2) ? s0 : sz, sox, XBITS); }
      if (SKIPD) asm volatile("s_mov_b64 exec, -1");
    };
    auto issue_t = [&](int s) {
      const int kt = __builtin_amdgcn_readlane(ordv, s) & 63;
      const unsigned mx = ldst + (s & (NS - 1)) * TB, mw = ldsw + (s & (NS - 1)) * TB + WREG, sox = kt * 128, sow = kt * (64 * N * 2);
#pragma unroll
      for (int i = 0; i < 4; ++i) { DMA(mw + i * 1024, vw[i], sw, sow, TBITS); if (i & 1) DMA(mx + (i >> 1) * 1024, vt[i >> 1], sx, sox, XBITS); }
    };
    static_assert(NS == 4 && NT >= 4, "waits written for NS = 4");
    TS(tb);
    issue(0, ord0 & 63);
    TS(tc);
    issue(1, (ord0 >> 8) & 63);
    TS(td);
    issue(2, (ord0 >> 16) & 63);
    TS(te);
    asm volatile("s_waitcnt vmcnt(24)" : "+v"(ordv) :: "memory");
    const char* const tw = (const char*)(((unsigned long long)xw[3] << 32) | xw[2]) + ((((1 - (g >> 1)) & 3) + (j >> 4) * 4) * 4 + 1) * 256 + ((j & 15) * 4 + lw) * (N * 2);
    unsigned to = (kh * NOWN + TD + (g >> 1 & 1)) * (64 * N * 2);
    u32x2 sink = {0u, 0u};
    for (int s = 0; s <= NF - 4; ++s) { wait_vm<16>(); bar(); LST(s); issue(s + 3, __builtin_amdgcn_readlane(ordv, s + 3));
      if (TD && s < NOWN - TD) { TOUCH(sink, tw, to); to += (s & 1) ? 2 * 64 * N * 2 - 128 : 128; } }
    wait_vm<16>(); bar(); LST(NF - 3); issue_t(NF);
    wait_vm<14>(); bar(); LST(NF - 2); issue_t(NF + 1);
    for (int s = NF - 1; s <= NK - 4; ++s) { wait_vm<12>(); bar(); LST(s); issue_t(s + 3); }
    wait_vm<12>(); bar(); LST(NK - 3);
    wait_vm<6>(); bar(); LST(NK - 2);
    wait_vm<0>(); bar(); LST(NK - 1);
    asm volatile("s_waitcnt lgkmcnt(0)" : "+s"(sink) :: "memory");
    char* const myws = P.myws;
    char* const myout = P.myout;
    unsigned* const tmo = (unsigned*)(myws + WS_TMO);
    while (lds_word(RED + 4) != 4) __builtin_amdgcn_s_sleep(3);
    call = lds_word(RED + 20);
    const unsigned long long t0 = __builtin_amdgcn_s_memrealtime();
    const unsigned o16 = lane * 16;
    static_assert(GRAN_SET == 1024, "one 16-B load per lane");
    const char* const gs = myws + WS_GRAN + (size_t)((call & 1) * NREP + (b & (NREP - 1))) * GRAN_SET;
    char* const src = myws + WS_IN + (size_t)((call & 1) * NWG + b) * SLOT + lw * STRM;
    bool dead = false, wantg = lw == 0;
    auto maxima = [&](u32x4 a) {
      if (__all(a[1] == call && a[3] == call)) { red[4] = wave_umax(max(a[0], a[2])) | 0x80000000u; wantg = false; STK(12); } };
    const __amdgpu_buffer_rsrc_t rg = RSRC(gs, lw == 0 ? GRAN_SET : 0);
    unsigned mk;
    for (;;) {
      u32x4 a;
      asm volatile("buffer_load_dwordx4 %0, %2, %4, 0 offen sc0 sc1\n\tbuffer_load_dword %1, %3, %5, 0 offen sc0 sc1\n\ts_waitcnt vmcnt(0)"
                   : "=&v"(a), "=&v"(mk) : "v"(o16), "v"(lane * 4), "s"(rg), "s"(RSRC(src, 256)) : "memory");
      if (wantg) maxima(a);
      if (__all(mk != 0)) break;
      if (expired(tmo, call, t0)) { dead = true; break; }
      if (!wantg) __builtin_amdgcn_s_sleep(PRESL);
    }
    const unsigned cnt = __popc(mk >> (32 - EV)) | __popc(mk << EV) << 16, incl = wave_scan(cnt), tot = __builtin_amdgcn_readlane((int)incl, 63);
    const unsigned pos = incl - cnt, len = (HDR + 2 * (tot & 0xffff) + 15) & ~15u, lenb = ((tot >> 16) + 15) & ~15u;
    const __amdgpu_buffer_rsrc_t ra = RSRC(src, len), rb_ = RSRC(src + STRA, lenb);
    u32x4 a, v[3], w[2];
    unsigned gmax = lds_word(RED + 16);
#define LOADS "buffer_load_dwordx4 %0, %6, %7, 0 offen sc0 sc1\n\tbuffer_load_dwordx4 %1, %6, %8, 0 offen sc0 sc1\n\tbuffer_load_dwordx4 %2, %6, %8, 0 offen offset:1024 sc0 sc1\n\t" \
              "buffer_load_dwordx4 %3, %6, %8, 0 offen offset:2048 sc0 sc1\n\tbuffer_load_dwordx4 %4, %6, %9, 0 offen sc0 sc1\n\t.if %10\n\tbuffer_load_dwordx4 %5, %6, %9, 0 offen offset:1024 sc0 sc1\n\t.endif"
#define LOADA : "=&v"(a), "=&v"(v[0]), "=&v"(v[1]), "=&v"(v[2]), "=&v"(w[0]), "=&v"(w[1]) : "v"(o16), "s"(rg), "s"(ra), "s"(RSRC(src + STRA, gmax ? lenb : 0)), "i"(EV < 16) : "memory"
    asm volatile(LOADS LOADA);
    unsigned ca[EV], ba[8], bs[8]; int nt[EV];
    ca[0] = 2 * (pos & 0xffff) + (unsigned)(size_t)smem + INA + lw * STRL + HDR;
#pragma unroll
    for (int i = 0; i < EV; ++i) {
      nt[i] = __builtin_amdgcn_sbfe((int)mk, 31 - i, 1);
      if (i + 1 < EV) asm("v_mad_i32_i24 %0, %1, -2, %2" : "=v"(ca[i + 1]) : "v"(nt[i]), "v"(ca[i]));
      asm volatile("" : "+v"(ca[i]), "+v"(nt[i]));
    }
#pragma unroll
    for (int c = EV / 4; c < 8; ++c) {
      const unsigned o = (pos >> 16) + (c > EV / 4 ? __popc((mk << EV) >> (32 + EV - 4 * c)) : 0);
      const unsigned p = (((mk >> (28 - 4 * c)) & 15) * 0x08040201u >> 3) & 0x01010101u;
      ba[c] = (unsigned)(size_t)smem + INB + lw * 4096 + (o & ~3u);
      bs[c] = __builtin_amdgcn_bitop3_b32(p * 255u, p * 0x01010100u + (o & 3) * 0x01010101u, 0x0c0c0c0cu, 0xca);
      asm volatile("" : "+v"(ba[c]), "+v"(bs[c]));
    }
    char* mine = myout + (size_t)((b >> 2) * 64 + (lw & 1) * 32 + (lane & 15)) * NCOL + (b & 3) * 128 + (lw >> 1) * 64 + (lane >> 4) * 16;
    asm volatile("s_waitcnt vmcnt(0)" : "+v"(a), "+v"(v[0]), "+v"(v[1]), "+v"(v[2]), "+v"(w[0]), "+v"(w[1]), "+v"(mine) :: "memory");
    auto second = [&]() {
      *(LAS u32x4*)(smem + INB + lw * 4096 + o16) = w[0];
      bool ok = (int)(w[0][0] & w[0][1] & w[0][2] & w[0][3]) < 0 || o16 >= lenb;
      if (EV < 16) { *(LAS u32x4*)(smem + INB + lw * 4096 + 1024 + o16) = w[1]; ok &= (int)(w[1][0] & w[1][1] & w[1][2] & w[1][3]) < 0 || 1024 + o16 >= lenb; }
      return (bool)__all(ok); };
    while (!dead) {
      if (wantg) maxima(a);
      bool ok = true;
#pragma unroll
      for (int c = 0; c < 3; ++c) ok &= (min(min(v[c][0], v[c][1]), min(v[c][2], v[c][3])) != 0) | (c * 1024 + o16 >= len);
      if (__all(ok)) break;
      dead = expired(tmo, call, t0);
      if (!wantg) __builtin_amdgcn_s_sleep(PRESL);
      gmax = lds_word(RED + 16);
      asm volatile(LOADS "\n\ts_waitcnt vmcnt(0)" LOADA);
    }
#pragma unroll
    for (int c = 0; c < 3; ++c) *(LAS u32x4*)(smem + INA + lw * STRL + c * 1024 + o16) = v[c];
    bool done = second();
    const bool late = gmax != 0;
    const u32x4 z = {0u, 0u, 0u, 0u};
    STK(11);
    asm volatile("buffer_load_dwordx4 %0, %3, %4, 0 offen sc0 sc1\n\tbuffer_load_dwordx4 %1, %3, %5, 0 offen sc0 sc1\n\t.if %6\n\tbuffer_load_dwordx4 %2, %3, %5, 0 offen offset:1024 sc0 sc1\n\t.endif"
                 : "=&v"(a), "=&v"(w[0]), "=&v"(w[1]) : "v"(o16), "s"(rg), "s"(RSRC(src + STRA, late && !done ? lenb : 0)), "i"(EV < 16) : "memory");
    STORE3(z, z, z, RSRC(src, late ? 0 : len));
    const float wmf = *(LAS float*)(smem + INA + lw * STRL + 256);
    f32x2 f[EV / 2];
#pragma unroll
    for (int i = 0; i < EV; ++i) f[i >> 1][i & 1] = __uint_as_float(__builtin_amdgcn_bitop3_b32(*(LAS unsigned short*)(size_t)ca[i], nt[i], 0x4b000000u, 0xea));
#pragma unroll
    for (int i = 0; i < EV / 2; ++i) { f[i] -= 8388609.0f; asm volatile("" : "+v"(f[i])); }
    if (wantg) {
      asm volatile("s_waitcnt vmcnt(3)" : "+v"(a) :: "memory");
      maxima(a);
      while (wantg) {
        if (dead || expired(tmo, call, t0)) { red[4] = 0xffffffffu; break; }
        u32x4 a2;
        asm volatile("buffer_load_dwordx4 %0, %1, %2, 0 offen sc0 sc1\n\ts_waitcnt vmcnt(0)" : "=&v"(a2) : "v"(o16), "s"(rg) : "memory");
        maxima(a2);
      }
    }
    while (!gmax && !(gmax = lds_word(RED + 16))) __builtin_amdgcn_s_sleep(1);
    dead |= gmax == 0xffffffffu;
    STK(7);
    unsigned q[8];
    auto row = [&](int m) {
      u32x4 r;
#pragma unroll
      for (int c = 0; c < 4; ++c) r[c] = dead ? 0x80808080u : (lane & 16) ? q[4 * m + (c ^ 1)] : q[4 * m + c];
      st16_sys(mine + m * 16 * NCOL, r); };
    { const float kq = wmf / fmaxf(__uint_as_float(gmax & 0x7fffffffu), 0x1p-110f) * (127.0f / 65534.0f);
#pragma unroll
      for (int i = 0; i < EV / 2; ++i) f[i] = __builtin_elementwise_fma(f[i], (f32x2){kq, kq}, (f32x2){12582912.0f, 12582912.0f});
#pragma unroll
      for (int c = 0; c < EV / 4; ++c) { q[c] = pack4(f[2 * c], f[2 * c + 1]); asm volatile("" : "+v"(q[c])); } }
    STK(8);
    asm volatile("s_waitcnt vmcnt(0)" : "+v"(a), "+v"(w[0]), "+v"(w[1]) :: "memory");
    if (late && !done) done = second();
    if (EV >= 16 && !late) row(0);
    int npoll = 0;
    while (!dead && !done) {
      asm volatile("buffer_load_dwordx4 %0, %2, %3, 0 offen sc0 sc1\n\t.if %4\n\tbuffer_load_dwordx4 %1, %2, %3, 0 offen offset:1024 sc0 sc1\n\t.endif\n\ts_waitcnt vmcnt(0)"
                   : "=&v"(w[0]), "=&v"(w[1]) : "v"(o16), "s"(rb_), "i"(EV < 16) : "memory");
      ++npoll;
      done = second();
      if (!done) dead = expired(tmo, call, t0);
    }
    asm volatile("" ::: "memory");
#pragma unroll
    for (int c = EV / 4; c < 8; ++c) { const unsigned lo = ((LAS unsigned*)(size_t)ba[c])[0], hi = ((LAS unsigned*)(size_t)ba[c])[1];
      q[c] = __builtin_amdgcn_perm(hi, lo, bs[c]) & 0x7f7f7f7fu; }
    if (EV < 16 || late) row(0);
    row(1);
    asm volatile("buffer_store_dwordx4 %0, %1, %2, 0 offen sc0 sc1\n\t.if %3\n\tbuffer_store_dwordx4 %0, %1, %2, 0 offen offset:1024 sc0 sc1\n\t.endif\n\ts_nop 1" :: "v"(z), "v"(o16), "s"(rb_), "i"(EV < 16) : "memory");
    if (late) STORE3(z, z, z, ra);
    STK(9);
    STW(7, 256); STW(8, 256); STW(9, 256); STW(11, 320); STW(12, 256);
    TSW(11, ta); TSW(12, tb); TSW(13, tc); TSW(14, td); TSW(15, te);
  } else {
    const int rank = P.rank;
    char* const myws = P.myws;
    unsigned* const tmo = (unsigned*)(myws + WS_TMO);
    STAMP(0);
    unsigned cnt0;
    asm volatile("global_load_dword %0, %1, off sc0 sc1" : "=v"(cnt0) : "v"((unsigned*)(myws + WS_CNT) + b) : "memory");
    if (tid < 8) red[tid] = 0;
    char* pw = P.ws[0];
#pragma unroll
    for (int i = 1; i < NR; ++i) { char* q = P.ws[i]; asm volatile("" : "+s"(q)); pw = (lane >> 3) == i ? q : pw; }
    pw += WS_GRAN + (size_t)(lane & 7) * GRAN_SET + (size_t)(rank * NGRP + (b & (NGRP - 1))) * 8;
    char* dslot = P.ws[((unsigned)cb >> 2) & (NR - 1)] + WS_IN + (size_t)(rank * 32 + (rb * 2 + kh) * 4 + (cb & 3)) * SLOT + wave * STRM;
    asm volatile("" : "+v"(pw));
    asm volatile("" : "+s"(dslot));
    const int wm = wave & 1, wn = wave >> 1;
    const int fast = ((cb & 3) ^ 1) * (cb ^ 31);
    unsigned ao[2][2], bo[4][2];
    { const int r = lane & 15, q = lane >> 4;
#pragma unroll
      for (int h = 0; h < 2; ++h)
#pragma unroll
        for (int ks = 0; ks < 2; ++ks) ao[h][ks] = ((h ^ kh) * 64 + wm * 32 + r) * 128 + (((4 * ks + q) ^ ((r >> 1) & 7)) << 4);
      const int qq = (lane & 15) >> 2, p = lane & 3;
#pragma unroll
      for (int n = 0; n < 4; ++n)
#pragma unroll
        for (int t = 0; t < 2; ++t) { const int row = 8 * q + 4 * t + qq, ch = 8 * wn + 2 * p + (n >> 1);
          bo[n][t] = WREG + 256 * row + 16 * (ch ^ (row & 11)) + 8 * ((n ^ p) & 1); }
    }
    f32x4 acc[4][4];
#pragma unroll
    for (int m = 0; m < 4; ++m)
#pragma unroll
      for (int n = 0; n < 4; ++n) acc[m][n] = (f32x4){0.f, 0.f, 0.f, 0.f};
    Frags f0, f1;
    auto rd = [&](Frags& f, int r, int st, int ks, bool whole) {
      constexpr int RW[12] = {0, 1, 8, 9, 2, 3, 10, 4, 5, 11, 6, 7}, RH[12] = {0, 1, 8, 9, 2, 3, 4, 5, 6, 7, -1, -1};
      const int c = r < 0 || r > 11 ? -1 : whole ? RW[r] : RH[r];
      if (c >= 8) f.a[c - 8] = *(LAS bf16x8*)(smem + ao[(c - 8) >> 1][ks] + (c & 1) * 2048 + st * TB);
      else if (c >= 0) f.b[c >> 1][c & 1] = __builtin_amdgcn_ds_read_tr16_b64_v4i16((LAS bf16x4*)(smem + bo[c >> 1][c & 1] + st * TB + ks * 8192));
    };
#define BF(f, n) __builtin_shufflevector(f.b[n][0], f.b[n][1], 0, 1, 2, 3, 4, 5, 6, 7)
    auto half = [&](Frags& fc, Frags& fn, const int st, const int ks, const bool CF, const int RD, auto&& extra) {
      constexpr int OM[16] = {0, 1, 0, 1, 2, 2, 0, 1, 2, 3, 3, 3, 0, 1, 2, 3};
      constexpr int ON[16] = {0, 0, 1, 1, 0, 1, 2, 2, 2, 0, 1, 2, 3, 3, 3, 3};
      if (CF) {
        const int r0 = ks ? RS : BP + 1;
#pragma unroll
        for (int i = 0; i < 16; ++i) {
          MFMA(acc[OM[i]][ON[i]], BF(fc, ON[i]), fc.a[OM[i]]);
          SB0();
          if (!ks && i == BP) {
            wait_lgkm0(); bar(); }
          rd(fn, i - r0, st, ks, RD == 2);
          if (EW && i == 15) { if (ks) __builtin_amdgcn_s_waitcnt(0xC07F | 5 << 8); else __builtin_amdgcn_s_waitcnt(0xC07F | EW << 8); }
          SB0();
        }
      } else {
        if (!ks && RD) { wait_lgkm0(); bar(); SB0(); }
#pragma unroll
        for (int i = 0; i < 8; ++i) {
          if (RD) { rd(fn, 2 * i, st, ks, false); rd(fn, 2 * i + 1, st, ks, false); }
          MFMA(acc[i & 1][i >> 1], BF(fc, i >> 1), fc.a[i & 1]);
          extra(i);
          SB0();
        }
      }
    };
    auto none = [](int) {};
    wait_lgkm0(); bar();
#pragma unroll
    for (int i = 0; i < 12; ++i) rd(f0, i, 0, 0, true);
    STAMP(1);
    SB0();
    half(f0, f1, 0, 1, true, 2, none);
    constexpr int NL = (NF - 1) / NS * NS;
    for (int s = 0; s < NL; s += NS) {
#pragma unroll
      for (int u = 1; u <= NS; ++u) { half(f1, f0, u & (NS - 1), 0, true, 2, none); half(f0, f1, u & (NS - 1), 1, true, 2, none); }
    }
#pragma unroll
    for (int s = NL + 1; s < NF; ++s) { half(f1, f0, s & (NS - 1), 0, true, 2, none); half(f0, f1, s & (NS - 1), 1, true, 2, none); }
    half(f1, f0, NF & (NS - 1), 0, true, 1, none);
    asm volatile("s_waitcnt vmcnt(0)\n\tv_readfirstlane_b32 %0, %1" : "=s"(call) : "v"(cnt0) : "memory");
    call += 1;
    const unsigned tag = 1u + (call & 1u);
    { char* dst = myws + WS_XCH + (size_t)(b ^ 1) * 32768 + wave * 8192 + lane * 16;
      half(f0, f1, NF & (NS - 1), 1, false, 1, [&](int i) {
        u32x4 v;
#pragma unroll
        for (int r = 0; r < 4; ++r) v[r] = (__float_as_uint(acc[2 + (i >> 2)][i & 3][r]) & ~3u) | tag;
        asm volatile("global_store_dwordx4 %0, %1, off offset:%2 sc0 sc1\n\ts_nop 1" :: "v"(dst + 4096), "v"(v), "i"(i * 1024 - 4096) : "memory"); }); }
    if (tid == 0) {
      unsigned* cnt = (unsigned*)(myws + WS_CNT) + b;
      asm volatile("global_store_dword %0, %1, off sc0 sc1" :: "v"(cnt), "v"(call) : "memory");
      if (b < NGRP) {
        char* nx = myws + WS_GRP + (size_t)(((call + 1) & 1) * NGRP + b) * 128 + 4;
        asm volatile("global_store_dword %0, %1, off sc0 sc1" :: "v"(nx), "v"(0u) : "memory"); }
    }
    const char* src = myws + WS_XCH + (size_t)b * 32768 + wave * 8192 + lane * 16 + 4096;
    u32x4 v[8];
#define XLOAD "global_load_dwordx4 %0, %8, off offset:-4096 sc0 sc1\n\tglobal_load_dwordx4 %1, %8, off offset:-3072 sc0 sc1\n\t" \
              "global_load_dwordx4 %2, %8, off offset:-2048 sc0 sc1\n\tglobal_load_dwordx4 %3, %8, off offset:-1024 sc0 sc1\n\t" \
              "global_load_dwordx4 %4, %8, off sc0 sc1\n\tglobal_load_dwordx4 %5, %8, off offset:1024 sc0 sc1\n\t" \
              "global_load_dwordx4 %6, %8, off offset:2048 sc0 sc1\n\tglobal_load_dwordx4 %7, %8, off offset:3072 sc0 sc1"
#pragma unroll
    for (int s = NF + 1; s < NK; ++s) {
      half(f1, f0, s & (NS - 1), 0, false, 1, [&](int i) {
        if (s == NK - XL && i == 0)
          asm volatile(XLOAD : "=&v"(v[0]), "=&v"(v[1]), "=&v"(v[2]), "=&v"(v[3]), "=&v"(v[4]), "=&v"(v[5]), "=&v"(v[6]), "=&v"(v[7]) : "v"(src) : "memory"); });
      half(f0, f1, s & (NS - 1), 1, false, 1, none); }
    half(f1, f0, 0, 0, false, 0, none);
    STK(2);
    asm volatile("s_waitcnt vmcnt(0)" : "+v"(v[0]), "+v"(v[1]), "+v"(v[2]), "+v"(v[3]), "+v"(v[4]), "+v"(v[5]), "+v"(v[6]), "+v"(v[7]) :: "memory");
    bool dead = false;
    { const unsigned long long t0 = __builtin_amdgcn_s_memrealtime();
      for (;;) {
        unsigned all = tag;
#pragma unroll
        for (int i = 0; i < 8; ++i) all &= (v[i][0] & v[i][1]) & (v[i][2] & v[i][3]);
        if (__all(all != 0)) break;
        if (expired(tmo, call, t0)) { dead = true; break; }
        asm volatile(XLOAD "\n\ts_waitcnt vmcnt(0)" : "=&v"(v[0]), "=&v"(v[1]), "=&v"(v[2]), "=&v"(v[3]), "=&v"(v[4]), "=&v"(v[5]), "=&v"(v[6]), "=&v"(v[7]) : "v"(src) : "memory");
      }
#pragma unroll
      for (int i = 0; i < 8; ++i)
#pragma unroll
        for (int r = 0; r < 4; ++r) acc[i >> 2][i & 3][r] += __uint_as_float(v[i][r] & ~3u);
    }
    STK(3);
    float mx = 0.f;
#pragma unroll
    for (int i = 0; i < 8; ++i) mx = fmaxf(fmaxf(mx, fmaxf(acc[i >> 2][i & 3][0], acc[i >> 2][i & 3][1])), fmaxf(acc[i >> 2][i & 3][2], acc[i >> 2][i & 3][3]));
    const unsigned wmx = wave_umax(__float_as_uint(mx));
    unsigned wn_ = 0, wgm = 0;
    if (lane == 0)
      asm volatile("ds_write_b32 %2, %5 offset:20\n\tds_max_u32 %2, %3\n\tds_add_rtn_u32 %0, %2, %4 offset:4\n\tds_read_b32 %1, %2" : "=&v"(wn_), "=&v"(wgm) : "v"(RED), "v"(dead ? 0xffffffffu : wmx), "v"(1u), "v"(call) : "memory");
    unsigned mi = 0;
#pragma unroll
    for (int i = 0; i < 32; ++i) mi = __builtin_amdgcn_alignbit(mi, __float_as_uint(acc[i >> 4][i >> 2 & 3][i & 3]), 31);
    asm volatile("s_waitcnt lgkmcnt(0)" : "+v"(wn_), "+v"(wgm), "+v"(mi) :: "memory");
    const bool last = __builtin_amdgcn_readfirstlane(wn_) == 3;
    u32x2 arr;
    if (last && lane == 0) {
      char* const grp = myws + WS_GRP + (size_t)((call & 1) * NGRP + (b & (NGRP - 1))) * 128;
      const u32x2 one = {1u, 0u};
      asm volatile("global_atomic_umax %1, %2, off offset:4 sc1\n\tglobal_atomic_add_x2 %0, %1, %3, off sc0 sc1" : "=&v"(arr) : "v"(grp), "v"(wgm), "v"(one) : "memory");
    }
    if (mi == 0xffffffffu) { mi = 0xfffffffeu; acc[1][3][3] = 0.f; }
    const unsigned cnt = __popc(~mi >> (32 - EV)) | __popc(~mi << EV) << 16, incl = wave_scan(cnt), tot = __builtin_amdgcn_readlane((int)incl, 63);
    const unsigned len = HDR + 2 * (tot & 0xffff), pos = incl - cnt;
    const float wmf = fmaxf(__uint_as_float(wmx), 0x1p-110f), ks = 65534.0f / wmf;
    const unsigned oa = OUTA + wave * STRL, ob = OUTB + wave * 8192, o16 = lane * 16;
    *(LAS unsigned*)(smem + oa + lane * 4) = ~mi;
    *(LAS float*)(smem + oa + 256) = wmf;
    *(LAS unsigned short*)(smem + oa + len + lane * 2) = 0xffff;
    { unsigned a = (unsigned)(size_t)smem + oa + HDR + 2 * (pos & 0xffff);
      const unsigned sink = (unsigned)(size_t)smem + oa + 2960;
#pragma unroll
      for (int i = 0; i < EV; i += 2) {
        const f32x2 y = {acc[i >> 4][i >> 2 & 3][i & 3], acc[i >> 4][i >> 2 & 3][(i & 3) + 1]};
        const f32x2 f = __builtin_elementwise_fma(y, (f32x2){ks, ks}, (f32x2){8388609.0f, 8388609.0f});
#pragma unroll
        for (int h = 0; h < 2; ++h) {
          const unsigned t = (unsigned)min(max((int)__float_as_uint(y[h]), -2), 0);
          ((LAS unsigned short*)(size_t)__builtin_amdgcn_bitop3_b32(t, sink, a, 0xca))[i + h] = (unsigned short)__float_as_uint(f[h]);
          a += t;
        }
      } }
    asm volatile("" ::: "memory");
    char* const dst = dslot + (size_t)(call & 1) * NWG * SLOT;
    f32x4 yb[4];
    { u32x4 v3[3];
#pragma unroll
      for (int c = 0; c < 3; ++c) v3[c] = *(LAS u32x4*)(smem + oa + c * 1024 + o16);
      if (last) {
        asm volatile("s_waitcnt vmcnt(0)" : "+v"(arr) :: "memory");
        const unsigned n = __builtin_amdgcn_readfirstlane(arr[0]), gm = __builtin_amdgcn_readfirstlane(arr[1]);
        if ((n & (GSZ - 1)) == GSZ - 1) {
          const u32x2 gr = {gm, call};
          asm volatile("global_store_dwordx2 %0, %1, off sc0 sc1\n\ts_nop 1" :: "v"(pw + (size_t)(call & 1) * NREP * GRAN_SET), "v"(gr) : "memory");
        }
      }
      const __amdgpu_buffer_rsrc_t rd = RSRC(dst, (len + 15) & ~15u);
      if (o16 < PACEB) asm volatile("buffer_store_dwordx4 %0, %1, %2, 0 offen sc0 sc1\n\ts_nop 1" :: "v"(v3[0]), "v"(o16), "s"(rd) : "memory");
      *(LAS float*)(smem + ob + ((tot >> 16) + lane) * 4) = 0.f;
      { unsigned a = (unsigned)(size_t)smem + ob + 4 * (pos >> 16);
        const unsigned sink = (unsigned)(size_t)smem + ob + 8192 - 128;
#pragma unroll
        for (int i = EV; i < 32; ++i) {
          const float y = acc[i >> 4][i >> 2 & 3][i & 3];
          const unsigned t = (unsigned)min(max((int)__float_as_uint(y), -4), 0);
          ((LAS float*)(size_t)__builtin_amdgcn_bitop3_b32(t, sink, a, 0xca))[i - EV] = y;
          a += t;
        } }
      asm volatile("" ::: "memory");
#pragma unroll
      for (int c = 0; c < 4; ++c) yb[c] = *(LAS f32x4*)(smem + ob + lane * 64 + c * 16);
      const unsigned len16 = (len + 15) & ~15u;
      for (unsigned lo = PACEB; lo < len16; ) {
        unsigned known;
        while (!(known = lds_word(RED + 16)) && __builtin_amdgcn_s_getreg(7 | 3 << 11)) { }
        const unsigned hi = known ? len16 : min(lo + PACEC, len16);
        const __amdgpu_buffer_rsrc_t rs = RSRC(dst, hi);
        if (lo < 1024 && o16 >= lo) asm volatile("buffer_store_dwordx4 %0, %1, %2, 0 offen sc0 sc1\n\ts_nop 1" :: "v"(v3[0]), "v"(o16), "s"(rs) : "memory");
        if (hi > 1024 && lo < 2048 && o16 + 1024 >= lo) asm volatile("buffer_store_dwordx4 %0, %1, %2, 0 offen offset:1024 sc0 sc1\n\ts_nop 1" :: "v"(v3[1]), "v"(o16), "s"(rs) : "memory");
        if (hi > 2048 && o16 + 2048 >= lo) asm volatile("buffer_store_dwordx4 %0, %1, %2, 0 offen offset:2048 sc0 sc1\n\ts_nop 1" :: "v"(v3[2]), "v"(o16), "s"(rs) : "memory");
        lo = hi;
      }
    }
    asm volatile("" : "+v"(yb[0]), "+v"(yb[1]), "+v"(yb[2]), "+v"(yb[3]));
    STK(4);
    unsigned gmax;
    while (!(gmax = lds_word(RED + 16))) { }
    STK(5);
    if (gmax != 0xffffffffu) {
      const float inv = 127.0f / fmaxf(__uint_as_float(gmax & 0x7fffffffu), 0x1p-110f);
      const __amdgpu_buffer_rsrc_t rd = RSRC(dst + STRA, ((tot >> 16) + 15) & ~15u);
      auto quant = [&]() {
        u32x4 q;
#pragma unroll
        for (int c = 0; c < 4; ++c)
          q[c] = pack4(__builtin_elementwise_fma((f32x2){yb[c][0], yb[c][1]}, (f32x2){inv, inv}, (f32x2){12583040.0f, 12583040.0f}),
                       __builtin_elementwise_fma((f32x2){yb[c][2], yb[c][3]}, (f32x2){inv, inv}, (f32x2){12583040.0f, 12583040.0f}));
        return q; };
      asm volatile("buffer_store_dwordx4 %0, %1, %2, 0 offen sc0 sc1\n\ts_nop 1" :: "v"(quant()), "v"(o16), "s"(rd) : "memory");
      if (EV < 16 && (tot >> 16) > 1024) {
#pragma unroll
        for (int c = 0; c < 4; ++c) yb[c] = *(LAS f32x4*)(smem + ob + 4096 + lane * 64 + c * 16);
        asm volatile("buffer_store_dwordx4 %0, %1, %2, 0 offen offset:1024 sc0 sc1\n\ts_nop 1" :: "v"(quant()), "v"(o16), "s"(rd) : "memory");
      }
    }
    STK(6);
    STW(2, 0); STW(3, 0); STW(4, 0); STW(5, 0); STW(6, 0);
  }
}
}

extern "C" void kernel_launch(int rank, int nranks, ncclComm_t comm, hipStream_t stream,
                              void* const* d_in, const int* in_sizes, int n_in,
                              void* d_out, int out_size, const int* in_dtypes, int out_dtype,
                              void* const* d_out_all, void* const* d_ws_all, size_t ws_bytes) {
  (void)comm; (void)in_dtypes; (void)out_dtype; (void)d_out;
  if (nranks != NR || n_in < 2 || in_sizes[0] != MS * K || in_sizes[1] != K * N || out_size != NR * MS * NCOL || ws_bytes < WS_NEED || (unsigned)rank >= (unsigned)NR) {
    fprintf(stderr, "kernel_launch: unexpected problem shape\n");
    return;
  }
  HIP_CHECK(hipFuncSetAttribute((const void*)fused_kernel, hipFuncAttributeMaxDynamicSharedMemorySize, LDS_BYTES));
  Params P;
  P.x = (const bf16_t*)d_in[0]; P.w = (const bf16_t*)d_in[1];
  for (int p = 0; p < NR; ++p) { P.out[p] = (char*)d_out_all[p]; P.ws[p] = (char*)d_ws_all[p]; }
  P.myws = P.ws[rank]; P.myout = P.out[rank];
  P.rank = rank; P.pad = 0;
  int A0 = 30, AS = 3, A1 = 80, RO = 2;
  static_assert(SHR == 8 && PSF == 2, "two of the eight shared steps are early ones");
  int DROP = 1;
  for (int c = 0; c < 32; ++c) {
    const int l0 = c & 7, l1 = (l0 + 1) & 7, kh = c >> 3 & 1;
    int n = 0, key[NK], u = -4, early = 0;
    for (int t = 0; t < NK; ++t) {
      const int v = (u + RO * (u >> 3)) & 7;
      const bool lead = t < 4 ? t == (l0 >> 1) : (v == l0 || (v == l1 && !(DROP && TD && c >= 16 && t >= TD && t < NOWN))) && (t < NOWN || early++ < PSF);
      const int a = A0 + (t - 4) * AS < A1 ? A0 + (t - 4) * AS : A1;
      key[t] = lead ? t * 10 - (t < 4 ? 100 : a) - 1 : t * 10;
      if (lead && key[t] > NOWN * 10 - 11) key[t] = NOWN * 10 - 11;
      ++u;
    }
    for (int k = -300; k < NK * 10; ++k)
      for (int t = 0; t < NK; ++t) if (key[t] == k) P.ord[c][n++] = t < NOWN ? kh * NOWN + t : (NOWN + t) | 128;
    while (n < 64) P.ord[c][n++] = 0;
  }
  hipLaunchKernelGGL(fused_kernel, dim3(NWG), dim3(NTHR), LDS_BYTES, stream, P);
  HIP_CHECK(hipGetLastError());
}
```

```cpp
#include <hip/hip_runtime.h>
#include <rccl/rccl.h>
#include <cstdio>
#include <cstdlib>
#include <cstddef>

#define HIP_CHECK(expr) do { hipError_t _e = (expr); if (_e != hipSuccess) { \
    fprintf(stderr, "HIP error %s at %s:%d\n", hipGetErrorString(_e), __FILE__, __LINE__); \
    return; } } while (0)

namespace {
typedef unsigned short bf16_t;
typedef short bf16x8 __attribute__((ext_vector_type(8)));
typedef short bf16x4 __attribute__((ext_vector_type(4)));
typedef float f32x4 __attribute__((ext_vector_type(4)));
typedef float f32x2 __attribute__((ext_vector_type(2)));
typedef unsigned u32x4 __attribute__((ext_vector_type(4)));
typedef unsigned u32x2 __attribute__((ext_vector_type(2)));
#define LAS __attribute__((address_space(3)))

constexpr int NR = 8;
constexpr int MS = 512, N = 4096, K = 4096;
constexpr int NCOL = N / NR;
#define NODMA 0
#define SKIPD 0
#define PRESL 2
#define AOFF 3072
#define PITCH 4096
#define KHX 0
#define PK 0
#define PB "sc0 sc1"
#define XB "sc0 sc1"
#define SB "sc0 sc1"
#define B_NT "sc0 sc1 nt"
#define B_S1 "sc1"
#define B_S1NT "sc1 nt"
#define XD 2
#define XI 7
#define XO 0
#define TD 8
constexpr int NWG = 256, NTHR = 512;
#define PACEC 384
#define PACEB 512
#define XL 4
constexpr int TB = 16384, NS = 4;
constexpr int NOWN = 28, NF = NOWN + 1, NT = 64 - 2 * NF, NK = NF + NT;
constexpr int ST(int s) { return (s + NS - NK % NS) % NS; }
constexpr int WREG = NS * TB;
constexpr int RED = 2 * NS * TB;
#define EV 16
constexpr int HDR = 260, STRA = AOFF, STRB = 64 * (32 - EV), STRM = PITCH, SLOT = 4 * STRM, STRL = 3072;
static_assert(EV % 4 == 0 && EV >= 4 && EV <= 20, "whole words");
#define BP 3
#define EW 3
#define RS 0
constexpr int LDS_BYTES = RED + 512;
#define LST(s) do { } while (0)
constexpr int OUTA = 0, OUTB = TB, INA = WREG, INB = WREG + TB;
constexpr int NGRP = 16, GSZ = NWG / NGRP, NREP = 8;
constexpr size_t WS_CNT = 0;
constexpr size_t WS_TMO = 4096;
constexpr size_t WS_GRP = 8192;
constexpr size_t WS_GRAN = 16384;
constexpr size_t GRAN_SET = (size_t)NR * NGRP * 8;
constexpr size_t WS_STAMP = 65536;
constexpr size_t WS_XCH = 1 << 20;
constexpr size_t WS_IN = WS_XCH + (size_t)NWG * 32768;
constexpr size_t WS_NEED = WS_IN + (size_t)2 * NWG * SLOT;
constexpr unsigned long long DEADLINE = 50000000ull;

struct Params {
  const bf16_t* x; const bf16_t* w;
  char* myws; char* myout;
  int rank; int pad;
  char* out[NR]; char* ws[NR];
  unsigned char ord[32][64];
};

#define RSRC(p, n) __builtin_amdgcn_make_buffer_rsrc((void*)(p), (short)0, (int)(n), 0x00020000)
#define STORE3(a, b, c, rs) asm volatile("buffer_store_dwordx4 %0, %3, %4, 0 offen sc0 sc1\n\tbuffer_store_dwordx4 %1, %3, %4, 0 offen offset:1024 sc0 sc1\n\t" \
    "buffer_store_dwordx4 %2, %3, %4, 0 offen offset:2048 sc0 sc1\n\ts_nop 1" :: "v"(a), "v"(b), "v"(c), "v"(o16), "s"(rs) : "memory")
#define WBITS ""
#define XBITS ""
#define TBITS ""
#define DMA(m0v, voff, srd, soff, bits) asm volatile("s_mov_b32 m0, %0\n\ts_nop 0\n\tbuffer_load_dwordx4 %1, %2, %3 offen " bits " lds" :: "s"(m0v), "v"(voff), "s"(srd), "s"(soff) : "memory")
template <int V> __device__ __forceinline__ void wait_vm() { asm volatile("s_waitcnt vmcnt(%0)" :: "i"(V) : "memory"); }
__device__ __forceinline__ void wait_lgkm0() { __builtin_amdgcn_s_waitcnt(0xC07F); }
__device__ __forceinline__ void bar() { asm volatile("" ::: "memory"); __builtin_amdgcn_s_barrier(); asm volatile("" ::: "memory"); }
__device__ __forceinline__ unsigned lds_word(unsigned a) {
  unsigned v;
  asm volatile("ds_read_b32 %0, %1\n\ts_waitcnt lgkmcnt(0)" : "=v"(v) : "v"(a) : "memory");
  return __builtin_amdgcn_readfirstlane(v);
}
__device__ __forceinline__ void st16_sys(void* p, u32x4 v) { asm volatile("global_store_dwordx4 %0, %1, off sc0 sc1\n\ts_nop 1" :: "v"(p), "v"(v) : "memory"); }
template <int O> __device__ __forceinline__ void st16_sys_o(void* p, u32x4 v) { asm volatile("global_store_dwordx4 %0, %1, off offset:%2 sc0 sc1\n\ts_nop 1" :: "v"(p), "v"(v), "i"(O) : "memory"); }
__device__ __forceinline__ unsigned wave_umax(unsigned v) {
  v = max(v, (unsigned)__builtin_amdgcn_update_dpp(0, (int)v, 0x111, 0xf, 0xf, true));
  v = max(v, (unsigned)__builtin_amdgcn_update_dpp(0, (int)v, 0x112, 0xf, 0xf, true));
  v = max(v, (unsigned)__builtin_amdgcn_update_dpp(0, (int)v, 0x114, 0xf, 0xf, true));
  v = max(v, (unsigned)__builtin_amdgcn_update_dpp(0, (int)v, 0x118, 0xf, 0xf, true));
  v = max(v, (unsigned)__builtin_amdgcn_update_dpp(0, (int)v, 0x142, 0xa, 0xf, true));
  v = max(v, (unsigned)__builtin_amdgcn_update_dpp(0, (int)v, 0x143, 0xc, 0xf, true));
  return (unsigned)__builtin_amdgcn_readlane((int)v, 63);
}
__device__ __forceinline__ unsigned wave_scan(unsigned v) {
  v += (unsigned)__builtin_amdgcn_update_dpp(0, (int)v, 0x111, 0xf, 0xf, true);
  v += (unsigned)__builtin_amdgcn_update_dpp(0, (int)v, 0x112, 0xf, 0xf, true);
  v += (unsigned)__builtin_amdgcn_update_dpp(0, (int)v, 0x114, 0xf, 0xf, true);
  v += (unsigned)__builtin_amdgcn_update_dpp(0, (int)v, 0x118, 0xf, 0xf, true);
  v += (unsigned)__builtin_amdgcn_update_dpp(0, (int)v, 0x142, 0xa, 0xf, true);
  v += (unsigned)__builtin_amdgcn_update_dpp(0, (int)v, 0x143, 0xc, 0xf, true);
  return v;
}
__device__ __forceinline__ bool expired(unsigned* tmo, unsigned call, unsigned long long t0) {
  const unsigned long long dt = __builtin_amdgcn_s_memrealtime() - t0;
  if (dt < 5000) return false;
  if (dt > DEADLINE ||
      (unsigned)__builtin_amdgcn_readfirstlane(__hip_atomic_load(tmo, __ATOMIC_RELAXED, __HIP_MEMORY_SCOPE_AGENT)) == call) {
    __hip_atomic_store(tmo, call, __ATOMIC_RELAXED, __HIP_MEMORY_SCOPE_AGENT);
    return true;
  }
  return false;
}
#define STAMPT(i, t) do { } while (0)
#define STAMP(i) STAMPT(i, 0)
#define STK(i) do { } while (0)
#define STW(i, t) do { } while (0)
#define TS(v) do { } while (0)
#define TSW(i, v) do { } while (0)

__device__ __forceinline__ unsigned pack4(f32x2 lo, f32x2 hi) {
  return __builtin_amdgcn_perm(__builtin_amdgcn_perm(__float_as_uint(hi[1]), __float_as_uint(hi[0]), 0x0c0c0400u), __builtin_amdgcn_perm(__float_as_uint(lo[1]), __float_as_uint(lo[0]), 0x0c0c0400u), 0x05040100u);
}
struct Frags { bf16x8 a[4]; bf16x4 b[4][2]; };

#define MFMA(acc, w_, x_) acc = __builtin_amdgcn_mfma_f32_16x16x32_bf16(w_, x_, acc, 0, 0, 0)
#define SB0() __builtin_amdgcn_sched_barrier(0)

#define TOUCH(sink, base, off) asm volatile("s_load_dwordx2 %0, %1, %2 offset:60" : "+s"(sink) : "s"(base), "s"(off) : "memory")

__global__ __launch_bounds__(NTHR) void fused_kernel(Params P) {
  TS(ta);
  extern __shared__ __attribute__((aligned(16))) char smem_[];
  LAS char* smem = (LAS char*)smem_;
  const int tid = threadIdx.x, lane = tid & 63;
  const int wave = __builtin_amdgcn_readfirstlane(tid >> 6);
  const int b = blockIdx.x;
  const int g = b & 7, j = b >> 3;
  const int rb = j & 3, kh = (g & 1) ^ KHX, cb = (j >> 2) * 4 + (((g >> 1) + (j >> 2)) & 3);
  unsigned call;
  LAS unsigned* red = (LAS unsigned*)(smem + RED);

  if (__builtin_expect(wave >= 4, 1)) {
    const int lw = wave - 4;
    u32x4 xw; unsigned ord0; int ordv;
    { const unsigned oo = offsetof(Params, ord) + (((cb & 3) == 1) * 16 + kh * 8 + (((j >> 2) + 2 * rb) & 7)) * 64;
      asm volatile("s_load_dwordx4 %0, %3, 0x0\n\ts_load_dword %1, %3, %4\n\tglobal_load_ubyte %2, %5, %3"
                   : "=&s"(xw), "=&s"(ord0), "=&v"(ordv) : "s"(__builtin_amdgcn_kernarg_segment_ptr()), "s"(oo), "v"(oo + lane) : "memory"); }
    unsigned vx[4], vw[4];
    const int xo = (j >> 2 & 1) * XO;
#pragma unroll
    for (int i = 0; i < 4; ++i) { const int row = (lw * 4 + (i ^ xo)) * 8 + (lane >> 3); vx[i] = row * (K * 2) + (((lane & 7) ^ ((row >> 1) & 7)) << 4); }
#pragma unroll
    for (int i = 0; i < 4; ++i) { const int row = (lw * 4 + i) * 4 + (lane >> 4); vw[i] = row * (N * 2) + (((lane & 15) ^ (row & 11)) << 4); }
    unsigned vt[2];
#pragma unroll
    for (int i = 0; i < 2; ++i) { const int row = kh * 64 + (lw * 2 + i) * 8 + (lane >> 3); vt[i] = row * (K * 2) + (((lane & 7) ^ ((row >> 1) & 7)) << 4); }
    const unsigned ldsw = (unsigned)(size_t)smem + lw * 4096, ldst = (unsigned)(size_t)smem + kh * 8192 + lw * 2048;
    asm volatile("s_waitcnt lgkmcnt(0)" : "+s"(xw), "+s"(ord0) :: "memory");
    const char* const px = (const char*)(((unsigned long long)xw[1] << 32) | xw[0]) + (size_t)rb * 128 * K * 2;
    const char* const pwm = (const char*)(((unsigned long long)xw[3] << 32) | xw[2]) + cb * 256;
    const __amdgpu_buffer_rsrc_t sx = RSRC(px, 128 * K * 2), sw = RSRC(pwm, K * N * 2 - cb * 256);
    const __amdgpu_buffer_rsrc_t s0 = RSRC(px, 0);
    auto issue = [&](int s, int kt) {
      const unsigned mx = ldsw + ST(s) * TB, mw = mx + WREG, sox = kt * 128, sow = kt * (64 * N * 2);
#pragma unroll
      for (int i = 0; i < 4; ++i) {
        if (SKIPD && i == 4 - SKIPD && s >= 4) asm volatile("s_mov_b64 exec, 0");
        DMA(mw + i * 1024, vw[i], (NODMA & 1) ? s0 : sw, sow, WBITS); DMA(mx + (i ^ xo) * 1024, vx[i], (NODMA & 2) ? s0 : sx, sox, XBITS); }
      if (SKIPD) asm volatile("s_mov_b64 exec, -1");
    };
    auto issue_t = [&](int s) {
      const int kt = __builtin_amdgcn_readlane(ordv, s);
      const unsigned mx = ldst + ST(s) * TB, mw = ldsw + ST(s) * TB + WREG, sox = kt * 128, sow = kt * (64 * N * 2);
#pragma unroll
      for (int i = 0; i < 4; ++i) { DMA(mw + i * 1024, vw[i], sw, sow, TBITS); if (i & 1) DMA(mx + (i >> 1) * 1024, vt[i >> 1], sx, sox, XBITS); }
    };
    static_assert(NS == 4 && NT >= 4, "waits written for NS = 4");
    TS(tb);
    issue(0, ord0 & 63);
    TS(tc);
    issue(1, (ord0 >> 8) & 63);
    TS(td);
    issue(2, (ord0 >> 16) & 63);
    TS(te);
    asm volatile("s_waitcnt vmcnt(24)" : "+v"(ordv) :: "memory");
    const char* const tw = (const char*)(((unsigned long long)xw[3] << 32) | xw[2]) + ((((1 - (g >> 1)) & 3) + (j >> 4) * 4) * 4 + 1) * 256 + ((j & 15) * 4 + lw) * (N * 2);
    unsigned to = (kh * NOWN + TD + (g >> 1 & 1)) * (64 * N * 2);
    u32x2 sink = {0u, 0u};
    for (int s = 0; s <= NF - 4; ++s) { wait_vm<16>(); bar(); LST(s); issue(s + 3, __builtin_amdgcn_readlane(ordv, s + 3));
      if (TD && s < NOWN - TD) { TOUCH(sink, tw, to); to += (s & 1) ? 2 * 64 * N * 2 - 128 : 128; } }
    wait_vm<16>(); bar(); LST(NF - 3); issue_t(NF);
    wait_vm<14>(); bar(); LST(NF - 2); issue_t(NF + 1);
    for (int s = NF - 1; s <= NK - 4; ++s) { wait_vm<12>(); bar(); LST(s); issue_t(s + 3); }
    wait_vm<12>(); bar(); LST(NK - 3);
    wait_vm<6>(); bar(); LST(NK - 2);
    wait_vm<0>(); bar(); LST(NK - 1);
    asm volatile("s_waitcnt lgkmcnt(0)" : "+s"(sink) :: "memory");
    char* const myws = P.myws;
    char* const myout = P.myout;
    unsigned* const tmo = (unsigned*)(myws + WS_TMO);
    while (lds_word(RED + 4) != 4) __builtin_amdgcn_s_sleep(3);
    call = lds_word(RED + 20);
    const unsigned long long t0 = __builtin_amdgcn_s_memrealtime();
    const unsigned o16 = lane * 16;
    static_assert(GRAN_SET == 1024, "one 16-B load per lane");
    const char* const gs = myws + WS_GRAN + (size_t)((call & 1) * NREP + (b & (NREP - 1))) * GRAN_SET;
    char* const src = myws + WS_IN + (size_t)((call & 1) * NWG + b) * SLOT + lw * STRM;
    bool dead = false, wantg = lw == 0;
    auto maxima = [&](u32x4 a) {
      if (__all(a[1] == call && a[3] == call)) { red[4] = wave_umax(max(a[0], a[2])) | 0x80000000u; wantg = false; STK(12); } };
    const __amdgpu_buffer_rsrc_t rg = RSRC(gs, lw == 0 ? GRAN_SET : 0);
    unsigned mk;
    for (;;) {
      u32x4 a;
      asm volatile("buffer_load_dwordx4 %0, %2, %4, 0 offen sc0 sc1\n\tbuffer_load_dword %1, %3, %5, 0 offen sc0 sc1\n\ts_waitcnt vmcnt(0)"
                   : "=&v"(a), "=&v"(mk) : "v"(o16), "v"(lane * 4), "s"(rg), "s"(RSRC(src, 256)) : "memory");
      if (wantg) maxima(a);
      if (__all(mk != 0)) break;
      if (expired(tmo, call, t0)) { dead = true; break; }
      if (!wantg) __builtin_amdgcn_s_sleep(PRESL);
    }
    const unsigned cnt = __popc(mk >> (32 - EV)) | __popc(mk << EV) << 16, incl = wave_scan(cnt), tot = __builtin_amdgcn_readlane((int)incl, 63);
    const unsigned pos = incl - cnt, len = (HDR + 2 * (tot & 0xffff) + 15) & ~15u, lenb = ((tot >> 16) + 15) & ~15u;
    const __amdgpu_buffer_rsrc_t ra = RSRC(src, len), rb_ = RSRC(src + STRA, lenb);
    u32x4 a, v[3], w[2];
    unsigned gmax = lds_word(RED + 16);
#define LOADS "buffer_load_dwordx4 %0, %6, %7, 0 offen sc0 sc1\n\tbuffer_load_dwordx4 %1, %6, %8, 0 offen sc0 sc1\n\tbuffer_load_dwordx4 %2, %6, %8, 0 offen offset:1024 sc0 sc1\n\t" \
              "buffer_load_dwordx4 %3, %6, %8, 0 offen offset:2048 sc0 sc1\n\tbuffer_load_dwordx4 %4, %6, %9, 0 offen sc0 sc1\n\t.if %10\n\tbuffer_load_dwordx4 %5, %6, %9, 0 offen offset:1024 sc0 sc1\n\t.endif"
#define LOADA : "=&v"(a), "=&v"(v[0]), "=&v"(v[1]), "=&v"(v[2]), "=&v"(w[0]), "=&v"(w[1]) : "v"(o16), "s"(rg), "s"(ra), "s"(RSRC(src + STRA, gmax ? lenb : 0)), "i"(EV < 16) : "memory"
    asm volatile(LOADS LOADA);
    unsigned ca[EV], ba[8], bs[8]; int nt[EV];
    ca[0] = 2 * (pos & 0xffff) + (unsigned)(size_t)smem + INA + lw * STRL + HDR;
#pragma unroll
    for (int i = 0; i < EV; ++i) {
      nt[i] = __builtin_amdgcn_sbfe((int)mk, 31 - i, 1);
      if (i + 1 < EV) asm("v_mad_i32_i24 %0, %1, -2, %2" : "=v"(ca[i + 1]) : "v"(nt[i]), "v"(ca[i]));
      asm volatile("" : "+v"(ca[i]), "+v"(nt[i]));
    }
#pragma unroll
    for (int c = EV / 4; c < 8; ++c) {
      const unsigned o = (pos >> 16) + (c > EV / 4 ? __popc((mk << EV) >> (32 + EV - 4 * c)) : 0);
      const unsigned p = (((mk >> (28 - 4 * c)) & 15) * 0x08040201u >> 3) & 0x01010101u;
      ba[c] = (unsigned)(size_t)smem + INB + lw * 4096 + (o & ~3u);
      bs[c] = __builtin_amdgcn_bitop3_b32(p * 255u, p * 0x01010100u + (o & 3) * 0x01010101u, 0x0c0c0c0cu, 0xca);
      asm volatile("" : "+v"(ba[c]), "+v"(bs[c]));
    }
    char* mine = myout + (size_t)((b >> 2) * 64 + (lw & 1) * 32 + (lane & 15)) * NCOL + (b & 3) * 128 + (lw >> 1) * 64 + (lane >> 4) * 16;
    asm volatile("s_waitcnt vmcnt(0)" : "+v"(a), "+v"(v[0]), "+v"(v[1]), "+v"(v[2]), "+v"(w[0]), "+v"(w[1]), "+v"(mine) :: "memory");
    auto second = [&]() {
      *(LAS u32x4*)(smem + INB + lw * 4096 + o16) = w[0];
      bool ok = (int)(w[0][0] & w[0][1] & w[0][2] & w[0][3]) < 0 || o16 >= lenb;
      if (EV < 16) { *(LAS u32x4*)(smem + INB + lw * 4096 + 1024 + o16) = w[1]; ok &= (int)(w[1][0] & w[1][1] & w[1][2] & w[1][3]) < 0 || 1024 + o16 >= lenb; }
      return (bool)__all(ok); };
    while (!dead) {
      if (wantg) maxima(a);
      bool ok = true;
#pragma unroll
      for (int c = 0; c < 3; ++c) ok &= (min(min(v[c][0], v[c][1]), min(v[c][2], v[c][3])) != 0) | (c * 1024 + o16 >= len);
      if (__all(ok)) break;
      dead = expired(tmo, call, t0);
      if (!wantg) __builtin_amdgcn_s_sleep(PRESL);
      gmax = lds_word(RED + 16);
      asm volatile(LOADS "\n\ts_waitcnt vmcnt(0)" LOADA);
    }
#pragma unroll
    for (int c = 0; c < 3; ++c) *(LAS u32x4*)(smem + INA + lw * STRL + c * 1024 + o16) = v[c];
    bool done = second();
    const bool late = gmax != 0;
    const u32x4 z = {0u, 0u, 0u, 0u};
    STK(11);
    asm volatile("buffer_load_dwordx4 %0, %3, %4, 0 offen sc0 sc1\n\tbuffer_load_dwordx4 %1, %3, %5, 0 offen sc0 sc1\n\t.if %6\n\tbuffer_load_dwordx4 %2, %3, %5, 0 offen offset:1024 sc0 sc1\n\t.endif"
                 : "=&v"(a), "=&v"(w[0]), "=&v"(w[1]) : "v"(o16), "s"(rg), "s"(RSRC(src + STRA, late && !done ? lenb : 0)), "i"(EV < 16) : "memory");
    STORE3(z, z, z, RSRC(src, late ? 0 : len));
    const float wmf = *(LAS float*)(smem + INA + lw * STRL + 256);
    f32x2 f[EV / 2];
#pragma unroll
    for (int i = 0; i < EV; ++i) f[i >> 1][i & 1] = __uint_as_float(__builtin_amdgcn_bitop3_b32(*(LAS unsigned short*)(size_t)ca[i], nt[i], 0x4b000000u, 0xea));
#pragma unroll
    for (int i = 0; i < EV / 2; ++i) { f[i] -= 8388609.0f; asm volatile("" : "+v"(f[i])); }
    if (wantg) {
      asm volatile("s_waitcnt vmcnt(3)" : "+v"(a) :: "memory");
      maxima(a);
      while (wantg) {
        if (dead || expired(tmo, call, t0)) { red[4] = 0xffffffffu; break; }
        u32x4 a2;
        asm volatile("buffer_load_dwordx4 %0, %1, %2, 0 offen sc0 sc1\n\ts_waitcnt vmcnt(0)" : "=&v"(a2) : "v"(o16), "s"(rg) : "memory");
        maxima(a2);
      }
    }
    while (!gmax && !(gmax = lds_word(RED + 16))) __builtin_amdgcn_s_sleep(1);
    dead |= gmax == 0xffffffffu;
    STK(7);
    unsigned q[8];
    auto row = [&](int m) {
      u32x4 r;
#pragma unroll
      for (int c = 0; c < 4; ++c) r[c] = dead ? 0x80808080u : (lane & 16) ? q[4 * m + (c ^ 1)] : q[4 * m + c];
      st16_sys(mine + m * 16 * NCOL, r); };
    { const float kq = wmf / fmaxf(__uint_as_float(gmax & 0x7fffffffu), 0x1p-110f) * (127.0f / 65534.0f);
#pragma unroll
      for (int i = 0; i < EV / 2; ++i) f[i] = __builtin_elementwise_fma(f[i], (f32x2){kq, kq}, (f32x2){12582912.0f, 12582912.0f});
#pragma unroll
      for (int c = 0; c < EV / 4; ++c) { q[c] = pack4(f[2 * c], f[2 * c + 1]); asm volatile("" : "+v"(q[c])); } }
    STK(8);
    asm volatile("s_waitcnt vmcnt(0)" : "+v"(a), "+v"(w[0]), "+v"(w[1]) :: "memory");
    if (late && !done) done = second();
    if (EV >= 16 && !late) row(0);
    int npoll = 0;
    while (!dead && !done) {
      asm volatile("buffer_load_dwordx4 %0, %2, %3, 0 offen sc0 sc1\n\t.if %4\n\tbuffer_load_dwordx4 %1, %2, %3, 0 offen offset:1024 sc0 sc1\n\t.endif\n\ts_waitcnt vmcnt(0)"
                   : "=&v"(w[0]), "=&v"(w[1]) : "v"(o16), "s"(rb_), "i"(EV < 16) : "memory");
      ++npoll;
      done = second();
      if (!done) dead = expired(tmo, call, t0);
    }
    asm volatile("" ::: "memory");
#pragma unroll
    for (int c = EV / 4; c < 8; ++c) { const unsigned lo = ((LAS unsigned*)(size_t)ba[c])[0], hi = ((LAS unsigned*)(size_t)ba[c])[1];
      q[c] = __builtin_amdgcn_perm(hi, lo, bs[c]) & 0x7f7f7f7fu; }
    if (EV < 16 || late) row(0);
    row(1);
    asm volatile("buffer_store_dwordx4 %0, %1, %2, 0 offen sc0 sc1\n\t.if %3\n\tbuffer_store_dwordx4 %0, %1, %2, 0 offen offset:1024 sc0 sc1\n\t.endif\n\ts_nop 1" :: "v"(z), "v"(o16), "s"(rb_), "i"(EV < 16) : "memory");
    if (late) STORE3(z, z, z, ra);
    STK(9);
    STW(7, 256); STW(8, 256); STW(9, 256); STW(11, 320); STW(12, 256);
    TSW(11, ta); TSW(12, tb); TSW(13, tc); TSW(14, td); TSW(15, te);
  } else {
    const int rank = P.rank;
    char* const myws = P.myws;
    unsigned* const tmo = (unsigned*)(myws + WS_TMO);
    STAMP(0);
    unsigned cnt0;
    asm volatile("global_load_dword %0, %1, off sc0 sc1" : "=v"(cnt0) : "v"((unsigned*)(myws + WS_CNT) + b) : "memory");
    if (tid < 8) red[tid] = 0;
    char* pw = P.ws[0];
#pragma unroll
    for (int i = 1; i < NR; ++i) { char* q = P.ws[i]; asm volatile("" : "+s"(q)); pw = (lane >> 3) == i ? q : pw; }
    pw += WS_GRAN + (size_t)(lane & 7) * GRAN_SET + (size_t)(rank * NGRP + (b & (NGRP - 1))) * 8;
    char* dslot = P.ws[((unsigned)cb >> 2) & (NR - 1)] + WS_IN + (size_t)(rank * 32 + (rb * 2 + kh) * 4 + (cb & 3)) * SLOT + wave * STRM;
    asm volatile("" : "+v"(pw));
    asm volatile("" : "+s"(dslot));
    const int wm = wave & 1, wn = wave >> 1;
    const int fast = ((cb & 3) ^ 1) * (cb ^ 31);
    unsigned ao[2][2], bo[4][2];
    { const int r = lane & 15, q = lane >> 4;
#pragma unroll
      for (int h = 0; h < 2; ++h)
#pragma unroll
        for (int ks = 0; ks < 2; ++ks) ao[h][ks] = ((h ^ kh) * 64 + wm * 32 + r) * 128 + (((4 * ks + q) ^ ((r >> 1) & 7)) << 4);
      const int qq = (lane & 15) >> 2, p = lane & 3;
#pragma unroll
      for (int n = 0; n < 4; ++n)
#pragma unroll
        for (int t = 0; t < 2; ++t) { const int row = 8 * q + 4 * t + qq, ch = 8 * wn + 2 * p + (n >> 1);
          bo[n][t] = WREG + 256 * row + 16 * (ch ^ (row & 11)) + 8 * ((n ^ p) & 1); }
    }
    f32x4 acc[4][4];
#pragma unroll
    for (int m = 0; m < 4; ++m)
#pragma unroll
      for (int n = 0; n < 4; ++n) acc[m][n] = (f32x4){0.f, 0.f, 0.f, 0.f};
    Frags f0, f1;
    auto rd = [&](Frags& f, int r, int st, int ks, bool whole) {
      constexpr int RW[12] = {0, 1, 8, 9, 2, 3, 10, 4, 5, 11, 6, 7}, RH[12] = {0, 1, 8, 9, 2, 3, 4, 5, 6, 7, -1, -1};
      const int c = r < 0 || r > 11 ? -1 : whole ? RW[r] : RH[r];
      if (c >= 8) f.a[c - 8] = *(LAS bf16x8*)(smem + ao[(c - 8) >> 1][ks] + (c & 1) * 2048 + st * TB);
      else if (c >= 0) f.b[c >> 1][c & 1] = __builtin_amdgcn_ds_read_tr16_b64_v4i16((LAS bf16x4*)(smem + bo[c >> 1][c & 1] + st * TB + ks * 8192));
    };
#define BF(f, n) __builtin_shufflevector(f.b[n][0], f.b[n][1], 0, 1, 2, 3, 4, 5, 6, 7)
    auto half = [&](Frags& fc, Frags& fn, const int st, const int ks, const bool CF, const int RD, auto&& extra) {
      constexpr int OM[16] = {0, 1, 0, 1, 2, 2, 0, 1, 2, 3, 3, 3, 0, 1, 2, 3};
      constexpr int ON[16] = {0, 0, 1, 1, 0, 1, 2, 2, 2, 0, 1, 2, 3, 3, 3, 3};
      if (CF) {
        const int r0 = ks ? RS : BP + 1;
#pragma unroll
        for (int i = 0; i < 16; ++i) {
          MFMA(acc[OM[i]][ON[i]], BF(fc, ON[i]), fc.a[OM[i]]);
          SB0();
          if (!ks && i == BP) {
            wait_lgkm0(); bar(); }
          rd(fn, i - r0, st, ks, RD == 2);
          if (EW && i == 15) { if (ks) __builtin_amdgcn_s_waitcnt(0xC07F | 5 << 8); else __builtin_amdgcn_s_waitcnt(0xC07F | EW << 8); }
          SB0();
        }
      } else {
        if (!ks && RD) { wait_lgkm0(); bar(); SB0(); }
#pragma unroll
        for (int i = 0; i < 8; ++i) {
          if (RD) { rd(fn, 2 * i, st, ks, false); rd(fn, 2 * i + 1, st, ks, false); }
          MFMA(acc[i & 1][i >> 1], BF(fc, i >> 1), fc.a[i & 1]);
          extra(i);
          SB0();
        }
      }
    };
    auto none = [](int) {};
    wait_lgkm0(); bar();
#pragma unroll
    for (int i = 0; i < 12; ++i) rd(f0, i, ST(0), 0, true);
    STAMP(1);
    SB0();
    half(f0, f1, ST(0), 1, true, 2, none);
    constexpr int NL = (NF - 1) / NS * NS;
    for (int s = 0; s < NL; s += NS) {
#pragma unroll
      for (int u = 1; u <= NS; ++u) { half(f1, f0, ST(u), 0, true, 2, none); half(f0, f1, ST(u), 1, true, 2, none); }
    }
#pragma unroll
    for (int s = NL + 1; s < NF; ++s) { half(f1, f0, ST(s), 0, true, 2, none); half(f0, f1, ST(s), 1, true, 2, none); }
    half(f1, f0, ST(NF), 0, true, 1, none);
    asm volatile("s_waitcnt vmcnt(0)\n\tv_readfirstlane_b32 %0, %1" : "=s"(call) : "v"(cnt0) : "memory");
    call += 1;
    const unsigned tag = 1u + (call & 1u);
    { char* dst = myws + WS_XCH + (size_t)(b ^ 1) * 32768 + wave * 8192 + lane * 16;
      half(f0, f1, ST(NF), 1, false, 1, [&](int i) {
        u32x4 v;
#pragma unroll
        for (int r = 0; r < 4; ++r) v[r] = (__float_as_uint(acc[2 + (i >> 2)][i & 3][r]) & ~3u) | tag;
        asm volatile("global_store_dwordx4 %0, %1, off offset:%2 " SB "\n\ts_nop 1" :: "v"(dst + 4096), "v"(v), "i"(i * 1024 - 4096) : "memory"); }); }
    if (tid == 0) {
      unsigned* cnt = (unsigned*)(myws + WS_CNT) + b;
      asm volatile("global_store_dword %0, %1, off sc0 sc1" :: "v"(cnt), "v"(call) : "memory");
      if (b < NGRP) {
        char* nx = myws + WS_GRP + (size_t)(((call + 1) & 1) * NGRP + b) * 128 + 4;
        asm volatile("global_store_dword %0, %1, off sc0 sc1" :: "v"(nx), "v"(0u) : "memory"); }
    }
    const char* src = myws + WS_XCH + (size_t)b * 32768 + wave * 8192 + lane * 16 + 4096;
    u32x4 v[8];
#define XLOAD "global_load_dwordx4 %0, %8, off offset:-4096 " XB "\n\tglobal_load_dwordx4 %1, %8, off offset:-3072 " XB "\n\t" \
              "global_load_dwordx4 %2, %8, off offset:-2048 " XB "\n\tglobal_load_dwordx4 %3, %8, off offset:-1024 " XB "\n\t" \
              "global_load_dwordx4 %4, %8, off " XB "\n\tglobal_load_dwordx4 %5, %8, off offset:1024 " XB "\n\t" \
              "global_load_dwordx4 %6, %8, off offset:2048 " XB "\n\tglobal_load_dwordx4 %7, %8, off offset:3072 " XB
    const char* const lastw = src + (63 - lane) * 16;
    unsigned probe, asked = 0, tmp;
#define PROBE "global_load_dword %9, %12, off offset:3084 " PB
#define PROBE2 "global_load_dword %0, %1, off offset:3084 " PB
#define XTRY "s_cmp_lg_u32 %10, 0\n\ts_cbranch_scc1 1f\n\ts_getreg_b32 %11, hwreg(7, 0, 4)\n\ts_cmp_lg_u32 %11, 0\n\ts_cbranch_scc1 1f\n\t" \
             "v_readfirstlane_b32 %11, %9\n\ts_and_b32 %10, %11, %13\n\ts_cbranch_scc1 2f\n\t" PROBE "\n\ts_branch 1f\n2:\n\t" XLOAD "\n1:"
#define XARG(m) : m(v[0]), m(v[1]), m(v[2]), m(v[3]), m(v[4]), m(v[5]), m(v[6]), m(v[7]), "+v"(src), m(probe), "+s"(asked), "=&s"(tmp) : "v"(lastw), "s"(tag) : "memory", "scc"
    auto xtry = [&](int h, int i) {
      if (i == 0 && h == XL + XD) asm volatile(PROBE XARG("=&v"));
      if ((i & XI) == 0 && h <= XL) asm volatile(XTRY XARG("+v")); };
#pragma unroll
    for (int s = NF + 1; s < NK; ++s) {
      half(f1, f0, ST(s), 0, false, 1, [&](int i) { xtry(2 * (NK - s), i); });
      half(f0, f1, ST(s), 1, false, 1, [&](int i) { xtry(2 * (NK - s) - 1, i); }); }
    half(f1, f0, 0, 0, false, 0, [&](int i) { xtry(0, i); });
    STK(2);
    bool dead = false;
    { const unsigned long long t0 = __builtin_amdgcn_s_memrealtime();
      asm volatile("s_waitcnt vmcnt(0)" XARG("+v"));
      for (unsigned all = asked;; all = tag) {
#pragma unroll
        for (int i = 0; i < 8; ++i) all &= (v[i][0] & v[i][1]) & (v[i][2] & v[i][3]);
        if (__all(all != 0)) break;
        while (!(dead = expired(tmo, call, t0)) && !(__builtin_amdgcn_readfirstlane(probe) & tag))
          asm volatile(PROBE2 "\n\ts_waitcnt vmcnt(0)" : "=&v"(probe) : "v"(lastw) : "memory");
        if (dead) break;
        asm volatile(XLOAD "\n\ts_waitcnt vmcnt(0)" : "=&v"(v[0]), "=&v"(v[1]), "=&v"(v[2]), "=&v"(v[3]), "=&v"(v[4]), "=&v"(v[5]), "=&v"(v[6]), "=&v"(v[7]) : "v"(src) : "memory");
      }
#pragma unroll
      for (int i = 0; i < 8; ++i)
#pragma unroll
        for (int r = 0; r < 4; ++r) acc[i >> 2][i & 3][r] += __uint_as_float(v[i][r] & ~3u);
    }
    STK(3);
    float mx = 0.f;
#pragma unroll
    for (int i = 0; i < 8; ++i) mx = fmaxf(fmaxf(mx, fmaxf(acc[i >> 2][i & 3][0], acc[i >> 2][i & 3][1])), fmaxf(acc[i >> 2][i & 3][2], acc[i >> 2][i & 3][3]));
    const unsigned wmx = wave_umax(__float_as_uint(mx));
    unsigned wn_ = 0, wgm = 0;
    if (lane == 0)
      asm volatile("ds_write_b32 %2, %5 offset:20\n\tds_max_u32 %2, %3\n\tds_add_rtn_u32 %0, %2, %4 offset:4\n\tds_read_b32 %1, %2" : "=&v"(wn_), "=&v"(wgm) : "v"(RED), "v"(dead ? 0xffffffffu : wmx), "v"(1u), "v"(call) : "memory");
    unsigned mi = 0;
#pragma unroll
    for (int i = 0; i < 32; ++i) mi = __builtin_amdgcn_alignbit(mi, __float_as_uint(acc[i >> 4][i >> 2 & 3][i & 3]), 31);
    asm volatile("s_waitcnt lgkmcnt(0)" : "+v"(wn_), "+v"(wgm), "+v"(mi) :: "memory");
    const bool last = __builtin_amdgcn_readfirstlane(wn_) == 3;
    u32x2 arr;
    if (last && lane == 0) {
      char* const grp = myws + WS_GRP + (size_t)((call & 1) * NGRP + (b & (NGRP - 1))) * 128;
      const u32x2 one = {1u, 0u};
      asm volatile("global_atomic_umax %1, %2, off offset:4 sc1\n\tglobal_atomic_add_x2 %0, %1, %3, off sc0 sc1" : "=&v"(arr) : "v"(grp), "v"(wgm), "v"(one) : "memory");
    }
    if (mi == 0xffffffffu) { mi = 0xfffffffeu; acc[1][3][3] = 0.f; }
    const unsigned cnt = __popc(~mi >> (32 - EV)) | __popc(~mi << EV) << 16, incl = wave_scan(cnt), tot = __builtin_amdgcn_readlane((int)incl, 63);
    const unsigned len = HDR + 2 * (tot & 0xffff), pos = incl - cnt;
    const float wmf = fmaxf(__uint_as_float(wmx), 0x1p-110f), ks = 65534.0f / wmf;
    const unsigned oa = OUTA + wave * STRL, ob = OUTB + wave * 8192, o16 = lane * 16;
    *(LAS unsigned*)(smem + oa + lane * 4) = ~mi;
    *(LAS float*)(smem + oa + 256) = wmf;
    *(LAS unsigned short*)(smem + oa + len + lane * 2) = 0xffff;
    { unsigned a = (unsigned)(size_t)smem + oa + HDR + 2 * (pos & 0xffff);
      const unsigned sink = (unsigned)(size_t)smem + oa + 2960;
#pragma unroll
      for (int i = 0; i < EV; i += 2) {
        const f32x2 y = {acc[i >> 4][i >> 2 & 3][i & 3], acc[i >> 4][i >> 2 & 3][(i & 3) + 1]};
        const f32x2 f = __builtin_elementwise_fma(y, (f32x2){ks, ks}, (f32x2){8388609.0f, 8388609.0f});
#pragma unroll
        for (int h = 0; h < 2; ++h) {
          const unsigned t = (unsigned)min(max((int)__float_as_uint(y[h]), -2), 0);
          ((LAS unsigned short*)(size_t)__builtin_amdgcn_bitop3_b32(t, sink, a, 0xca))[i + h] = (unsigned short)__float_as_uint(f[h]);
          a += t;
        }
      } }
    asm volatile("" ::: "memory");
    char* const dst = dslot + (size_t)(call & 1) * NWG * SLOT;
    f32x4 yb[4];
    { u32x4 v3[3];
#pragma unroll
      for (int c = 0; c < 3; ++c) v3[c] = *(LAS u32x4*)(smem + oa + c * 1024 + o16);
      if (last) {
        asm volatile("s_waitcnt vmcnt(0)" : "+v"(arr) :: "memory");
        const unsigned n = __builtin_amdgcn_readfirstlane(arr[0]), gm = __builtin_amdgcn_readfirstlane(arr[1]);
        if ((n & (GSZ - 1)) == GSZ - 1) {
          const u32x2 gr = {gm, call};
          asm volatile("global_store_dwordx2 %0, %1, off sc0 sc1\n\ts_nop 1" :: "v"(pw + (size_t)(call & 1) * NREP * GRAN_SET), "v"(gr) : "memory");
        }
      }
      const __amdgpu_buffer_rsrc_t rd = RSRC(dst, (len + 15) & ~15u);
      if (o16 < PACEB) asm volatile("buffer_store_dwordx4 %0, %1, %2, 0 offen sc0 sc1\n\ts_nop 1" :: "v"(v3[0]), "v"(o16), "s"(rd) : "memory");
      *(LAS float*)(smem + ob + ((tot >> 16) + lane) * 4) = 0.f;
      { unsigned a = (unsigned)(size_t)smem + ob + 4 * (pos >> 16);
        const unsigned sink = (unsigned)(size_t)smem + ob + 8192 - 128;
#pragma unroll
        for (int i = EV; i < 32; ++i) {
          const float y = acc[i >> 4][i >> 2 & 3][i & 3];
          const unsigned t = (unsigned)min(max((int)__float_as_uint(y), -4), 0);
          ((LAS float*)(size_t)__builtin_amdgcn_bitop3_b32(t, sink, a, 0xca))[i - EV] = y;
          a += t;
        } }
      asm volatile("" ::: "memory");
#pragma unroll
      for (int c = 0; c < 4; ++c) yb[c] = *(LAS f32x4*)(smem + ob + lane * 64 + c * 16);
      const unsigned len16 = (len + 15) & ~15u;
      for (unsigned lo = PACEB; lo < len16; ) {
        unsigned known;
        while (!(known = lds_word(RED + 16)) && __builtin_amdgcn_s_getreg(7 | 3 << 11)) { }
        const unsigned hi = known ? len16 : min(lo + PACEC, len16);
        const __amdgpu_buffer_rsrc_t rs = RSRC(dst, hi);
        if (lo < 1024 && o16 >= lo) asm volatile("buffer_store_dwordx4 %0, %1, %2, 0 offen sc0 sc1\n\ts_nop 1" :: "v"(v3[0]), "v"(o16), "s"(rs) : "memory");
        if (hi > 1024 && lo < 2048 && o16 + 1024 >= lo) asm volatile("buffer_store_dwordx4 %0, %1, %2, 0 offen offset:1024 sc0 sc1\n\ts_nop 1" :: "v"(v3[1]), "v"(o16), "s"(rs) : "memory");
        if (hi > 2048 && o16 + 2048 >= lo) asm volatile("buffer_store_dwordx4 %0, %1, %2, 0 offen offset:2048 sc0 sc1\n\ts_nop 1" :: "v"(v3[2]), "v"(o16), "s"(rs) : "memory");
        lo = hi;
      }
    }
    asm volatile("" : "+v"(yb[0]), "+v"(yb[1]), "+v"(yb[2]), "+v"(yb[3]));
    STK(4);
    unsigned gmax;
    while (!(gmax = lds_word(RED + 16))) { }
    STK(5);
    if (gmax != 0xffffffffu) {
      const float inv = 127.0f / fmaxf(__uint_as_float(gmax & 0x7fffffffu), 0x1p-110f);
      const __amdgpu_buffer_rsrc_t rd = RSRC(dst + STRA, ((tot >> 16) + 15) & ~15u);
      auto quant = [&]() {
        u32x4 q;
#pragma unroll
        for (int c = 0; c < 4; ++c)
          q[c] = pack4(__builtin_elementwise_fma((f32x2){yb[c][0], yb[c][1]}, (f32x2){inv, inv}, (f32x2){12583040.0f, 12583040.0f}),
                       __builtin_elementwise_fma((f32x2){yb[c][2], yb[c][3]}, (f32x2){inv, inv}, (f32x2){12583040.0f, 12583040.0f}));
        return q; };
      asm volatile("buffer_store_dwordx4 %0, %1, %2, 0 offen sc0 sc1\n\ts_nop 1" :: "v"(quant()), "v"(o16), "s"(rd) : "memory");
      if (EV < 16 && (tot >> 16) > 1024) {
#pragma unroll
        for (int c = 0; c < 4; ++c) yb[c] = *(LAS f32x4*)(smem + ob + 4096 + lane * 64 + c * 16);
        asm volatile("buffer_store_dwordx4 %0, %1, %2, 0 offen offset:1024 sc0 sc1\n\ts_nop 1" :: "v"(quant()), "v"(o16), "s"(rd) : "memory");
      }
    }
    STK(6);
    STW(2, 0); STW(3, 0); STW(4, 0); STW(5, 0); STW(6, 0);
  }
}
}

extern "C" void kernel_launch(int rank, int nranks, ncclComm_t comm, hipStream_t stream,
                              void* const* d_in, const int* in_sizes, int n_in,
                              void* d_out, int out_size, const int* in_dtypes, int out_dtype,
                              void* const* d_out_all, void* const* d_ws_all, size_t ws_bytes) {
  (void)comm; (void)in_dtypes; (void)out_dtype; (void)d_out;
  if (nranks != NR || n_in < 2 || in_sizes[0] != MS * K || in_sizes[1] != K * N || out_size != NR * MS * NCOL || ws_bytes < WS_NEED || (unsigned)rank >= (unsigned)NR) {
    fprintf(stderr, "kernel_launch: unexpected problem shape\n");
    return;
  }
  HIP_CHECK(hipFuncSetAttribute((const void*)fused_kernel, hipFuncAttributeMaxDynamicSharedMemorySize, LDS_BYTES));
  Params P;
  P.x = (const bf16_t*)d_in[0]; P.w = (const bf16_t*)d_in[1];
  for (int p = 0; p < NR; ++p) { P.out[p] = (char*)d_out_all[p]; P.ws[p] = (char*)d_ws_all[p]; }
  P.myws = P.ws[rank]; P.myout = P.out[rank];
  P.rank = rank; P.pad = 0;
  int A0 = 30, AS = 3, A1 = 80, RO = 2;
  int DROP = 1, D0 = 0, D1 = 0, DT = 8;
  for (int c = 0; c < 32; ++c) {
    const int l0 = c & 7, l1 = (l0 + 1) & 7, kh = c >> 3 & 1;
    int n = 0, key[NOWN + 8], u = -4;
    for (int t = 0; t < NOWN + 8; ++t) {
      const int v = (u + RO * (u >> 3)) & 7;
      const bool lead = t < 4 ? t == (l0 >> 1) : v == l0 || (v == l1 && !(DROP && TD && c >= 16 && t >= TD && t < NOWN));
      const int a = A0 + (t - 4) * AS < A1 ? A0 + (t - 4) * AS : A1;
      key[t] = lead ? t * 10 - (t < 4 ? 100 : a) - 1 : t * 10;
      if (lead && t >= DT && t < NOWN) key[t] -= v == l0 ? D0 : D1;
      if (lead && key[t] > NOWN * 10 - 11) key[t] = NOWN * 10 - 11;
      if (t >= NOWN && v == (kh ? l0 : l1)) key[t] = -1000;
      ++u;
    }
    for (int k = -300; k < (NOWN + 8) * 10; ++k)
      for (int t = 0; t < NOWN + 8; ++t) if (key[t] == k) P.ord[c][n++] = t < NOWN ? kh * NOWN + t : NOWN + t;
    while (n < 64) P.ord[c][n++] = 0;
  }
  hipLaunchKernelGGL(fused_kernel, dim3(NWG), dim3(NTHR), LDS_BYTES, stream, P);
  HIP_CHECK(hipGetLastError());
}
```

```cpp
#include <hip/hip_runtime.h>
#include <rccl/rccl.h>
#include <cstdio>
#include <cstdlib>
#include <cstddef>

#define HIP_CHECK(expr) do { hipError_t _e = (expr); if (_e != hipSuccess) { \
    fprintf(stderr, "HIP error %s at %s:%d\n", hipGetErrorString(_e), __FILE__, __LINE__); \
    return; } } while (0)

namespace {
typedef unsigned short bf16_t;
typedef short bf16x8 __attribute__((ext_vector_type(8)));
typedef short bf16x4 __attribute__((ext_vector_type(4)));
typedef float f32x4 __attribute__((ext_vector_type(4)));
typedef float f32x2 __attribute__((ext_vector_type(2)));
typedef unsigned u32x4 __attribute__((ext_vector_type(4)));
typedef unsigned u32x2 __attribute__((ext_vector_type(2)));
typedef unsigned u32x16 __attribute__((ext_vector_type(16)));
#define LAS __attribute__((address_space(3)))

constexpr int NR = 8;
constexpr int MS = 512, N = 4096, K = 4096;
constexpr int NCOL = N / NR;
#define NODMA 0
#define SKIPD 0
#define PRESL 2
#define AOFF 3072
#define PITCH 4096
#define KHX 0
#define PK 4
#define PB "sc0 sc1"
#define XB "sc0 sc1"
#define SB "sc0 sc1"
#define B_NT "sc0 sc1 nt"
#define B_P ""
#define B_2 "sc0 sc1\n\ts_mov_b64 exec, 0xffff\n\tglobal_store_dwordx4 %0, %1, %2 offset:1024 sc0 sc1"
#define B_2P "\n\ts_mov_b64 exec, 0xffff\n\tglobal_store_dwordx4 %0, %1, %2 offset:1024"
#define B_S1 "sc1"
#define B_S1NT "sc1 nt"
#define XD 2
#define XI 7
#define XN 2
#define XJ 0
#define XS 1
#define XO 0
#define TD 8
constexpr int NWG = 256, NTHR = 512;
#define PACEC 512
#define PACEB 512
#define XL 6
constexpr int TB = 16384, NS = 4;
#define NX 1
constexpr int NOWN = 28, NF = NOWN + NX, NT = 64 - 2 * NF, NK = NF + NT;
constexpr int ST(int s) { return (s + NS - NK % NS) % NS; }
constexpr int WREG = NS * TB;
constexpr int RED = 2 * NS * TB;
#define RA 64
#define RB 64
#define TP 1
#define FB 0
#define PIN 1
#define EV 16
constexpr int HDR = 260, STRA = AOFF, STRB = 64 * (32 - EV), STRM = PITCH, SLOT = 4 * STRM, STRL = 3072;
static_assert(EV % 4 == 0 && EV >= 4 && EV <= 20, "whole words");
#define BP 3
#define EW 3
#define RS 0
constexpr int LDS_BYTES = RED + 512;
#define LST(s) do { } while (0)
constexpr int OUTA = 0, OUTB = TB, INA = WREG, INB = WREG + TB;
#define NG 4
#define NRP 16
#define SP 1
#define GLC "glc"
constexpr int NGRP = NG, GSZ = NWG / NGRP, NREP = NRP, NPOLL = SP && NGRP > 4 ? NGRP / 4 : 1;
#define AB "sc1"
#define GL 128
#define SS 4
#define GP (NG < 4 ? 128 : NG * 32)
constexpr size_t WS_CNT = 0;
constexpr size_t WS_TMO = 4096;
constexpr size_t WS_GRP = 8192;
constexpr size_t WS_GRAN = 16384;
constexpr size_t GRAN_SET = (size_t)NR * NGRP * 4;
constexpr size_t WS_STAMP = 65536;
#define XPITCH 5120
constexpr int XW = XPITCH;
constexpr size_t WS_XCH = 1 << 20;
constexpr size_t WS_IN = WS_XCH + (size_t)NWG * 4 * XW;
constexpr size_t WS_NEED = WS_IN + (size_t)2 * NWG * SLOT;
constexpr unsigned long long DEADLINE = 50000000ull;

struct Params {
  const bf16_t* x; const bf16_t* w;
  char* myws; char* myout;
  int rank; int pad;
  char* out[NR]; char* ws[NR];
  unsigned char ord[32][64];
};

#define RSRC(p, n) __builtin_amdgcn_make_buffer_rsrc((void*)(p), (short)0, (int)(n), 0x00020000)
#define STORE3(a, b, c, rs) asm volatile("buffer_store_dwordx4 %0, %3, %4, 0 offen sc0 sc1\n\tbuffer_store_dwordx4 %1, %3, %4, 0 offen offset:1024 sc0 sc1\n\t" \
    "buffer_store_dwordx4 %2, %3, %4, 0 offen offset:2048 sc0 sc1\n\ts_nop 1" :: "v"(a), "v"(b), "v"(c), "v"(o16), "s"(rs) : "memory")
#define WBITS ""
#define XBITS ""
#define TBITS ""
#define DMA(m0v, voff, srd, soff, bits) asm volatile("s_mov_b32 m0, %0\n\ts_nop 0\n\tbuffer_load_dwordx4 %1, %2, %3 offen " bits " lds" :: "s"(m0v), "v"(voff), "s"(srd), "s"(soff) : "memory")
template <int V> __device__ __forceinline__ void wait_vm() { asm volatile("s_waitcnt vmcnt(%0)" :: "i"(V) : "memory"); }
__device__ __forceinline__ void wait_lgkm0() { __builtin_amdgcn_s_waitcnt(0xC07F); }
__device__ __forceinline__ void bar() { asm volatile("" ::: "memory"); __builtin_amdgcn_s_barrier(); asm volatile("" ::: "memory"); }
__device__ __forceinline__ unsigned lds_word(unsigned a) {
  unsigned v;
  asm volatile("ds_read_b32 %0, %1\n\ts_waitcnt lgkmcnt(0)" : "=v"(v) : "v"(a) : "memory");
  return __builtin_amdgcn_readfirstlane(v);
}
__device__ __forceinline__ void st16_sys(void* p, u32x4 v) { asm volatile("global_store_dwordx4 %0, %1, off sc0 sc1\n\ts_nop 1" :: "v"(p), "v"(v) : "memory"); }
template <int O> __device__ __forceinline__ void st16_sys_o(void* p, u32x4 v) { asm volatile("global_store_dwordx4 %0, %1, off offset:%2 sc0 sc1\n\ts_nop 1" :: "v"(p), "v"(v), "i"(O) : "memory"); }
__device__ __forceinline__ unsigned wave_umax(unsigned v) {
  v = max(v, (unsigned)__builtin_amdgcn_update_dpp(0, (int)v, 0x111, 0xf, 0xf, true));
  v = max(v, (unsigned)__builtin_amdgcn_update_dpp(0, (int)v, 0x112, 0xf, 0xf, true));
  v = max(v, (unsigned)__builtin_amdgcn_update_dpp(0, (int)v, 0x114, 0xf, 0xf, true));
  v = max(v, (unsigned)__builtin_amdgcn_update_dpp(0, (int)v, 0x118, 0xf, 0xf, true));
  v = max(v, (unsigned)__builtin_amdgcn_update_dpp(0, (int)v, 0x142, 0xa, 0xf, true));
  v = max(v, (unsigned)__builtin_amdgcn_update_dpp(0, (int)v, 0x143, 0xc, 0xf, true));
  return (unsigned)__builtin_amdgcn_readlane((int)v, 63);
}
__device__ __forceinline__ unsigned wave_scan(unsigned v) {
  v += (unsigned)__builtin_amdgcn_update_dpp(0, (int)v, 0x111, 0xf, 0xf, true);
  v += (unsigned)__builtin_amdgcn_update_dpp(0, (int)v, 0x112, 0xf, 0xf, true);
  v += (unsigned)__builtin_amdgcn_update_dpp(0, (int)v, 0x114, 0xf, 0xf, true);
  v += (unsigned)__builtin_amdgcn_update_dpp(0, (int)v, 0x118, 0xf, 0xf, true);
  v += (unsigned)__builtin_amdgcn_update_dpp(0, (int)v, 0x142, 0xa, 0xf, true);
  v += (unsigned)__builtin_amdgcn_update_dpp(0, (int)v, 0x143, 0xc, 0xf, true);
  return v;
}
__device__ __forceinline__ bool expired(unsigned* tmo, unsigned call, unsigned long long t0) {
  const unsigned long long dt = __builtin_amdgcn_s_memrealtime() - t0;
  if (dt < 5000) return false;
  if (dt > DEADLINE ||
      (unsigned)__builtin_amdgcn_readfirstlane(__hip_atomic_load(tmo, __ATOMIC_RELAXED, __HIP_MEMORY_SCOPE_AGENT)) == call) {
    __hip_atomic_store(tmo, call, __ATOMIC_RELAXED, __HIP_MEMORY_SCOPE_AGENT);
    return true;
  }
  return false;
}
#define STAMPT(i, t) do { } while (0)
#define STAMP(i) STAMPT(i, 0)
#define STK(i) do { } while (0)
#define STW(i, t) do { } while (0)
#define TS(v) do { } while (0)
#define TSW(i, v) do { } while (0)

__device__ __forceinline__ unsigned pack4(f32x2 lo, f32x2 hi) {
  return __builtin_amdgcn_perm(__builtin_amdgcn_perm(__float_as_uint(hi[1]), __float_as_uint(hi[0]), 0x0c0c0400u), __builtin_amdgcn_perm(__float_as_uint(lo[1]), __float_as_uint(lo[0]), 0x0c0c0400u), 0x05040100u);
}
struct Frags { bf16x8 a[4]; bf16x4 b[4][2]; };

#define MFMA(acc, w_, x_) acc = __builtin_amdgcn_mfma_f32_16x16x32_bf16(w_, x_, acc, 0, 0, 0)
#define SB0() __builtin_amdgcn_sched_barrier(0)

#define TOUCH(sink, base, off) asm volatile("s_load_dwordx2 %0, %1, %2 offset:60" : "+s"(sink) : "s"(base), "s"(off) : "memory")

__global__ __launch_bounds__(NTHR) void fused_kernel(Params P) {
  TS(ta);
  extern __shared__ __attribute__((aligned(16))) char smem_[];
  LAS char* smem = (LAS char*)smem_;
  const int tid = threadIdx.x, lane = tid & 63;
  const int wave = __builtin_amdgcn_readfirstlane(tid >> 6);
  const int b = blockIdx.x;
  const int g = b & 7, j = b >> 3;
  const int rb = j & 3, kh = (g & 1) ^ KHX, cb = (j >> 2) * 4 + (((g >> 1) + (j >> 2)) & 3);
  unsigned call;
  LAS unsigned* red = (LAS unsigned*)(smem + RED);

  if (__builtin_expect(wave >= 4, 1)) {
    const int lw = wave - 4;
    u32x4 xw; unsigned ord0; int ordv;
    { const unsigned oo = offsetof(Params, ord) + (((cb & 3) == 1) * 16 + kh * 8 + (((j >> 2) + 2 * rb) & 7)) * 64;
      asm volatile("s_load_dwordx4 %0, %3, 0x0\n\ts_load_dword %1, %3, %4\n\tglobal_load_ubyte %2, %5, %3"
                   : "=&s"(xw), "=&s"(ord0), "=&v"(ordv) : "s"(__builtin_amdgcn_kernarg_segment_ptr()), "s"(oo), "v"(oo + lane) : "memory"); }
    unsigned vx[4], vw[4];
    const int xo = (j >> 2 & 1) * XO;
#pragma unroll
    for (int i = 0; i < 4; ++i) { const int row = (lw * 4 + (i ^ xo)) * 8 + (lane >> 3); vx[i] = row * (K * 2) + (((lane & 7) ^ ((row >> 1) & 7)) << 4); }
#pragma unroll
    for (int i = 0; i < 4; ++i) { const int row = (lw * 4 + i) * 4 + (lane >> 4); vw[i] = row * (N * 2) + (((lane & 15) ^ (row & 11)) << 4); }
    unsigned vt[2];
#pragma unroll
    for (int i = 0; i < 2; ++i) { const int row = kh * 64 + (lw * 2 + i) * 8 + (lane >> 3); vt[i] = row * (K * 2) + (((lane & 7) ^ ((row >> 1) & 7)) << 4); }
    const unsigned ldsw = (unsigned)(size_t)smem + lw * 4096, ldst = (unsigned)(size_t)smem + kh * 8192 + lw * 2048;
    asm volatile("s_waitcnt lgkmcnt(0)" : "+s"(xw), "+s"(ord0) :: "memory");
    const char* const px = (const char*)(((unsigned long long)xw[1] << 32) | xw[0]) + (size_t)rb * 128 * K * 2;
    const char* const pwm = (const char*)(((unsigned long long)xw[3] << 32) | xw[2]) + cb * 256;
    const __amdgpu_buffer_rsrc_t sx = RSRC(px, 128 * K * 2), sw = RSRC(pwm, K * N * 2 - cb * 256);
    const __amdgpu_buffer_rsrc_t s0 = RSRC(px, 0);
    auto issue = [&](int s, int kt) {
      const unsigned mx = ldsw + ST(s) * TB, mw = mx + WREG, sox = kt * 128, sow = kt * (64 * N * 2);
#pragma unroll
      for (int i = 0; i < 4; ++i) {
        if (SKIPD && i == 4 - SKIPD && s >= 4) asm volatile("s_mov_b64 exec, 0");
        DMA(mw + i * 1024, vw[i], (NODMA & 1) ? s0 : sw, sow, WBITS); DMA(mx + (i ^ xo) * 1024, vx[i], (NODMA & 2) ? s0 : sx, sox, XBITS); }
      if (SKIPD) asm volatile("s_mov_b64 exec, -1");
    };
    auto issue_t = [&](int s) {
      const int kt = __builtin_amdgcn_readlane(ordv, s);
      const unsigned mx = ldst + ST(s) * TB, mw = ldsw + ST(s) * TB + WREG, sox = kt * 128, sow = kt * (64 * N * 2);
#pragma unroll
      for (int i = 0; i < 4; ++i) { DMA(mw + i * 1024, vw[i], sw, sow, TBITS); if (i & 1) DMA(mx + (i >> 1) * 1024, vt[i >> 1], sx, sox, XBITS); }
    };
    static_assert(NS == 4 && NT >= 4, "waits written for NS = 4");
    TS(tb);
    issue(0, ord0 & 63);
    TS(tc);
    issue(1, (ord0 >> 8) & 63);
    TS(td);
    issue(2, (ord0 >> 16) & 63);
    TS(te);
    asm volatile("s_waitcnt vmcnt(24)" : "+v"(ordv) :: "memory");
    const char* const tw = (const char*)(((unsigned long long)xw[3] << 32) | xw[2]) + ((((1 - (g >> 1)) & 3) + (j >> 4) * 4) * 4 + 1) * 256 + ((j & 15) * 4 + lw) * (N * 2);
    unsigned to = (kh * NOWN + TD + (g >> 1 & 1)) * (64 * N * 2);
    u32x2 sink = {0u, 0u};
    for (int s = 0; s <= NF - 4; ++s) { wait_vm<16>(); bar(); LST(s); issue(s + 3, __builtin_amdgcn_readlane(ordv, s + 3));
      if (TD && s < NOWN - TD) { TOUCH(sink, tw, to); to += (s & 1) ? 2 * 64 * N * 2 - 128 : 128; } }
    wait_vm<16>(); bar(); LST(NF - 3); issue_t(NF);
    wait_vm<14>(); bar(); LST(NF - 2); issue_t(NF + 1);
    for (int s = NF - 1; s <= NK - 4; ++s) { wait_vm<12>(); bar(); LST(s); issue_t(s + 3); }
    wait_vm<12>(); bar(); LST(NK - 3);
    wait_vm<6>(); bar(); LST(NK - 2);
    wait_vm<0>(); bar(); LST(NK - 1);
    asm volatile("s_waitcnt lgkmcnt(0)" : "+s"(sink) :: "memory");
    char* const myws = P.myws;
    char* const myout = P.myout;
    unsigned* const tmo = (unsigned*)(myws + WS_TMO);
    while (lds_word(RED + 4) != 4) __builtin_amdgcn_s_sleep(3);
    call = lds_word(RED + 20);
    const unsigned long long t0 = __builtin_amdgcn_s_memrealtime();
    const unsigned o16 = lane * 16;
    const char* const gs = myws + WS_GRAN + (size_t)((call & 1) * NREP + (b & (NREP - 1))) * GP;
    char* const src = myws + WS_IN + (size_t)((call & 1) * NWG + b) * SLOT + lw * STRM;
    bool dead = false, wantg = lw < NPOLL;
    const unsigned tag2 = 1u + (call >> 1 & 1u);
    auto maxima = [&](u32x4 a) {
      if (!SP && __all((a[0] & a[1] & a[2] & a[3] & tag2) || o16 >= GRAN_SET)) { red[4] = wave_umax(max(max(a[0], a[1]), max(a[2], a[3]))) | 0x80000003u; wantg = false; STK(12); } };
    auto spoll = [&]() {
      u32x16 p, q;
      if (NG >= 4) asm volatile("s_load_dwordx16 %0, %2, 0x0 " GLC "\n\ts_load_dwordx16 %1, %2, 0x40 " GLC "\n\ts_waitcnt lgkmcnt(0)" : "=&s"(p), "=&s"(q) : "s"(gs + lw * 128) : "memory");
      else { asm volatile("s_load_dwordx16 %0, %1, 0x0 " GLC "\n\ts_waitcnt lgkmcnt(0)" : "=&s"(p) : "s"(gs) : "memory");
        if (NG == 1) for (int i = 0; i < 8; ++i) p[8 + i] = p[i];
        q = p; }
      unsigned long long all = ~0ull;
#pragma unroll
      for (int i = 0; i < 16; i += 2) all &= ((unsigned long long)p[i + 1] << 32 | p[i]) & ((unsigned long long)q[i + 1] << 32 | q[i]);
      if ((unsigned)all & (unsigned)(all >> 32) & tag2) {
        unsigned m = 0;
#pragma unroll
        for (int i = 0; i < 16; ++i) m = max(m, max(p[i], q[i]));
        m |= 0x80000003u;
        if (NPOLL == 1) red[4] = m;
        else { unsigned n, g;
          asm volatile("s_mov_b64 exec, 1\n\tds_max_u32 %2, %3 offset:24\n\tds_add_rtn_u32 %0, %2, %4 offset:28\n\tds_read_b32 %1, %2 offset:24\n\ts_mov_b64 exec, -1\n\ts_waitcnt lgkmcnt(0)" : "=&v"(n), "=&v"(g) : "v"(RED), "v"(m), "v"(1u) : "memory");
          if (__builtin_amdgcn_readfirstlane(n) == NPOLL - 1) red[4] = __builtin_amdgcn_readfirstlane(g); }
        wantg = false; STK(12); }
      else if (SS) __builtin_amdgcn_s_sleep(SS); };
    auto meanwhile = [&]() { if (SP) while (wantg) { spoll(); if (!__builtin_amdgcn_s_getreg(7 | 3 << 11)) break; } };
    const __amdgpu_buffer_rsrc_t rg = RSRC(gs, lw == 0 && !SP ? GRAN_SET : 0);
    unsigned mk;
    for (;;) {
      u32x4 a;
      asm volatile("buffer_load_dwordx4 %0, %2, %4, 0 offen sc0 sc1\n\tbuffer_load_dword %1, %3, %5, 0 offen sc0 sc1"
                   : "=&v"(a), "=&v"(mk) : "v"(o16), "v"(lane * 4), "s"(rg), "s"(RSRC(src, 256)) : "memory");
      meanwhile();
      asm volatile("s_waitcnt vmcnt(0)" : "+v"(a), "+v"(mk) :: "memory");
      if (wantg) maxima(a);
      if (__all(mk != 0)) break;
      if (expired(tmo, call, t0)) { dead = true; break; }
      if (!wantg) __builtin_amdgcn_s_sleep(PRESL);
    }
    const unsigned cnt = __popc(mk >> (32 - EV)) | __popc(mk << EV) << 16, incl = wave_scan(cnt), tot = __builtin_amdgcn_readlane((int)incl, 63);
    unsigned pos = incl - cnt;
    const unsigned len = (HDR + 2 * (tot & 0xffff) + RA - 1) & ~(RA - 1u), lenb = ((tot >> 16) + RB - 1) & ~(RB - 1u);
    const __amdgpu_buffer_rsrc_t ra = RSRC(src, len), rb_ = RSRC(src + STRA, lenb);
    u32x4 a, v[3], w[2];
    unsigned gmax = lds_word(RED + 16);
#define LOADS "buffer_load_dwordx4 %0, %6, %7, 0 offen sc0 sc1\n\tbuffer_load_dwordx4 %1, %6, %8, 0 offen sc0 sc1\n\tbuffer_load_dwordx4 %2, %6, %8, 0 offen offset:1024 sc0 sc1\n\t" \
              "buffer_load_dwordx4 %3, %6, %8, 0 offen offset:2048 sc0 sc1\n\tbuffer_load_dwordx4 %4, %6, %9, 0 offen sc0 sc1\n\t.if %10\n\tbuffer_load_dwordx4 %5, %6, %9, 0 offen offset:1024 sc0 sc1\n\t.endif"
#define LOADA : "=&v"(a), "=&v"(v[0]), "=&v"(v[1]), "=&v"(v[2]), "=&v"(w[0]), "=&v"(w[1]) : "v"(o16), "s"(rg), "s"(ra), "s"(rb_), "i"(EV < 16) : "memory"
    asm volatile(LOADS LOADA);
    if (PIN) asm volatile("" : "+v"(mk), "+v"(pos));
    const int sl = TP ? (lane & 3) << 4 | lane >> 2 : lane;
    if (TP) { mk = __builtin_amdgcn_ds_bpermute(sl * 4, mk); pos = __builtin_amdgcn_ds_bpermute(sl * 4, pos); }
    unsigned ca[EV], ba[8], bs[8]; int nt[EV];
    ca[0] = 2 * (pos & 0xffff) + (unsigned)(size_t)smem + INA + lw * STRL + HDR;
#pragma unroll
    for (int i = 0; i < EV; ++i) {
      nt[i] = __builtin_amdgcn_sbfe((int)mk, 31 - i, 1);
      if (i + 1 < EV) asm("v_mad_i32_i24 %0, %1, -2, %2" : "=v"(ca[i + 1]) : "v"(nt[i]), "v"(ca[i]));
      asm volatile("" : "+v"(ca[i]), "+v"(nt[i]));
    }
#pragma unroll
    for (int c = EV / 4; c < 8; ++c) {
      const unsigned o = (pos >> 16) + (c > EV / 4 ? __popc((mk << EV) >> (32 + EV - 4 * c)) : 0);
      const unsigned p = (((mk >> (28 - 4 * c)) & 15) * 0x08040201u >> 3) & 0x01010101u;
      ba[c] = (unsigned)(size_t)smem + INB + lw * 4096 + (o & ~3u);
      bs[c] = __builtin_amdgcn_bitop3_b32(p * 255u, p * 0x01010100u + (o & 3) * 0x01010101u, 0x0c0c0c0cu, 0xca);
      asm volatile("" : "+v"(ba[c]), "+v"(bs[c]));
    }
    char* mine = myout + (size_t)((b >> 2) * 64 + (lw & 1) * 32 + (sl & 15)) * NCOL + (b & 3) * 128 + (lw >> 1) * 64 + (sl >> 4) * 16;
    meanwhile();
    asm volatile("s_waitcnt vmcnt(0)" : "+v"(a), "+v"(v[0]), "+v"(v[1]), "+v"(v[2]), "+v"(w[0]), "+v"(w[1]), "+v"(mine) :: "memory");
    auto second = [&]() {
      *(LAS u32x4*)(smem + INB + lw * 4096 + o16) = w[0];
      bool ok = (int)(w[0][0] & w[0][1] & w[0][2] & w[0][3]) < 0 || o16 >= lenb;
      if (EV < 16) { *(LAS u32x4*)(smem + INB + lw * 4096 + 1024 + o16) = w[1]; ok &= (int)(w[1][0] & w[1][1] & w[1][2] & w[1][3]) < 0 || 1024 + o16 >= lenb; }
      return (bool)__all(ok); };
    while (!dead) {
      if (wantg) maxima(a);
      bool ok = true;
#pragma unroll
      for (int c = 0; c < 3; ++c) ok &= (min(min(v[c][0], v[c][1]), min(v[c][2], v[c][3])) != 0) | (c * 1024 + o16 >= len);
      if (__all(ok)) break;
      dead = expired(tmo, call, t0);
      if (!wantg) __builtin_amdgcn_s_sleep(PRESL);
      gmax = lds_word(RED + 16);
      asm volatile(LOADS LOADA);
      meanwhile();
      asm volatile("s_waitcnt vmcnt(0)" : "+v"(a), "+v"(v[0]), "+v"(v[1]), "+v"(v[2]), "+v"(w[0]), "+v"(w[1]) :: "memory");
    }
#pragma unroll
    for (int c = 0; c < 3; ++c) *(LAS u32x4*)(smem + INA + lw * STRL + c * 1024 + o16) = v[c];
    bool done = second();
    const bool late = gmax != 0;
    const u32x4 z = {0u, 0u, 0u, 0u};
    STK(11);
    asm volatile("buffer_load_dwordx4 %0, %3, %4, 0 offen sc0 sc1\n\tbuffer_load_dwordx4 %1, %3, %5, 0 offen sc0 sc1\n\t.if %6\n\tbuffer_load_dwordx4 %2, %3, %5, 0 offen offset:1024 sc0 sc1\n\t.endif"
                 : "=&v"(a), "=&v"(w[0]), "=&v"(w[1]) : "v"(o16), "s"(rg), "s"(RSRC(src + STRA, late && !done ? lenb : 0)), "i"(EV < 16) : "memory");
    STORE3(z, z, z, RSRC(src, late ? 0 : len));
    const float wmf = *(LAS float*)(smem + INA + lw * STRL + 256);
    f32x2 f[EV / 2];
#pragma unroll
    for (int i = 0; i < EV; ++i) f[i >> 1][i & 1] = __uint_as_float(__builtin_amdgcn_bitop3_b32(*(LAS unsigned short*)(size_t)ca[i], nt[i], 0x4b000000u, 0xea));
#pragma unroll
    for (int i = 0; i < EV / 2; ++i) { f[i] -= 8388609.0f; asm volatile("" : "+v"(f[i])); }
    if (SP) while (wantg) { spoll(); if (wantg && (dead || expired(tmo, call, t0))) { red[4] = 0xffffffffu; break; } }
    else if (wantg) {
      asm volatile("s_waitcnt vmcnt(3)" : "+v"(a) :: "memory");
      maxima(a);
      while (wantg) {
        if (dead || expired(tmo, call, t0)) { red[4] = 0xffffffffu; break; }
        u32x4 a2;
        asm volatile("buffer_load_dwordx4 %0, %1, %2, 0 offen sc0 sc1\n\ts_waitcnt vmcnt(0)" : "=&v"(a2) : "v"(o16), "s"(rg) : "memory");
        maxima(a2);
      }
    }
    while (!gmax && !(gmax = lds_word(RED + 16))) __builtin_amdgcn_s_sleep(1);
    dead |= gmax == 0xffffffffu;
    STK(7);
    unsigned q[8];
    auto row = [&](int m) {
      u32x4 r;
#pragma unroll
      for (int c = 0; c < 4; ++c) r[c] = dead ? 0x80808080u : (sl & 16) ? q[4 * m + (c ^ 1)] : q[4 * m + c];
      if (FB == 0 || (FB == 2 && m == 1)) st16_sys(mine + m * 16 * NCOL, r);
      else if (FB == 3) asm volatile("global_store_dwordx4 %0, %1, off nt\n\ts_nop 1" :: "v"(mine + m * 16 * NCOL), "v"(r) : "memory");
      else asm volatile("global_store_dwordx4 %0, %1, off\n\ts_nop 1" :: "v"(mine + m * 16 * NCOL), "v"(r) : "memory"); };
    { const float kq = wmf / fmaxf(__uint_as_float(gmax & 0x7fffffffu), 0x1p-110f) * (127.0f / 65534.0f);
#pragma unroll
      for (int i = 0; i < EV / 2; ++i) f[i] = __builtin_elementwise_fma(f[i], (f32x2){kq, kq}, (f32x2){12582912.0f, 12582912.0f});
#pragma unroll
      for (int c = 0; c < EV / 4; ++c) { q[c] = pack4(f[2 * c], f[2 * c + 1]); asm volatile("" : "+v"(q[c])); } }
    STK(8);
    asm volatile("s_waitcnt vmcnt(0)" : "+v"(a), "+v"(w[0]), "+v"(w[1]) :: "memory");
    if (late && !done) done = second();
    if (EV >= 16 && !late) row(0);
    int npoll = 0;
    while (!dead && !done) {
      asm volatile("buffer_load_dwordx4 %0, %2, %3, 0 offen sc0 sc1\n\t.if %4\n\tbuffer_load_dwordx4 %1, %2, %3, 0 offen offset:1024 sc0 sc1\n\t.endif\n\ts_waitcnt vmcnt(0)"
                   : "=&v"(w[0]), "=&v"(w[1]) : "v"(o16), "s"(rb_), "i"(EV < 16) : "memory");
      ++npoll;
      done = second();
      if (!done) dead = expired(tmo, call, t0);
    }
    asm volatile("" ::: "memory");
#pragma unroll
    for (int c = EV / 4; c < 8; ++c) { const unsigned lo = ((LAS unsigned*)(size_t)ba[c])[0], hi = ((LAS unsigned*)(size_t)ba[c])[1];
      q[c] = __builtin_amdgcn_perm(hi, lo, bs[c]) & 0x7f7f7f7fu; }
    if (EV < 16 || late) row(0);
    row(1);
    asm volatile("buffer_store_dwordx4 %0, %1, %2, 0 offen sc0 sc1\n\t.if %3\n\tbuffer_store_dwordx4 %0, %1, %2, 0 offen offset:1024 sc0 sc1\n\t.endif\n\ts_nop 1" :: "v"(z), "v"(o16), "s"(rb_), "i"(EV < 16) : "memory");
    if (late) STORE3(z, z, z, ra);
    STK(9);
    STW(7, 256); STW(8, 256); STW(9, 256); STW(11, 320); STW(12, 256);
    TSW(11, ta); TSW(12, tb); TSW(13, tc); TSW(14, td); TSW(15, te);
  } else {
    const int rank = P.rank;
    char* const myws = P.myws;
    unsigned* const tmo = (unsigned*)(myws + WS_TMO);
    STAMP(0);
    const unsigned long long t0 = __builtin_amdgcn_s_memrealtime();
    unsigned cnt0;
    asm volatile("global_load_dword %0, %1, off sc0 sc1" : "=v"(cnt0) : "v"((unsigned*)(myws + WS_CNT) + b) : "memory");
    if (tid < 8) red[tid] = 0;
    char* pw = P.ws[0];
#pragma unroll
    for (int i = 1; i < NR; ++i) { char* q = P.ws[i]; asm volatile("" : "+s"(q)); pw = (lane >> 3) == i ? q : pw; }
    pw += WS_GRAN + (size_t)(lane & 7) * GP + (size_t)(rank * NGRP + (b & (NGRP - 1))) * 4;
    char* dslot = P.ws[((unsigned)cb >> 2) & (NR - 1)] + WS_IN + (size_t)(rank * 32 + (rb * 2 + kh) * 4 + (cb & 3)) * SLOT + wave * STRM;
    asm volatile("" : "+v"(pw));
    asm volatile("" : "+s"(dslot));
    const int wm = wave & 1, wn = wave >> 1;
    const int fast = ((cb & 3) ^ 1) * (cb ^ 31);
    unsigned ao[2][2], bo[4][2];
    { const int r = lane & 15, q = lane >> 4;
#pragma unroll
      for (int h = 0; h < 2; ++h)
#pragma unroll
        for (int ks = 0; ks < 2; ++ks) ao[h][ks] = ((h ^ kh) * 64 + wm * 32 + r) * 128 + (((4 * ks + q) ^ ((r >> 1) & 7)) << 4);
      const int qq = (lane & 15) >> 2, p = lane & 3;
#pragma unroll
      for (int n = 0; n < 4; ++n)
#pragma unroll
        for (int t = 0; t < 2; ++t) { const int row = 8 * q + 4 * t + qq, ch = 8 * wn + 2 * p + (n >> 1);
          bo[n][t] = WREG + 256 * row + 16 * (ch ^ (row & 11)) + 8 * ((n ^ p) & 1); }
    }
    f32x4 acc[4][4];
#pragma unroll
    for (int m = 0; m < 4; ++m)
#pragma unroll
      for (int n = 0; n < 4; ++n) acc[m][n] = (f32x4){0.f, 0.f, 0.f, 0.f};
    Frags f0, f1;
    auto rd = [&](Frags& f, int r, int st, int ks, bool whole) {
      constexpr int RW[12] = {0, 1, 8, 9, 2, 3, 10, 4, 5, 11, 6, 7}, RH[12] = {0, 1, 8, 9, 2, 3, 4, 5, 6, 7, -1, -1};
      const int c = r < 0 || r > 11 ? -1 : whole ? RW[r] : RH[r];
      if (c >= 8) f.a[c - 8] = *(LAS bf16x8*)(smem + ao[(c - 8) >> 1][ks] + (c & 1) * 2048 + st * TB);
      else if (c >= 0) f.b[c >> 1][c & 1] = __builtin_amdgcn_ds_read_tr16_b64_v4i16((LAS bf16x4*)(smem + bo[c >> 1][c & 1] + st * TB + ks * 8192));
    };
#define BF(f, n) __builtin_shufflevector(f.b[n][0], f.b[n][1], 0, 1, 2, 3, 4, 5, 6, 7)
    auto half = [&](Frags& fc, Frags& fn, const int st, const int ks, const bool CF, const int RD, auto&& extra) {
      constexpr int OM[32] = {0, 1, 0, 1, 2, 2, 0, 1, 2, 3, 3, 3, 0, 1, 2, 3,   2, 2, 0, 1, 2, 3, 3, 3, 2, 3, 0, 1, 0, 1, 0, 1};
      constexpr int ON[32] = {0, 0, 1, 1, 0, 1, 2, 2, 2, 0, 1, 2, 3, 3, 3, 3,   0, 1, 0, 0, 2, 0, 1, 2, 3, 3, 1, 1, 2, 2, 3, 3};
      if (CF) {
        const int r0 = ks ? RS : BP + 1, o = RD == 2 ? 0 : 16;
#pragma unroll
        for (int i = 0; i < 16; ++i) {
          MFMA(acc[OM[o + i]][ON[o + i]], BF(fc, ON[o + i]), fc.a[OM[o + i]]);
          SB0();
          if (!ks && i == BP) {
            wait_lgkm0(); bar(); }
          rd(fn, i - r0, st, ks, RD == 2);
          extra(i);
          if (EW && i == 15) { if (ks) __builtin_amdgcn_s_waitcnt(0xC07F | 5 << 8); else __builtin_amdgcn_s_waitcnt(0xC07F | EW << 8); }
          SB0();
        }
      } else {
        if (!ks && RD) { wait_lgkm0(); bar(); SB0(); }
#pragma unroll
        for (int i = 0; i < 8; ++i) {
          if (RD) { rd(fn, 2 * i, st, ks, false); rd(fn, 2 * i + 1, st, ks, false); }
          MFMA(acc[i & 1][i >> 1], BF(fc, i >> 1), fc.a[i & 1]);
          extra(i);
          SB0();
        }
      }
    };
    auto none = [](int) {};
    wait_lgkm0(); bar();
#pragma unroll
    for (int i = 0; i < 12; ++i) rd(f0, i, ST(0), 0, true);
    STAMP(1);
    SB0();
    half(f0, f1, ST(0), 1, true, 2, none);
    constexpr int NL = (NF - 1) / NS * NS;
    const unsigned o16 = lane * 16;
    for (int s = 0; s < NL; s += NS) {
#pragma unroll
      for (int u = 1; u <= NS; ++u) { half(f1, f0, ST(u), 0, true, 2, none); half(f0, f1, ST(u), 1, true, 2, none);
      }
    }
#pragma unroll
    for (int s = NL + 1; s < NF; ++s) { half(f1, f0, ST(s), 0, true, 2, none); half(f0, f1, ST(s), 1, true, 2, none); }
    asm volatile("s_waitcnt vmcnt(0)\n\tv_readfirstlane_b32 %0, %1" : "=s"(call) : "v"(cnt0) : "memory");
    call += 1;
    const unsigned tag = 1u + (call & 1u);
    { const char* dst = myws + WS_XCH + ((b ^ 1) * 4 + wave) * XW + 2048;
      float am = 0x1p-110f, ra;
      u32x4 cw; unsigned odd;
      auto give = [&](int t) {
        constexpr int RM[12] = {-1, -1, 0, 1, -1, -1, 2, 4, 5, 6, 3, 7};
        if (t < 12 && RM[t] >= 0) { const f32x4 y = acc[2 + (RM[t] >> 2)][RM[t] & 3];
          am = fmaxf(fmaxf(am, fabsf(y[0])), fabsf(y[1])); am = fmaxf(fmaxf(am, fabsf(y[2])), fabsf(y[3])); }
        if (t == 11) { ra = __builtin_amdgcn_rcpf(am); cw[3] = (__float_as_uint(am * (1.0f / 32767.0f)) & ~3u) | tag; }
#pragma unroll
        for (int w = 0; w < 16; ++w) if (t == 12 + (w == 15 ? 3 : w + (w >= 3)) * XN / 4) {
          const f32x4 y = acc[2 + (w >> 3)][w >> 1 & 3];
          const f32x2 u = (f32x2){y[(w & 1) * 2], y[(w & 1) * 2 + 1]} * ra;
          const unsigned c = __builtin_bit_cast(unsigned, __builtin_amdgcn_cvt_pknorm_i16(u[0], u[1]));
          if (w == 15) odd = c; else cw[w % 3] = c;
          if (w % 3 == 2) {
            if (w > 2) cw[3] = w == 5 ? odd << 16 | tag : w == 8 ? (odd & 0xffff0000u) | tag : tag;
            asm volatile("global_store_dwordx4 %0, %1, %2 offset:%3 " SB "\n\ts_nop 1" :: "v"(o16), "v"(cw), "s"(dst), "i"(w / 3 * 1024 - 2048) : "memory"); }
        } };
      half(f1, f0, ST(NF), 0, true, 1, give);
      half(f0, f1, ST(NF), 1, false, 1, [&](int i) { give(16 + i); }); }
    STK(14);
    if (tid == 0) {
      unsigned* cnt = (unsigned*)(myws + WS_CNT) + b;
      asm volatile("global_store_dword %0, %1, off sc0 sc1" :: "v"(cnt), "v"(call) : "memory");
      if (b < NGRP) {
        char* nx = myws + WS_GRP + (size_t)(((call + 1) & 1) * NGRP + b) * GL + 4;
        asm volatile("global_store_dword %0, %1, off sc0 sc1" :: "v"(nx), "v"(0u) : "memory"); }
    }
    const char* src = myws + WS_XCH + (b * 4 + wave) * XW + 2048;
    u32x4 v[5];
#define XLOAD "global_load_dwordx4 %0, %8, %10 offset:-2048 " XB "\n\tglobal_load_dwordx4 %1, %8, %10 offset:-1024 " XB "\n\t" \
              "global_load_dwordx4 %2, %8, %10 " XB "\n\tglobal_load_dwordx4 %3, %8, %10 offset:1024 " XB "\n\tglobal_load_dwordx4 %4, %8, %10 offset:2048 " XB
    unsigned probe, asked = 0, tmp;
#define PROBE "v_mov_b32 %5, 3"
#define XTRY "s_cmp_lg_u32 %6, 0\n\ts_cbranch_scc1 1f\n\ts_getreg_b32 %7, hwreg(7, 0, 4)\n\ts_cmp_lg_u32 %7, 0\n\ts_cbranch_scc1 1f\n\t" \
             "v_readfirstlane_b32 %7, %5\n\ts_and_b32 %6, %7, %9\n\ts_cbranch_scc1 2f\n\t" PROBE "\n\ts_branch 1f\n2:\n\t" XLOAD "\n1:"
#define XARG(m) : m(v[0]), m(v[1]), m(v[2]), m(v[3]), m(v[4]), m(probe), "+s"(asked), "=&s"(tmp) : "v"(o16), "s"(tag), "s"(src), "v"(0u) : "memory", "scc"
    auto xtry = [&](int h, int i) {
      if (i == 0 && h == XL + XD) asm volatile(PROBE XARG("=&v"));
      if (PK == 4) { if (i == XJ && h == XL) asm volatile("s_mov_b32 %6, %9\n\t" XLOAD XARG("+v")); }
      else if ((i & XI) == 0 && h <= XL) {
        asm volatile(XTRY XARG("+v"));
      } };
#pragma unroll
    for (int s = NF + 1; s < NK; ++s) {
      half(f1, f0, ST(s), 0, false, 1, [&](int i) { xtry(2 * (NK - s), i); });
      half(f0, f1, ST(s), 1, false, 1, [&](int i) { xtry(2 * (NK - s) - 1, i); }); }
    unsigned all; f32x2 pv[16];
    auto take = [&](int i) {
      if (i < 2) all &= v[2 * i + 1][3] & v[2 * i + 2][3];
#pragma unroll
      for (int w = 2 * i; w < 2 * i + 2; ++w) {
        if (w < 15) { const int c = (int)v[w / 3][w % 3]; pv[w] = (f32x2){(float)(short)c, (float)(c >> 16)}; }
        else pv[w] = (f32x2){(float)((int)v[1][3] >> 16), (float)((int)v[2][3] >> 16)}; } };
    half(f1, f0, 0, 0, false, 0, [&](int i) {
      if (i == 0) { const unsigned was = __builtin_amdgcn_s_getreg(7 | 3 << 11) ? 0u : asked; asm volatile("" ::: "memory"); xtry(0, 0); all = was & v[0][3]; }
      if (XS) { take(i); asm volatile("" : "+v"(pv[2 * i]), "+v"(pv[2 * i + 1]), "+v"(all)); } });
    STK(2);
    bool dead = false;
    if (!XS || __builtin_expect(!__all(all != 0), 0)) {
      asm volatile("s_waitcnt vmcnt(0)" XARG("+v"));
      all = asked & v[0][3];
#pragma unroll
      for (int i = 0; i < 8; ++i) take(i);
      while (!__all(all != 0)) {
        while (!(dead = expired(tmo, call, t0)) && !(__builtin_amdgcn_readfirstlane(probe) & tag))
          asm volatile("global_load_dword %0, %1, %2 offset:3068 " PB "\n\ts_waitcnt vmcnt(0)" : "=&v"(probe) : "v"(0u), "s"(src) : "memory");
        if (dead) break;
        asm volatile(XLOAD "\n\ts_waitcnt vmcnt(0)" : "=&v"(v[0]), "=&v"(v[1]), "=&v"(v[2]), "=&v"(v[3]), "=&v"(v[4]) : "i"(0), "i"(0), "i"(0), "v"(o16), "i"(0), "s"(src) : "memory");
        all = tag & v[0][3];
#pragma unroll
        for (int i = 0; i < 8; ++i) take(i);
      }
    }
    const float vs = __uint_as_float(v[0][3]);
#pragma unroll
    for (int w = 0; w < 16; ++w) {
      f32x4& y = acc[w >> 3][w >> 1 & 3];
      const f32x2 r = __builtin_elementwise_fma(pv[w], (f32x2){vs, vs}, (f32x2){y[(w & 1) * 2], y[(w & 1) * 2 + 1]});
      y[(w & 1) * 2] = r[0]; y[(w & 1) * 2 + 1] = r[1];
    }
    STK(3);
    float mx = 0.f;
#pragma unroll
    for (int i = 0; i < 32; i += 2) mx = fmaxf(fmaxf(mx, acc[i >> 4][i >> 2 & 3][i & 3]), acc[i >> 4][i >> 2 & 3][(i & 3) + 1]);
    const unsigned wmx = wave_umax(__float_as_uint(mx));
    unsigned wn_ = 0, wgm = 0;
    if (lane == 0)
      asm volatile("ds_write_b32 %2, %5 offset:20\n\tds_max_u32 %2, %3\n\tds_add_rtn_u32 %0, %2, %4 offset:4\n\tds_read_b32 %1, %2" : "=&v"(wn_), "=&v"(wgm) : "v"(RED), "v"(dead ? 0xffffffffu : wmx), "v"(1u), "v"(call) : "memory");
    unsigned mi = 0;
#pragma unroll
    for (int i = 0; i < 32; ++i) mi = __builtin_amdgcn_alignbit(mi, __float_as_uint(acc[i >> 4][i >> 2 & 3][i & 3]), 31);
    asm volatile("s_waitcnt lgkmcnt(0)" : "+v"(wn_), "+v"(wgm), "+v"(mi) :: "memory");
    const bool last = __builtin_amdgcn_readfirstlane(wn_) == 3;
    u32x2 arr;
    if (last && lane == 0) {
      char* const grp = myws + WS_GRP + (size_t)((call & 1) * NGRP + (b & (NGRP - 1))) * GL;
      const u32x2 one = {1u, 0u};
      asm volatile("global_atomic_umax %1, %2, off offset:4 " AB "\n\tglobal_atomic_add_x2 %0, %1, %3, off sc0 " AB : "=&v"(arr) : "v"(grp), "v"(wgm), "v"(one) : "memory");
    }
    if (mi == 0xffffffffu) { mi = 0xfffffffeu; acc[1][3][3] = 0.f; }
    const unsigned cnt = __popc(~mi >> (32 - EV)) | __popc(~mi << EV) << 16, incl = wave_scan(cnt), tot = __builtin_amdgcn_readlane((int)incl, 63);
    const unsigned len = HDR + 2 * (tot & 0xffff), pos = incl - cnt;
    const float wmf = fmaxf(__uint_as_float(wmx), 0x1p-110f), ks = 65534.0f / wmf;
    const unsigned oa = OUTA + wave * STRL, ob = OUTB + wave * 8192;
    *(LAS unsigned*)(smem + oa + lane * 4) = ~mi;
    *(LAS float*)(smem + oa + 256) = wmf;
    *(LAS unsigned short*)(smem + oa + len + lane * 2) = 0xffff;
    { unsigned a = (unsigned)(size_t)smem + oa + HDR + 2 * (pos & 0xffff);
      const unsigned sink = (unsigned)(size_t)smem + oa + 2960;
#pragma unroll
      for (int i = 0; i < EV; i += 2) {
        const f32x2 y = {acc[i >> 4][i >> 2 & 3][i & 3], acc[i >> 4][i >> 2 & 3][(i & 3) + 1]};
        const f32x2 f = __builtin_elementwise_fma(y, (f32x2){ks, ks}, (f32x2){8388609.0f, 8388609.0f});
#pragma unroll
        for (int h = 0; h < 2; ++h) {
          const unsigned t = (unsigned)min(max((int)__float_as_uint(y[h]), -2), 0);
          ((LAS unsigned short*)(size_t)__builtin_amdgcn_bitop3_b32(t, sink, a, 0xca))[i + h] = (unsigned short)__float_as_uint(f[h]);
          a += t;
        }
      } }
    asm volatile("" ::: "memory");
    char* const dst = dslot + (size_t)(call & 1) * NWG * SLOT;
    f32x4 yb[4];
    { u32x4 v3[3];
#pragma unroll
      for (int c = 0; c < 3; ++c) v3[c] = *(LAS u32x4*)(smem + oa + c * 1024 + o16);
      if (last) {
        asm volatile("s_waitcnt vmcnt(0)" : "+v"(arr) :: "memory");
        const unsigned n = __builtin_amdgcn_readfirstlane(arr[0]), gm = __builtin_amdgcn_readfirstlane(arr[1]);
        if ((n & (GSZ - 1)) == GSZ - 1) {
          const unsigned gr = (gm & ~3u) | (1u + (call >> 1 & 1u));
#pragma unroll
          for (int k = 0; k < (NREP + 7) / 8; ++k)
            if (NREP >= 8 || (lane & 7) < NREP) asm volatile("global_store_dword %0, %1, off offset:%2 sc0 sc1" :: "v"(pw + (size_t)(call & 1) * NREP * GP), "v"(gr), "i"(k * 8 * GP) : "memory");
        }
      }
      const __amdgpu_buffer_rsrc_t rd = RSRC(dst, (len + RA - 1) & ~(RA - 1u));
      if (o16 < PACEB) asm volatile("buffer_store_dwordx4 %0, %1, %2, 0 offen sc0 sc1\n\ts_nop 1" :: "v"(v3[0]), "v"(o16), "s"(rd) : "memory");
      *(LAS float*)(smem + ob + ((tot >> 16) + lane) * 4) = 0.f;
      if (RB > 64) *(LAS float*)(smem + ob + ((tot >> 16) + lane + 64) * 4) = 0.f;
      { unsigned a = (unsigned)(size_t)smem + ob + 4 * (pos >> 16);
        const unsigned sink = (unsigned)(size_t)smem + ob + 8192 - 128;
#pragma unroll
        for (int i = EV; i < 32; ++i) {
          const float y = acc[i >> 4][i >> 2 & 3][i & 3];
          const unsigned t = (unsigned)min(max((int)__float_as_uint(y), -4), 0);
          ((LAS float*)(size_t)__builtin_amdgcn_bitop3_b32(t, sink, a, 0xca))[i - EV] = y;
          a += t;
        } }
      asm volatile("" ::: "memory");
#pragma unroll
      for (int c = 0; c < 4; ++c) yb[c] = *(LAS f32x4*)(smem + ob + lane * 64 + c * 16);
      const unsigned len16 = (len + RA - 1) & ~(RA - 1u);
      for (unsigned lo = PACEB; lo < len16; ) {
        unsigned known;
        while (!(known = lds_word(RED + 16)) && __builtin_amdgcn_s_getreg(7 | 3 << 11)) { }
        const unsigned hi = known ? len16 : min(lo + PACEC, len16);
        const __amdgpu_buffer_rsrc_t rs = RSRC(dst, hi);
        if (lo < 1024 && o16 >= lo) asm volatile("buffer_store_dwordx4 %0, %1, %2, 0 offen sc0 sc1\n\ts_nop 1" :: "v"(v3[0]), "v"(o16), "s"(rs) : "memory");
        if (hi > 1024 && lo < 2048 && o16 + 1024 >= lo) asm volatile("buffer_store_dwordx4 %0, %1, %2, 0 offen offset:1024 sc0 sc1\n\ts_nop 1" :: "v"(v3[1]), "v"(o16), "s"(rs) : "memory");
        if (hi > 2048 && o16 + 2048 >= lo) asm volatile("buffer_store_dwordx4 %0, %1, %2, 0 offen offset:2048 sc0 sc1\n\ts_nop 1" :: "v"(v3[2]), "v"(o16), "s"(rs) : "memory");
        lo = hi;
      }
    }
    asm volatile("" : "+v"(yb[0]), "+v"(yb[1]), "+v"(yb[2]), "+v"(yb[3]));
    STK(4);
    unsigned gmax;
    while (!(gmax = lds_word(RED + 16))) { }
    STK(5);
    if (gmax != 0xffffffffu) {
      const float inv = 127.0f / fmaxf(__uint_as_float(gmax & 0x7fffffffu), 0x1p-110f);
      const __amdgpu_buffer_rsrc_t rd = RSRC(dst + STRA, ((tot >> 16) + RB - 1) & ~(RB - 1u));
      auto quant = [&]() {
        u32x4 q;
#pragma unroll
        for (int c = 0; c < 4; ++c)
          q[c] = pack4(__builtin_elementwise_fma((f32x2){yb[c][0], yb[c][1]}, (f32x2){inv, inv}, (f32x2){12583040.0f, 12583040.0f}),
                       __builtin_elementwise_fma((f32x2){yb[c][2], yb[c][3]}, (f32x2){inv, inv}, (f32x2){12583040.0f, 12583040.0f}));
        return q; };
      asm volatile("buffer_store_dwordx4 %0, %1, %2, 0 offen sc0 sc1\n\ts_nop 1" :: "v"(quant()), "v"(o16), "s"(rd) : "memory");
      if (EV < 16 && (tot >> 16) > 1024) {
#pragma unroll
        for (int c = 0; c < 4; ++c) yb[c] = *(LAS f32x4*)(smem + ob + 4096 + lane * 64 + c * 16);
        asm volatile("buffer_store_dwordx4 %0, %1, %2, 0 offen offset:1024 sc0 sc1\n\ts_nop 1" :: "v"(quant()), "v"(o16), "s"(rd) : "memory");
      }
    }
    STK(6);
    STW(14, 0); STW(15, 0); STW(2, 0); STW(3, 0); STW(4, 0); STW(5, 0); STW(6, 0);
  }
}
}

extern "C" void kernel_launch(int rank, int nranks, ncclComm_t comm, hipStream_t stream,
                              void* const* d_in, const int* in_sizes, int n_in,
                              void* d_out, int out_size, const int* in_dtypes, int out_dtype,
                              void* const* d_out_all, void* const* d_ws_all, size_t ws_bytes) {
  (void)comm; (void)in_dtypes; (void)out_dtype; (void)d_out;
  if (nranks != NR || n_in < 2 || in_sizes[0] != MS * K || in_sizes[1] != K * N || out_size != NR * MS * NCOL || ws_bytes < WS_NEED || (unsigned)rank >= (unsigned)NR) {
    fprintf(stderr, "kernel_launch: unexpected problem shape\n");
    return;
  }
  HIP_CHECK(hipFuncSetAttribute((const void*)fused_kernel, hipFuncAttributeMaxDynamicSharedMemorySize, LDS_BYTES));
  Params P;
  P.x = (const bf16_t*)d_in[0]; P.w = (const bf16_t*)d_in[1];
  for (int p = 0; p < NR; ++p) { P.out[p] = (char*)d_out_all[p]; P.ws[p] = (char*)d_ws_all[p]; }
  P.myws = P.ws[rank]; P.myout = P.out[rank];
  P.rank = rank; P.pad = 0;
  int A0 = 30, AS = 3, A1 = 80, RO = 2;
  int DROP = 1, D0 = 0, D1 = 0, DT = 8, PS = 4, E0 = 0, E1 = 21;
  for (int c = 0; c < 32; ++c) {
    const int l0 = c & 7, l1 = (l0 + 1) & 7, kh = c >> 3 & 1;
    int n = 0, key[NOWN + 8], u = -4;
    for (int t = 0; t < NOWN + 8; ++t) {
      const int v = (u + RO * (u >> 3) + (NX == 2 && t >= NOWN ? kh * PS : 0)) & 7;
      const bool lead = t < 4 ? t == (l0 >> 1) : v == l0 || (v == l1 && !(DROP && TD && c >= 16 && t >= TD && t < NOWN));
      const int a = A0 + (t - 4) * AS < A1 ? A0 + (t - 4) * AS : A1;
      key[t] = lead ? t * 10 - (t < 4 ? 100 : a) - 1 : t * 10;
      if (lead && t >= DT && t < NOWN) key[t] -= v == l0 ? D0 : D1;
      if (lead && key[t] > NOWN * 10 - 11) key[t] = NOWN * 10 - 11;
      if (NX == 1 && t >= NOWN && v == (kh ? l0 : l1)) key[t] = -1000;
      if (NX == 2 && t >= NOWN && ((v + (kh ? -PS : PS) - l0) & 7) < 2) key[t] = -1000;
      if (NX == 2 && t >= NOWN && lead && E0) key[t] = NOWN * 10 - (v == l0 ? E0 : E1);
      ++u;
    }
    for (int k = -300; k < (NOWN + 8) * 10; ++k)
      for (int t = 0; t < NOWN + 8; ++t) if (key[t] == k) P.ord[c][n++] = t < NOWN ? kh * NOWN + t : NOWN + t;
    while (n < 64) P.ord[c][n++] = 0;
  }
  hipLaunchKernelGGL(fused_kernel, dim3(NWG), dim3(NTHR), LDS_BYTES, stream, P);
  HIP_CHECK(hipGetLastError());
}
```
